```python
import numpy as np

kernel_name = "gpu_a2a_gemm_m4096_k4096_n4096_w8a8_dqrelu"


ABSMAX_RTOL = 0.001

COMPUTE_DTYPE = "int8"
EPILOGUE = "dequant_relu"
IN_QUANT = "int8"
OUT_QUANT = None
IN_DTYPES = {"x": "int8", "w_mat": "int8",
             "scale_x": "float32", "scale_w": "float32"}
WORLD_SIZE = 8

def setup_inputs():

    x = np.random.randint(-127, 128, size=(4096, 4096)).astype(np.int8)
    w_mat = np.random.randint(-127, 128, size=(4096, 4096)).astype(np.int8)
    scale_x = (0.02 * (1.0 + np.random.rand(1))).astype(np.float32)
    scale_w = (0.02 * (1.0 + np.random.rand(1))).astype(np.float32)
    return {"x": x, "w_mat": w_mat, "scale_x": scale_x, "scale_w": scale_w}

def reference(x, w_mat, scale_x, scale_w):
    acc = x.astype(np.float64) @ w_mat.astype(np.float64)
    y = (acc.astype(np.float32) * (scale_x * scale_w)).astype(np.float32)
    return np.maximum(y, 0.0).astype(np.float32)
```

```cpp
#include <hip/hip_runtime.h>
#include <rccl/rccl.h>
#include <cstdio>
#include <type_traits>

#define HIP_CHECK(expr) do { hipError_t _e = (expr); if (_e != hipSuccess) { \
    fprintf(stderr, "HIP error %s at %s:%d\n", hipGetErrorString(_e), __FILE__, __LINE__); \
    return; } } while (0)

namespace {

typedef int v2i __attribute__((ext_vector_type(2)));
typedef int v4i __attribute__((ext_vector_type(4)));
typedef unsigned u32x4 __attribute__((ext_vector_type(4)));
typedef float v4f __attribute__((ext_vector_type(4)));
#define LDSP __attribute__((address_space(3)))

constexpr int NR = 8;
constexpr int MS = 512;
constexpr int KC = 512;
constexpr int NN = 4096;
constexpr int NMT = 7;
constexpr int NSEND_WG = 32;

constexpr unsigned ABUF_BYTES = 2u << 20;
constexpr unsigned FLAG_OFF = 2 * ABUF_BYTES;
constexpr unsigned CNT_OFF = FLAG_OFF + 8 * 8 * 8 * 128;
constexpr unsigned TMO_OFF = CNT_OFF + 256 * 128;
constexpr unsigned DIAG_OFF = TMO_OFF + 128;
constexpr unsigned WS_USED = DIAG_OFF + 256 * 64 * 8;

#define STAGGER_TICKS 8
constexpr unsigned long long DEADLINE_TICKS = 50000000ull;

#define NA_ 4
#define NB_ 7
constexpr int NA = NA_, NB = NB_;
constexpr unsigned BSLOT = 16384, ASLOT = 10240;
constexpr unsigned LDS_B = 0, LDS_A = NB * BSLOT, LDS_CTL = LDS_A + NA * ASLOT;
constexpr unsigned LDS_DIAG = LDS_CTL + 64;
constexpr int POLLW = 4;
constexpr unsigned LDS_BYTES = LDS_DIAG + 512;

struct Params {
  const signed char* x;
  const signed char* w;
  const float* sx;
  const float* sw;
  float* out;
  unsigned char* myws;
  unsigned char* ws[NR];
  int rank;
  int pad;
};

template <int N> __device__ __forceinline__ void wait_vm() { asm volatile("s_waitcnt vmcnt(%0)" :: "i"(N) : "memory"); }
template <int N, int I = 0, class F> __device__ __forceinline__ void static_for(F&& f) {
  if constexpr (I < N) { f(std::integral_constant<int, I>{}); static_for<N, I + 1>(f); }
}
#define SPINS 1
#define SPIN_SLEEP() __builtin_amdgcn_s_sleep(SPINS)
#define SCHED() __builtin_amdgcn_sched_barrier(0)

__device__ __forceinline__ u32x4 make_srd(const void* p, unsigned bytes) {
  unsigned long long a = (unsigned long long)p;
  u32x4 r;
  r[0] = __builtin_amdgcn_readfirstlane((unsigned)a);
  r[1] = __builtin_amdgcn_readfirstlane((unsigned)(a >> 32) & 0xffffu);
  r[2] = bytes;
  r[3] = 0x00020000u;
  return r;
}
__device__ __forceinline__ void dma16(u32x4 srd, unsigned voff, unsigned soff, unsigned lds) {
  asm volatile("s_mov_b32 m0, %3\n\ts_nop 0\n\tbuffer_load_dwordx4 %0, %1, %2 offen lds"
               :: "v"(voff), "s"(srd), "s"(soff), "s"(lds) : "memory");
}
__device__ __forceinline__ void dma16_sys(u32x4 srd, unsigned voff, unsigned soff, unsigned lds) {
  asm volatile("s_mov_b32 m0, %3\n\ts_nop 0\n\tbuffer_load_dwordx4 %0, %1, %2 offen sc0 sc1 lds"
               :: "v"(voff), "s"(srd), "s"(soff), "s"(lds) : "memory");
}
__device__ __forceinline__ void load4_sys(unsigned& v, const void* ptr) {
  asm volatile("global_load_dword %0, %1, off sc0 sc1" : "+v"(v) : "v"(ptr) : "memory");
}
__device__ __forceinline__ unsigned min4(u32x4 v) {
  return __builtin_amdgcn_readfirstlane(min(min(v[0], v[1]), min(v[2], v[3])));
}
__device__ __forceinline__ void publish(volatile LDSP unsigned* w, unsigned v) {
  __hip_atomic_fetch_max((LDSP unsigned*)w, v, __ATOMIC_RELAXED, __HIP_MEMORY_SCOPE_WORKGROUP);
}
__device__ __forceinline__ void load16(u32x4& v, const void* ptr) {
  asm volatile("global_load_dwordx4 %0, %1, off" : "=v"(v) : "v"(ptr) : "memory");
}
__device__ __forceinline__ void store16_sys(void* ptr, u32x4 v) {
  asm volatile("global_store_dwordx4 %0, %1, off sc0 sc1\n\ts_nop 1" :: "v"(ptr), "v"(v) : "memory");
}
__device__ __forceinline__ void store4_sys(void* ptr, unsigned v) {
  asm volatile("global_store_dword %0, %1, off sc0 sc1" :: "v"(ptr), "v"(v) : "memory");
}
template <int N> __device__ __forceinline__ void wait_vm_lgkm0() { asm volatile("s_waitcnt vmcnt(%0) lgkmcnt(0)" :: "i"(N) : "memory"); }
constexpr int w_issued(int t) { return t <= 7 ? 44 : (44 + ((t - 7) * 7 + 1) / 2 > 128 ? 128 : 44 + ((t - 7) * 7 + 1) / 2); }
constexpr int in_flight(int t) {
  int nx = w_issued(t) - w_issued(t - 3);
  for (int k = t - 1; k <= t; ++k) nx += k < 32 ? 2 : 0;
  int tau = 3;
  while (w_issued(tau) < 4 * (t - 1)) ++tau;
  int nw = w_issued(t) - 4 * (t - 1);
  for (int k = tau + 1; k <= t; ++k) nw += k < 32 ? 2 : 0;
  return nx < nw ? nx : nw;
}
constexpr bool w_schedule_ok() {
  for (int t = 4; t < 35; ++t) {
    if ((w_issued(t) - 1) / 4 > (t <= 7 ? 3 : t - 4) + NB) return false;
    if (t >= 6 && t < 34 && (w_issued(t) < 4 * (t - 1) || in_flight(t) > 63)) return false;
  }
  return w_issued(34) == 128;
}
static_assert(w_schedule_ok() && NB == 7 && NA == 4, "W schedule");
__device__ __forceinline__ int unit_begin(int mt) { return mt < 4 ? 5 * mt : 4 * mt + 4; }

#define STAMP(i) do { } while (0)

struct Frag { v4i a[5]; v4i w[2]; };

__global__ __launch_bounds__(576, 3) void a2a_gemm_kernel(Params p) {
  extern __shared__ __attribute__((aligned(16))) unsigned char smem[];
  const int b = blockIdx.x;
  const int wave = __builtin_amdgcn_readfirstlane(threadIdx.x >> 6);
  const int lane = threadIdx.x & 63;
  const int rank = p.rank;
  unsigned char* const myws = p.myws;
  unsigned* const cnt = (unsigned*)(myws + CNT_OFF + b * 128);
  STAMP(0);

  if (b < NSEND_WG) {
    unsigned call;
    const int dj = b % 7, v = (b / 7) + 4 * wave;
    if (b < 28 && v < 14) {
      const unsigned long long tstart = __builtin_amdgcn_s_memrealtime();
      const int dst = (rank + 1 + dj) & 7;
      const int sj = (rank - dst - 1) & 7;
      const signed char* const src = p.x + (size_t)(dst * MS + (lane >> 3)) * KC + (lane & 7) * 16;
      int row0[4], q[4], fl[4]; bool big[4];
#pragma unroll
      for (int n = 0; n < 4; ++n) {
        const int pi = v + 14 * n, h = pi & 1;
        const int mt = ((pi >> 3) + 7 - sj) % 7;
        q[n] = (pi >> 1) & 3;
        big[n] = mt < 4;
        row0[n] = unit_begin(mt) * 16 + h * (big[n] ? 40 : 32);
        fl[n] = (rank * 8 + mt) * 8 + q[n] * 2 + h;
      }
      u32x4 r[4][5];
      auto load = [&](int n) {
        const signed char* s = src + row0[n] * KC + q[n] * 128;
#pragma unroll
        for (int i = 0; i < 4; ++i) load16(r[n][i], s + i * 8 * KC);
        load16(r[n][4], s + (big[n] ? 4 : 3) * 8 * KC);
      };
      load(0); load(1);
      asm volatile("s_load_dword %0, %1, 0x0\n\ts_waitcnt lgkmcnt(0)" : "=s"(call) : "s"(cnt) : "memory");
      call += 1;
      unsigned char* const dws = p.ws[dst];
      unsigned char* const dbase = dws + (call & 1) * ABUF_BYTES + rank * (ABUF_BYTES / 8) + lane * 16;
      while (__builtin_amdgcn_s_memrealtime() - tstart < (unsigned long long)(v * STAGGER_TICKS)) __builtin_amdgcn_s_sleep(1);
      asm volatile("s_waitcnt vmcnt(5)" : "+v"(r[0][0]), "+v"(r[0][1]), "+v"(r[0][2]), "+v"(r[0][3]), "+v"(r[0][4]) :: "memory");
      STAMP(1);
#pragma unroll
      for (int n = 0; n < 4; ++n) {
        unsigned char* d = dbase + q[n] * 65536 + row0[n] * 128;
#pragma unroll
        for (int i = 0; i < 4; ++i) store16_sys(d + i * 1024, r[n][i]);
        if (big[n]) store16_sys(d + 4096, r[n][4]);
        if (n + 2 < 4) load(n + 2);
        if (n == 0) asm volatile("s_waitcnt vmcnt(0)" : "+v"(r[1][0]), "+v"(r[1][1]), "+v"(r[1][2]), "+v"(r[1][3]), "+v"(r[1][4]),
                                 "+v"(r[2][0]), "+v"(r[2][1]), "+v"(r[2][2]), "+v"(r[2][3]), "+v"(r[2][4]) :: "memory");
        else if (n == 1) asm volatile("s_waitcnt vmcnt(0)" : "+v"(r[3][0]), "+v"(r[3][1]), "+v"(r[3][2]), "+v"(r[3][3]), "+v"(r[3][4]) :: "memory");
        else wait_vm<0>();
        if (lane == 0) store4_sys(dws + FLAG_OFF + fl[n] * 128, call);
        STAMP(2 + n);
      }
    } else {
      asm volatile("s_load_dword %0, %1, 0x0\n\ts_waitcnt lgkmcnt(0)" : "=s"(call) : "s"(cnt) : "memory");
      call += 1;
    }
    __syncthreads();
    STAMP(6);
    if (threadIdx.x == 0) __hip_atomic_store(cnt, call, __ATOMIC_RELAXED, __HIP_MEMORY_SCOPE_AGENT);
    return;
  }

  const int g = (b - NSEND_WG) >> 3, xcd = b & 7;
  const int mt = g % NMT, nt = (g / NMT) * 8 + xcd;
  const int ub = unit_begin(mt), nu = mt < 4 ? 5 : 4;
  volatile LDSP unsigned* const ltmo = (volatile LDSP unsigned*)(smem + LDS_CTL);
  const unsigned lds0 = (unsigned)(size_t)(LDSP unsigned char*)smem;
  auto src_of = [&](int js) { int sj = js - 1 - mt; sj += sj < 0 ? 7 : 0; return js == 0 ? rank : (rank + 1 + sj) & 7; };

  if (wave == 8) {
    if (lane == 0) *ltmo = 0;
    unsigned call;
    asm volatile("s_load_dword %0, %1, 0x0\n\ts_waitcnt lgkmcnt(0)" : "=s"(call) : "s"(cnt) : "memory");
    call += 1;
    unsigned* const tmo = (unsigned*)(myws + TMO_OFF);
    const int Sl = (lane >> 1) + 4;
    const unsigned* const myflag = (const unsigned*)(myws + FLAG_OFF + ((src_of((Sl >> 2) & 7) * 8 + mt) * 8 + (Sl & 3) * 2 + (lane & 1)) * 128);
#pragma unroll
    for (int t = 0; t < 5; ++t) __builtin_amdgcn_s_barrier();
    int avail = 4;
    unsigned long long seen = 0;
    unsigned p0 = 0, p1 = 0;
    auto note = [&](unsigned v) { seen |= __ballot(v >= call); avail = 4 + (__builtin_ctzll(~seen) >> 1); };
    auto look = [&](unsigned& v) {
      v = 0;
      if (Sl >= avail && Sl < avail + POLLW && Sl < 32) load4_sys(v, myflag);
    };
    auto need = [&](int S) {
      if (avail > S) return;
      const unsigned long long t0 = __builtin_amdgcn_s_memrealtime();
      for (unsigned it = 0;; ++it) {
        unsigned v = 0;
        if (Sl >= S && Sl < S + POLLW && Sl < 32) v = __hip_atomic_load(myflag, __ATOMIC_RELAXED, __HIP_MEMORY_SCOPE_SYSTEM);
        note(v);
        if (avail > S) return;
        if ((it & 63) == 63) {
          const unsigned t = __builtin_amdgcn_readfirstlane(__hip_atomic_load(tmo, __ATOMIC_RELAXED, __HIP_MEMORY_SCOPE_AGENT));
          if (t == call || __builtin_amdgcn_s_memrealtime() - t0 > DEADLINE_TICKS) {
            __hip_atomic_store(tmo, call, __ATOMIC_RELAXED, __HIP_MEMORY_SCOPE_AGENT);
            *ltmo = 1;
            seen = ~0ull >> 8;
            avail = 32;
            asm volatile("s_waitcnt lgkmcnt(0)" ::: "memory");
            return;
          }
        }
      }
    };
    for (int S = 4; S < 32; S += 2) {
      need(S);
      const bool more = avail < 32;
      if (more) look(p0);
      __builtin_amdgcn_s_barrier();
      if (avail <= S + 1) {
        asm volatile("s_waitcnt vmcnt(0)" : "+v"(p0) :: "memory");
        note(p0);
        need(S + 1);
        __builtin_amdgcn_s_barrier();
      } else {
        __builtin_amdgcn_s_barrier();
        asm volatile("s_waitcnt vmcnt(0)" : "+v"(p0) :: "memory");
        if (more) note(p0);
      }
    }
    (void)p1;
    return;
  }
  if (wave >= 4) {
    const int lw = wave - 4;
    const u32x4 srdW = make_srd(p.w, 4096u * NN);
    const u32x4 srdX = make_srd(p.x, 4096u * KC);
    const u32x4 srdS = make_srd(myws, FLAG_OFF);
    const unsigned vW0 = (lane >> 3) * NN + (((lane & 7) ^ ((lane >> 4) & 3)) << 4), vW1 = vW0 ^ 64;
    const int a0 = nu == 5 ? (lw >> 1) * 5 + (lw & 1) * 3 : lw * 2;
    const bool a3 = nu == 5 && !(lw & 1);
    const unsigned sw0 = (((lane & 7) ^ ((lane >> 4) & 3) ^ ((a0 & 1) << 2)) << 4);
    const unsigned vX0 = (lane >> 3) * KC + sw0, vX1 = vX0 ^ 64;
    const unsigned vS0 = (lane >> 3) * 128 + sw0, vS1 = vS0 ^ 64;
    const int arow = ub * 16 + a0 * 8;
    const unsigned wcol = lw * 32 * NN + nt * 128, lB = lds0 + LDS_B + lw * 4096, lA = lds0 + LDS_A + a0 * 1024;
    unsigned srcs = 0;
#pragma unroll
    for (int js = 0; js < 8; ++js) srcs |= (unsigned)src_of(js) << (4 * js);
    auto issue_W = [&](auto nc) {
      constexpr int n = nc.value, S = n >> 2, i = n & 3;
      const unsigned so = (((srcs >> (4 * (S >> 2))) & 7) << 21) + wcol + ((S & 3) * 128 + i * 8) * NN;
      dma16(srdW, (i & 2) ? vW1 : vW0, so, lB + (S % NB) * BSLOT + i * 1024);
    };
    auto issue_X_local = [&](int S) {
      const unsigned so = (unsigned)(rank * MS + arow) * KC + S * 128;
      const unsigned l = lA + S * ASLOT;
      dma16(srdX, vX0, so, l);
      dma16(srdX, vX1, so + 8 * KC, l + 1024);
      if (a3) dma16(srdX, vX0, so + 16 * KC, l + 2048);
    };
    auto W_range = [&](auto lo, auto cntc) { static_for<cntc.value>([&](auto k) { issue_W(std::integral_constant<int, lo.value + k.value>{}); }); };
#define IC(x) std::integral_constant<int, (x)>{}
    W_range(IC(0), IC(4)); issue_X_local(0);
    W_range(IC(4), IC(4)); issue_X_local(1);
    W_range(IC(8), IC(4)); issue_X_local(2);
    unsigned call;
    asm volatile("s_load_dword %0, %1, 0x0\n\ts_waitcnt lgkmcnt(0)" : "=s"(call) : "s"(cnt) : "memory");
    call += 1;
    const unsigned abase = (call & 1) * ABUF_BYTES + arow * 128;
    wait_vm<12>(); __builtin_amdgcn_s_barrier();
    W_range(IC(12), IC(4)); issue_X_local(3); W_range(IC(16), IC(4));
    wait_vm<16>(); __builtin_amdgcn_s_barrier();
    W_range(IC(20), IC(8));
    wait_vm<18>(); __builtin_amdgcn_s_barrier();
    W_range(IC(28), IC(8));
    wait_vm<20>(); __builtin_amdgcn_s_barrier();
    W_range(IC(36), IC(4));
    __builtin_amdgcn_s_barrier();
    W_range(IC(40), IC(4));
    static_for<31>([&](auto tc) {
      constexpr int t = tc.value + 4;
      __builtin_amdgcn_s_barrier();
      if constexpr (t < 32) {
        const unsigned so = abase + (((srcs >> (4 * (t >> 2))) & 7) << 18) + (t & 3) * 65536;
        const unsigned l = lA + (t % NA) * ASLOT;
        dma16_sys(srdS, vS0, so, l);
        dma16_sys(srdS, vS1, so + 1024, l + 1024);
        if (a3) dma16_sys(srdS, vS0, so + 2048, l + 2048);
      }
      W_range(IC(w_issued(t - 1)), IC(w_issued(t) - w_issued(t - 1)));
      if constexpr (t >= 6 && t < 34) wait_vm<in_flight(t)>();
    });
    if (threadIdx.x == 256) __hip_atomic_store(cnt, call, __ATOMIC_RELAXED, __HIP_MEMORY_SCOPE_AGENT);
    return;
  }

  float sx, sw;
  asm volatile("s_load_dword %0, %2, 0x0\n\ts_load_dword %1, %3, 0x0\n\ts_waitcnt lgkmcnt(0)"
               : "=&s"(sx), "=&s"(sw) : "s"(p.sx), "s"(p.sw) : "memory");
  const int fg = lane >> 4, fq = (lane & 15) >> 1, fp = lane & 1;
  unsigned pW0 = lds0 + LDS_B + (16 * fg + fq) * 128 + (((2 * wave) ^ (((fq >> 1) & 3) | ((fg & 1) << 2))) << 4) + 8 * fp;
  unsigned pW1 = pW0 ^ 16;
  unsigned pA0 = lds0 + LDS_A + (lane & 15) * 128 + ((fg ^ fq) << 4), pA1 = pA0 ^ 64;
  v4i acc[5][2];
#pragma unroll
  for (int u = 0; u < 5; ++u) { acc[u][0] = v4i{0, 0, 0, 0}; acc[u][1] = v4i{0, 0, 0, 0}; }
  Frag f0, f1;
#pragma unroll
  for (int u = 0; u < 5; ++u) f1.a[u] = v4i{0, 0, 0, 0};
  f1.w[0] = v4i{0, 0, 0, 0}; f1.w[1] = v4i{0, 0, 0, 0};
#define MM(c, u, n) acc[u][n] = __builtin_amdgcn_mfma_i32_16x16x64_i8(c.w[n], c.a[u], acc[u][n], 0, 0, 0); SCHED()
#define TR(p, o) __builtin_amdgcn_ds_read_tr8_b64_v2i32((LDSP v2i*)(size_t)((p) + (o)))
#define RA(p, o) (*(LDSP v4i*)(size_t)((p) + (o)))
  int bs = 0, as = 0;
  __builtin_amdgcn_s_waitcnt(0xC07F);
  for (int S = 0; S < 32; ++S) {
    v2i w00, w01, w10, w11;
    const unsigned dB = bs == NB - 1 ? -(NB - 1) * (int)BSLOT : BSLOT, dA = as == NA - 1 ? -(NA - 1) * (int)ASLOT : ASLOT;
    bs = bs == NB - 1 ? 0 : bs + 1; as = as == NA - 1 ? 0 : as + 1;
    MM(f1, 0, 0);
    MM(f1, 0, 1);
    __builtin_amdgcn_s_waitcnt(0xC07F);
    asm volatile("" ::: "memory");
    if (S == 4) { __builtin_amdgcn_s_barrier(); __builtin_amdgcn_s_barrier(); __builtin_amdgcn_s_barrier(); __builtin_amdgcn_s_barrier(); }
    __builtin_amdgcn_s_barrier();
    asm volatile("" ::: "memory");
    w00 = TR(pW0, 0); SCHED();
    MM(f1, 1, 0); w01 = TR(pW0, 1024); SCHED();
    MM(f1, 1, 1); f0.a[0] = RA(pA0, 0); SCHED();
    MM(f1, 2, 0); w10 = TR(pW1, 0); SCHED();
    MM(f1, 2, 1); w11 = TR(pW1, 1024); SCHED();
    MM(f1, 3, 0); f0.a[1] = RA(pA0, 2048); SCHED();
    MM(f1, 3, 1); f0.a[2] = RA(pA0, 4096); SCHED();
    MM(f1, 4, 0); f0.a[3] = RA(pA0, 6144); SCHED();
    MM(f1, 4, 1); f0.a[4] = RA(pA0, 8192); pA0 += dA; SCHED();
    f0.w[0] = v4i{w00[0], w00[1], w01[0], w01[1]}; f0.w[1] = v4i{w10[0], w10[1], w11[0], w11[1]};
    MM(f0, 0, 0); w00 = TR(pW0, 8192); w01 = TR(pW0, 9216); SCHED();
    MM(f0, 0, 1); f1.a[0] = RA(pA1, 0); pW0 += dB; SCHED();
    MM(f0, 1, 0); w10 = TR(pW1, 8192); w11 = TR(pW1, 9216); SCHED();
    MM(f0, 1, 1); f1.a[1] = RA(pA1, 2048); pW1 += dB; SCHED();
    MM(f0, 2, 0); f1.a[2] = RA(pA1, 4096); SCHED();
    MM(f0, 2, 1); f1.a[3] = RA(pA1, 6144); SCHED();
    MM(f0, 3, 0); f1.a[4] = RA(pA1, 8192); SCHED();
    MM(f0, 3, 1); pA1 += dA; SCHED();
    MM(f0, 4, 0);
    MM(f0, 4, 1);
    f1.w[0] = v4i{w00[0], w00[1], w01[0], w01[1]}; f1.w[1] = v4i{w10[0], w10[1], w11[0], w11[1]};
  }
  const unsigned bad = *ltmo;
#pragma unroll
  for (int u = 0; u < 5; ++u) { MM(f1, u, 0); MM(f1, u, 1); }
  STAMP(11);

  const float s = bad ? __builtin_nanf("") : sx * sw;
  float* o = p.out + (size_t)(ub * 16 + (lane & 15)) * NN + nt * 128 + wave * 32 + (lane >> 4) * 4;
#pragma unroll
  for (int u = 0; u < 5; ++u) {
    if (u < nu) {
#pragma unroll
      for (int nb = 0; nb < 2; ++nb) {
        v4f y;
#pragma unroll
        for (int r = 0; r < 4; ++r) { float v = (float)acc[u][nb][r] * s; y[r] = v < 0.f ? 0.f : v; }
        *(v4f*)(o + (size_t)u * 16 * NN + nb * 16) = y;
      }
    }
  }
}

}

extern "C" void kernel_launch(int rank, int nranks, ncclComm_t comm,
                              hipStream_t stream,
                              void* const* d_in, const int* in_sizes, int n_in,
                              void* d_out, int out_size,
                              const int* in_dtypes, int out_dtype,
                              void* const* d_out_all,
                              void* const* d_ws_all, size_t ws_bytes) {
  if (nranks != NR || (unsigned)rank >= (unsigned)NR || ws_bytes < WS_USED || n_in < 4 || out_size != MS * NN) {
    fprintf(stderr, "unexpected problem geometry\n");
    return;
  }
  static bool init = false;
  if (!init) {
    HIP_CHECK(hipFuncSetAttribute((const void*)a2a_gemm_kernel, hipFuncAttributeMaxDynamicSharedMemorySize, LDS_BYTES));
    init = true;
  }
  Params p;
  p.x = (const signed char*)d_in[0];
  p.w = (const signed char*)d_in[1];
  p.sx = (const float*)d_in[2];
  p.sw = (const float*)d_in[3];
  p.out = (float*)d_out;
  for (int i = 0; i < NR; ++i) p.ws[i] = (unsigned char*)d_ws_all[i];
  p.myws = (unsigned char*)d_ws_all[rank];
  p.rank = rank;
  p.pad = 0;
  a2a_gemm_kernel<<<256, 576, LDS_BYTES, stream>>>(p);
  HIP_CHECK(hipGetLastError());
}
```

```cpp
#include <hip/hip_runtime.h>
#include <rccl/rccl.h>
#include <cstdio>
#include <type_traits>

#define HIP_CHECK(expr) do { hipError_t _e = (expr); if (_e != hipSuccess) { \
    fprintf(stderr, "HIP error %s at %s:%d\n", hipGetErrorString(_e), __FILE__, __LINE__); \
    return; } } while (0)

namespace {

typedef int v4i __attribute__((ext_vector_type(4)));
typedef unsigned u32x4 __attribute__((ext_vector_type(4)));
#define LDSP __attribute__((address_space(3)))

constexpr int NR = 8;
constexpr int MS = 512;
constexpr int KC = 512;
constexpr int NN = 4096;
constexpr int NMT = 7;
constexpr int NSEND_WG = 32;

constexpr unsigned XBUF = 4 * 512 * 128;
constexpr unsigned SRC_FLAG = 2 * XBUF;
#define FSTEP 64
constexpr unsigned FLAG_STEP = FSTEP, FLAG_LINE = FSTEP == 4 ? 256 : 8 * FSTEP;
constexpr unsigned SRC_STRIDE = SRC_FLAG + 16384;
constexpr unsigned CNT_OFF = NR * SRC_STRIDE;
constexpr unsigned TMO_OFF = CNT_OFF + 256 * 128;
constexpr unsigned DIAG_OFF = TMO_OFF + 128;
constexpr unsigned WS_USED = DIAG_OFF + 256 * 128 * 4;

#define PACE_TICKS 8
#define LOADER_PRIO 3
#define SCU 4
#define ROTMASK 0xf8
constexpr unsigned long long DEADLINE_TICKS = 50000000ull;

constexpr int NA = 4, NB = 7;
constexpr unsigned BSLOT = 16384, ASLOT = 10240;
constexpr unsigned LDS_B = 0, LDS_A = NB * BSLOT, LDS_SNAP = LDS_A + NA * ASLOT, LDS_TMO = LDS_SNAP + 256;
constexpr unsigned LDS_JUNK = LDS_TMO + 64, LDS_BYTES = LDS_JUNK + 256;
#define POLLD 4
#define POLLW 6
#define POLLP 1
#define SPIN_SLEEP 0

struct Link { const signed char* rows; unsigned char* region; };
struct Params {
  const signed char* x;
  const signed char* w;
  unsigned char* myws;
  float* out;
  const float* sx;
  const float* sw;
  int rank;
  int pad[3];
  Link link[NR];
};
typedef unsigned u32x8 __attribute__((ext_vector_type(8)));
template <class T> __device__ __forceinline__ T* ptr_of(unsigned lo, unsigned hi) { return (T*)(((unsigned long long)hi << 32) | lo); }

template <int N> __device__ __forceinline__ void wait_vm() { asm volatile("s_waitcnt vmcnt(%0)" :: "i"(N) : "memory"); }
template <int N, int I = 0, class F> __device__ __forceinline__ void static_for(F&& f) {
  if constexpr (I < N) { f(std::integral_constant<int, I>{}); static_for<N, I + 1>(f); }
}
#define IC(x) std::integral_constant<int, (x)>{}
#define BARRIER() __builtin_amdgcn_s_barrier()

#define XBITS " sc0 sc1"
#define DMA_ONE(n, bits) ".if %[cnt] > " #n "\n\tbuffer_load_dwordx4 %[v" #n "], %[srd], %[so] offen offset:1024*" #n bits " lds\n\t.endif\n\t"
#define DMA_BURST(bits) \
  asm volatile(DMA_ONE(0, bits) DMA_ONE(1, bits) DMA_ONE(2, bits) DMA_ONE(3, bits) \
               :: [v0] "v"(v0), [v1] "v"(v1), [v2] "v"(v2), [v3] "v"(v3), [srd] "s"(srd), [so] "s"(soff), [cnt] "i"(N) : "memory")
template <int N>
__device__ __forceinline__ void dma(u32x4 srd, unsigned v0, unsigned v1, unsigned v2, unsigned v3, unsigned soff) {
  DMA_BURST("");
}
template <int N>
__device__ __forceinline__ void dma_sys(u32x4 srd, unsigned v0, unsigned v1, unsigned v2, unsigned v3, unsigned soff) {
  DMA_BURST(XBITS);
}
#define SET_M0(base, IMM, TAIL) asm volatile("s_add_u32 m0, %0, %1\n\t" TAIL :: "s"(base), "i"(IMM) : "memory", "scc")
template <int IMM> __device__ __forceinline__ void set_m0(unsigned base) { SET_M0(base, IMM, "s_nop 0"); }
template <int IMM> __device__ __forceinline__ void set_m0_barrier(unsigned base) { SET_M0(base, IMM, "s_barrier"); }
template <int IMM> __device__ __forceinline__ void set_m0_nop_barrier(unsigned base) { SET_M0(base, IMM, "s_nop 3\n\ts_barrier"); }
__device__ __forceinline__ void dma4_sys(u32x4 srd, unsigned voff, unsigned lds, unsigned long long mask) {
  asm volatile("s_mov_b32 m0, %2\n\ts_mov_b64 exec, %3\n\tbuffer_load_dword %0, %1, 0 offen sc0 sc1 lds\n\ts_mov_b64 exec, -1"
               :: "v"(voff), "s"(srd), "s"(lds), "s"(mask) : "memory");
}
__device__ __forceinline__ void touch(u32x4 srd, unsigned voff, unsigned soff, unsigned lds) {
  asm volatile("s_mov_b32 m0, %3\n\ts_nop 0\n\tbuffer_load_dword %0, %1, %2 offen lds" :: "v"(voff), "s"(srd), "s"(soff), "s"(lds) : "memory");
}
__device__ __forceinline__ void load16(u32x4& v, u32x4 srd, unsigned voff, unsigned soff) {
  asm volatile("buffer_load_dwordx4 %0, %1, %2, %3 offen" : "=v"(v) : "v"(voff), "s"(srd), "s"(soff) : "memory");
}
__device__ __forceinline__ void store16_sys(void* ptr, u32x4 v) {
  asm volatile("global_store_dwordx4 %0, %1, off sc0 sc1\n\ts_nop 1" :: "v"(ptr), "v"(v) : "memory");
}
__device__ __forceinline__ void store4_sys(void* ptr, unsigned v) {
  asm volatile("global_store_dword %0, %1, off sc0 sc1" :: "v"(ptr), "v"(v) : "memory");
}
__device__ __forceinline__ unsigned sload(const void* p) {
  unsigned v;
  asm volatile("s_load_dword %0, %1, 0x0\n\ts_waitcnt lgkmcnt(0)" : "=s"(v) : "s"(p) : "memory");
  return v;
}
__device__ __forceinline__ unsigned now32() { return (unsigned)__builtin_amdgcn_s_memrealtime(); }
__device__ __forceinline__ int unit_begin(int mt) { return 4 * mt + min(mt, 4); }

constexpr int LEAD = NA - 1;
constexpr int w_lo(int t, int odd) { return 88 + 7 * (t - 8) + (odd ? 3 : 0); }
constexpr int w_cnt(int t, int odd) { return t < 8 || t > 31 ? 0 : odd ? 4 : 3; }
constexpr int in_flight(int t, int odd, int nx) {
  const int S = t + 1 - LEAD;
  int n = 0;
  for (int k = t; k >= S; --k) {
    if (k < t) for (int i = w_cnt(k, odd) - 1; i >= 0; --i) { if (((w_lo(k, odd) + i) >> 3) <= S) return n; ++n; }
    if (k > S && k < 32) n += nx;
  }
  return n;
}
constexpr bool w_schedule_ok() {
  for (int t = 8; t < 32; ++t) {
    if (((w_lo(t, 1) + 3) >> 3) > t - LEAD - 1 + NB) return false;
    if ((w_lo(t, 0) >> 3) <= t + 1 - LEAD) return false;
  }
  return w_lo(32, 0) == 256;
}
static_assert(w_schedule_ok() && NB == 7 && NA == 4, "W schedule");
constexpr int w_slot(int S) { return S < 3 ? 4 + S : S == 3 ? 4 : (S - 4) % NB; }

#define STAMP(i) do { } while (0)
#define STAMPC(i) do { } while (0)
#define DUMP(slot) do { } while (0)

#define WB0(s) "v[" #s "+0:" #s "+3]"
#define WB1(s) "v[" #s "+4:" #s "+7]"
#define XU(s, u) "v[" #s "+8+4*" #u ":" #s "+11+4*" #u "]"
#define MF(acc, w, x) "v_mfma_i32_16x16x64_i8 %[" #acc "], " x ", " w ", %[" #acc "]\n\t"
#define TRL(s, o, p, off) "ds_read_b64_tr_b8 v[" #s "+" #o ":" #s "+" #o "+1], %[" #p "] offset:" #off "\n\t"
#define RX(s, u, p) "ds_read_b128 " XU(s, u) ", %[" #p "] offset:2048*" #u "\n\t"
#define IF5(x) ".if %[nu] == 5\n\t" x ".endif\n\t"
#define IF4(x) ".if %[nu] == 4\n\t" x ".endif\n\t"
#define TICK_STAMP
#define DG_OPERAND
#define DG_PARAM
#define DG_ARG
#define LOOP_STAMP

template <int NU>
__device__ __forceinline__ void k_loop(v4i (&acc)[5][2], unsigned pW0, unsigned pW1, unsigned pA0, unsigned pA1 DG_PARAM) {
  unsigned S = 0, bs = 5, as = 1, dB = BSLOT, dA = ASLOT, wrapB = -2 * (int)BSLOT, t0;
  asm volatile(
    "s_branch 2f\n\t"
    "1:\n\t"
    "s_waitcnt lgkmcnt(%[nu]-1)\n\t"
    MF(c00, WB0(92), XU(92, 0))
    MF(c01, WB1(92), XU(92, 0))
    "s_waitcnt lgkmcnt(0)\n\t"
    "s_barrier\n\t"
    LOOP_STAMP
    TRL(64, 0, pw0, 0) IF4(TRL(64, 2, pw0, 1024))
    MF(c11, WB1(92), XU(92, 1)) IF5(TRL(64, 2, pw0, 1024)) IF4(RX(64, 0, pa0))
    MF(c10, WB0(92), XU(92, 1)) IF5(RX(64, 0, pa0)) IF4(TRL(64, 4, pw1, 0))
    MF(c20, WB0(92), XU(92, 2)) IF5(TRL(64, 4, pw1, 0)) IF4(TRL(64, 6, pw1, 1024))
    MF(c21, WB1(92), XU(92, 2)) IF5(TRL(64, 6, pw1, 1024)) IF4(RX(64, 1, pa0))
    MF(c31, WB1(92), XU(92, 3)) IF5(RX(64, 1, pa0)) IF4(RX(64, 2, pa0))
    MF(c30, WB0(92), XU(92, 3)) IF5(RX(64, 2, pa0)) IF4(RX(64, 3, pa0))
    IF5(MF(c40, WB0(92), XU(92, 4)) RX(64, 3, pa0)
    MF(c41, WB1(92), XU(92, 4)) RX(64, 4, pa0))
    "3:\n\t"
    "s_waitcnt lgkmcnt(%[nu]-1)\n\t"
    MF(c00, WB0(64), XU(64, 0)) TRL(92, 0, pw0, 8192) TRL(92, 2, pw0, 9216)
    MF(c01, WB1(64), XU(64, 0)) RX(92, 0, pa1)
    "s_waitcnt lgkmcnt(%[nu])\n\t"
    MF(c11, WB1(64), XU(64, 1)) TRL(92, 4, pw1, 8192) TRL(92, 6, pw1, 9216)
    MF(c10, WB0(64), XU(64, 1)) RX(92, 1, pa1) "v_add_u32 %[pa0], %[dA], %[pa0]\n\t"
    MF(c20, WB0(64), XU(64, 2)) RX(92, 2, pa1) "v_add_u32 %[pw0], %[dB], %[pw0]\n\t"
    MF(c21, WB1(64), XU(64, 2)) RX(92, 3, pa1)
    "s_waitcnt lgkmcnt(8)\n\t"
    MF(c31, WB1(64), XU(64, 3)) IF5(RX(92, 4, pa1)) "v_add_u32 %[pw1], %[dB], %[pw1]\n\t"
    IF5(MF(c30, WB0(64), XU(64, 3))) "v_add_u32 %[pa1], %[dA], %[pa1]\n\t"
    "s_cmp_eq_u32 %[bs], %[nb1]\n\ts_cselect_b32 %[dB], %[wrapB], %[stepB]\n\ts_cselect_b32 %[bs], -1, %[bs]\n\t"
    IF5(MF(c40, WB0(64), XU(64, 4)))
    "s_cmp_eq_u32 %[as], %[na1]\n\ts_cselect_b32 %[dA], %[wrapA], %[stepA]\n\ts_cselect_b32 %[as], -1, %[as]\n\t"
    "s_add_u32 %[bs], %[bs], 1\n\ts_add_u32 %[as], %[as], 1\n\ts_add_u32 %[S], %[S], 1\n\ts_bitcmp0_b64 %[special], %[S]\n\t"
    IF5(MF(c41, WB1(64), XU(64, 4))) IF4(MF(c30, WB0(64), XU(64, 3)))
    "s_cbranch_scc1 1b\n\t"
    "s_waitcnt lgkmcnt(0)\n\t"
    MF(c00, WB0(92), XU(92, 0)) MF(c01, WB1(92), XU(92, 0)) MF(c11, WB1(92), XU(92, 1)) MF(c10, WB0(92), XU(92, 1))
    MF(c20, WB0(92), XU(92, 2)) MF(c21, WB1(92), XU(92, 2)) MF(c31, WB1(92), XU(92, 3)) MF(c30, WB0(92), XU(92, 3))
    IF5(MF(c40, WB0(92), XU(92, 4)) MF(c41, WB1(92), XU(92, 4)))
    "s_cmp_eq_u32 %[S], 32\n\t"
    "s_cbranch_scc1 5f\n\t"
    "s_mov_b32 %[wrapB], %[back5]\n\ts_mov_b32 %[bs], 1\n\ts_mov_b32 %[dB], %[stepB]\n\t"
    "v_add_u32 %[pw0], %[wrapB], %[pw0]\n\tv_add_u32 %[pw1], %[wrapB], %[pw1]\n\t"
    "s_mov_b32 %[wrapB], %[back6]\n\t"
    ".rept %[nbar]\n\ts_barrier\n\t.endr\n\t"
    "2:\n\t"
    "s_barrier\n\t"
    TICK_STAMP
    TRL(64, 0, pw0, 0) TRL(64, 2, pw0, 1024) RX(64, 0, pa0) TRL(64, 4, pw1, 0) TRL(64, 6, pw1, 1024)
    RX(64, 1, pa0) RX(64, 2, pa0) RX(64, 3, pa0) IF5(RX(64, 4, pa0))
    "s_branch 3b\n\t"
    "5:\n\t"
    "s_nop 7\n\t"
    : [c00] "+v"(acc[0][0]), [c01] "+v"(acc[0][1]), [c10] "+v"(acc[1][0]), [c11] "+v"(acc[1][1]), [c20] "+v"(acc[2][0]),
      [c21] "+v"(acc[2][1]), [c30] "+v"(acc[3][0]), [c31] "+v"(acc[3][1]), [c40] "+v"(acc[4][0]), [c41] "+v"(acc[4][1]),
      [pw0] "+v"(pW0), [pw1] "+v"(pW1), [pa0] "+v"(pA0), [pa1] "+v"(pA1),
      [S] "+&s"(S), [bs] "+&s"(bs), [as] "+&s"(as), [dB] "+&s"(dB), [dA] "+&s"(dA), [wrapB] "+&s"(wrapB), [t0] "=&s"(t0) DG_OPERAND
    : [nu] "i"(NU), [nb1] "i"(NB - 1), [na1] "i"(NA - 1), [stepB] "i"(BSLOT), [stepA] "i"(ASLOT),
      [back5] "i"(-5 * (int)BSLOT), [back6] "i"(-(NB - 1) * (int)BSLOT), [wrapA] "s"(-(NA - 1) * (int)ASLOT),
      [special] "s"((1ull << 4) | (1ull << 32)), [nbar] "i"(LEAD + 1)
    : "memory", "scc", "s90", "s91", "s92", "s93",
      "v64", "v65", "v66", "v67", "v68", "v69", "v70", "v71", "v72", "v73", "v74", "v75", "v76", "v77", "v78", "v79",
      "v80", "v81", "v82", "v83", "v84", "v85", "v86", "v87", "v88", "v89", "v90", "v91", "v92", "v93", "v94", "v95",
      "v96", "v97", "v98", "v99", "v100", "v101", "v102", "v103", "v104", "v105", "v106", "v107", "v108", "v109", "v110",
      "v111", "v112", "v113", "v114", "v115", "v116", "v117", "v118", "v119");
}

constexpr int in_flight_loop(int k, int odd, int nx) {
  int m = 63;
  for (int it = 1; it < 3; ++it) m = in_flight(8 + 8 * it + k, odd, nx) < m ? in_flight(8 + 8 * it + k, odd, nx) : m;
  return m;
}

template <int G, int N, int SLOT>
__device__ __forceinline__ void w_burst(u32x4 srdW, const unsigned (&vW)[2][4], unsigned stage_off, unsigned lB) {
  unsigned so;
  asm volatile("s_add_u32 m0, %1, %2\n\ts_add_u32 %0, %3, %4" : "=s"(so) : "s"(lB), "i"(SLOT * BSLOT + G * 1024), "s"(stage_off), "i"(G * 8 * NN) : "memory", "scc");
  dma<N>(srdW, vW[(G >> 1) & 1][0], vW[((G + 1) >> 1) & 1][1], vW[((G + 2) >> 1) & 1][2], vW[((G + 3) >> 1) & 1][3], so);
}

template <int ODD, int NX>
__device__ __forceinline__ void load_loop(u32x4 srdW, u32x4 srdS, const unsigned (&vW)[2][4], unsigned vS0, unsigned vS1,
                                          unsigned tabW, unsigned tabX, unsigned lB, unsigned lA) {
  auto lane_of = [](unsigned tab, int i) { unsigned r = __builtin_amdgcn_readlane(tab, i); asm volatile("" : "+s"(r)); return r; };
  unsigned sx = lane_of(tabX, 4), sw = lane_of(tabW, 11), swn = lane_of(tabW, 12);
  static_for<4>([&](auto kc) {
    constexpr int t = 4 + kc.value;
    set_m0_nop_barrier<(t % NA) * ASLOT>(lA);
    dma_sys<NX>(srdS, vS0, vS1, vS0, vS1, sx);
    sx = lane_of(tabX, t + 1);
    if constexpr (t >= 3 + LEAD) wait_vm<in_flight(t, ODD, NX)>();
  });
  asm volatile("s_nop 3");
#pragma unroll 1
  for (int it = 0; it < 3; ++it) {
    static_for<8>([&](auto kc) {
      constexpr int k = kc.value, t = 8 + k;
      constexpr int h = w_lo(t, ODD), n = w_cnt(t, ODD), Sc = w_lo(t, 0) >> 3;
      set_m0_barrier<(t % NA) * ASLOT>(lA);
      dma_sys<NX>(srdS, vS0, vS1, vS0, vS1, sx);
      sx = lane_of(tabX, t + 1 + 8 * it);
      unsigned swnn = 0;
      if constexpr ((w_lo(t + 1, 0) >> 3) != Sc) swnn = lane_of(tabW, Sc + 2 + 7 * it);
      if constexpr (in_flight(t, ODD, NX) < in_flight_loop(k, ODD, NX)) { if (it == 0) wait_vm<in_flight(t, ODD, NX)>(); }
      wait_vm<in_flight_loop(k, ODD, NX)>();
      if constexpr (((h + n - 1) >> 3) == (h >> 3)) {
        w_burst<h & 7, n, w_slot(h >> 3)>(srdW, vW, (h >> 3) == Sc ? sw : swn, lB);
      } else {
        w_burst<h & 7, 8 - (h & 7), w_slot(Sc)>(srdW, vW, sw, lB);
        w_burst<0, n - 8 + (h & 7), w_slot(Sc + 1)>(srdW, vW, swn, lB);
      }
      if constexpr ((w_lo(t + 1, 0) >> 3) != Sc) { sw = swn; swn = swnn; }
    });
  }
  static_for<LEAD>([&](auto kc) {
    constexpr int t = 32 + kc.value;
    BARRIER();
    if constexpr (t < 31 + LEAD) wait_vm<in_flight(t, ODD, NX)>();
  });
}

__global__ __attribute__((flatten)) __launch_bounds__(576, 3) void a2a_gemm_kernel(Params p) {
  extern __shared__ __attribute__((aligned(16))) unsigned char smem[];
  const int b = blockIdx.x;
  const int wave = __builtin_amdgcn_readfirstlane(threadIdx.x >> 6);
  const int lane = threadIdx.x & 63;
  STAMP(0);
  const int b7 = (b & 31) * 37 >> 8, dj = (b & 31) - 7 * b7;
  asm volatile("s_load_dword s100, %0, 0x0\n\ts_load_dword s101, %0, %1"
               :: "s"(__builtin_amdgcn_kernarg_segment_ptr()), "s"(64 + 16 * dj) : "memory", "s100", "s101");
#define ARGS() u32x8 ka, kb; u32x4 kl; \
  asm volatile("s_load_dwordx8 %0, %3, 0x0\n\ts_load_dwordx8 %1, %3, 0x20\n\ts_load_dwordx4 %2, %3, %4\n\ts_waitcnt lgkmcnt(0)" \
               : "=&s"(ka), "=&s"(kb), "=&s"(kl) : "s"(__builtin_amdgcn_kernarg_segment_ptr()), "s"(64 + 16 * dj) : "memory"); \
  const int rank = kb[4]; unsigned char* const myws = ptr_of<unsigned char>(ka[4], ka[5]); \
  unsigned* const cnt = (unsigned*)(myws + CNT_OFF + b * 128); (void)rank; (void)cnt
#define SRD(lo, hi, bytes) u32x4{lo, (hi) & 0xffffu, bytes, 0x00020000u}

  if (b >= NSEND_WG) {
  const int g = (b - NSEND_WG) >> 3, xcd = b & 7;
  const int ntl = g * 37 >> 8, mt = g - NMT * ntl, nt = xcd * 4 + ntl;
  const int ub = unit_begin(mt), nu = mt < 4 ? 5 : 4;
  volatile LDSP unsigned* const ltmo = (volatile LDSP unsigned*)(smem + LDS_TMO);
  const unsigned lds0 = (unsigned)(size_t)(LDSP unsigned char*)smem;
  auto src_of = [&](int rank, int js) { int sj = js - 1 - mt; sj += sj < 0 ? 7 : 0; return js == 0 ? rank : (rank + 1 + sj) & 7; };
  auto q_of = [&](int S) { return (S + (S < 4 ? mt + ntl : ((ROTMASK >> (S >> 2)) & 1) ? ntl : 0)) & 3; };

  if (wave < 4) {
  const int fg = lane >> 4, fq = (lane & 15) >> 1, fp = lane & 1;
  const unsigned pW0 = lds0 + LDS_B + w_slot(0) * BSLOT + (16 * fg + fq) * 128 + (((2 * wave) ^ (((fq >> 1) & 3) | ((fg & 1) << 2))) << 4) + 8 * fp;
  const unsigned pA0 = lds0 + LDS_A + (lane & 15) * 128 + ((fg ^ fq) << 4);
  unsigned vO[8];
#pragma unroll
  for (int i = 0; i < 8; ++i) {
    vO[i] = ((ub * 16 + (lane >> 5) * 8 + i) * NN + nt * 128 + wave * 32 + (lane & 31)) * 4;
    asm volatile("" : "+v"(vO[i]));
  }
  ARGS();
  const u32x4 srdO = SRD(ka[6], ka[7], (unsigned)MS * NN * 4);
  {
    const int slow = (3 - (int)(ka[2] >> 7) - xcd * 4) & 7;
    if (wave == 0) {
      touch(srdO, (ub * 16 + (lane >> 5) * (nu * 16 - 1)) * NN * 4 + nt * 512, 0, lds0 + LDS_JUNK);
    } else if (wave < 3 && slow < 4 && slow != ntl) {
      const int part = ((ntl - slow - 1) & 3) * 2 + wave - 1;
      const unsigned row = min(part * 43 + min(lane, 42), 255);
      touch(SRD(ka[2], ka[3], 4096u * NN), row * NN, (unsigned)(src_of(rank, 1) * KC + 256) * NN + (xcd * 4 + slow) * 128, lds0 + LDS_JUNK);
    }
  }
  float sx, sw;
  asm volatile("s_load_dword %0, %2, 0x0\n\ts_load_dword %1, %3, 0x0\n\ts_waitcnt lgkmcnt(0)"
               : "=&s"(sx), "=&s"(sw) : "s"(ptr_of<float>(kb[0], kb[1])), "s"(ptr_of<float>(kb[2], kb[3])) : "memory");
  float scale = sx * sw;
  asm volatile("" : "+v"(scale));
  v4i acc[5][2];
#pragma unroll
  for (int u = 0; u < 5; ++u) { acc[u][0] = v4i{0, 0, 0, 0}; acc[u][1] = v4i{0, 0, 0, 0}; }
  if (nu == 5) k_loop<5>(acc, pW0, pW0 ^ 16, pA0, pA0 ^ 64 DG_ARG);
  else k_loop<4>(acc, pW0, pW0 ^ 16, pA0, pA0 ^ 64 DG_ARG);
  if (*ltmo) scale = __builtin_nanf("");
  STAMP(11);
  STAMPC(17);

#pragma unroll
  for (int u = 0; u < 5; ++u) {
    if (u < nu) {
#pragma unroll
      for (int r = 0; r < 4; ++r) {
        const float y0 = __builtin_elementwise_maximum((float)acc[u][0][r] * scale, 0.f);
        const float y1 = __builtin_elementwise_maximum((float)acc[u][1][r] * scale, 0.f);
        const auto y = __builtin_amdgcn_permlane16_swap(__builtin_bit_cast(unsigned, y0), __builtin_bit_cast(unsigned, y1), false, false);
#define OUT_BITS " sc1"
        asm volatile("buffer_store_dword %0, %1, %2, %3 offen" OUT_BITS :: "v"(y[0]), "v"(vO[r]), "s"(srdO), "s"(u * 16 * NN * 4) : "memory");
        asm volatile("buffer_store_dword %0, %1, %2, %3 offen" OUT_BITS :: "v"(y[1]), "v"(vO[r + 4]), "s"(srdO), "s"(u * 16 * NN * 4) : "memory");
      }
    }
  }
  return;
  }

  if (wave == 8) {
    volatile LDSP unsigned* const snap = (volatile LDSP unsigned*)(smem + LDS_SNAP) + lane;
    *snap = 0;
    if (lane == 0) *ltmo = 0;
    ARGS();
    __builtin_amdgcn_s_setprio(3);
    const unsigned call = sload(cnt) + 1;
    unsigned* const tmo = (unsigned*)(myws + TMO_OFF);
    const u32x4 srdS = SRD(ka[4], ka[5], CNT_OFF);
    const unsigned vR = src_of(rank, lane >> 3) * SRC_STRIDE;
    const unsigned vF = vR + SRC_FLAG + mt * FLAG_LINE + (q_of(lane >> 1) * 2 + (lane & 1)) * FLAG_STEP;
    touch(srdS, vR + ((lane & 1) ? SRC_FLAG + 8192 : (call & 1) * XBUF), 0, lds0 + LDS_JUNK);
    auto poll = [&](int from) { dma4_sys(srdS, vF, lds0 + LDS_SNAP, ((1ull << (2 * POLLW)) - 1) << (2 * from)); };
    auto arrived = [&]() { const unsigned long long m = ~(__ballot(*snap >= call) | 0xff); return m ? (int)__builtin_ctzll(m) >> 1 : 32; };
#pragma unroll
    for (int t = 0; t < 4; ++t) BARRIER();
    poll(4);
    BARRIER();
    STAMP(9);
    int avail = 4, next = 0;
    for (int S = 4; S < 32; ++S) {
      if (avail < 32 && avail <= S + POLLD) {
        avail = arrived();
        if (avail <= S) {
          const unsigned long long t0 = __builtin_amdgcn_s_memrealtime();
          for (unsigned it = 0;; ++it) {
            poll(avail);
            wait_vm<0>();
            avail = arrived();
            if (avail > S) break;
            if ((it & 63) == 63) {
              const unsigned t = __builtin_amdgcn_readfirstlane(__hip_atomic_load(tmo, __ATOMIC_RELAXED, __HIP_MEMORY_SCOPE_AGENT));
              if (t == call || __builtin_amdgcn_s_memrealtime() - t0 > DEADLINE_TICKS) {
                __hip_atomic_store(tmo, call, __ATOMIC_RELAXED, __HIP_MEMORY_SCOPE_AGENT);
                *ltmo = 1;
                asm volatile("s_waitcnt lgkmcnt(0)" ::: "memory");
                avail = 32;
                break;
              }
            }
            if (SPIN_SLEEP) __builtin_amdgcn_s_sleep(SPIN_SLEEP);
          }
        }
        if (avail < 32 && avail <= S + POLLD && S >= next) { poll(avail); next = S + POLLP; }
      }
      if ((S & 3) == 0) STAMP(S >> 2);
      BARRIER();
    }
    STAMP(8);
    DUMP(1);
    return;
  }

  {
    const int lw = wave - 4;
    const int a0 = nu == 5 ? (lw >> 1) * 5 + (lw & 1) * 3 : lw * 2;
    const bool odd = lw & 1, a3 = nu == 5 && !odd;
    const int arow = ub * 16 + a0 * 8;
    const unsigned swz = (lane & 7) ^ ((lane >> 4) & 3);
    unsigned vW[2][4], vX[3];
#pragma unroll
    for (int i = 0; i < 4; ++i) {
      vW[0][i] = ((lw >> 1) * 64 + (lane >> 3)) * NN + nt * 128 + (swz << 4) + i * (8 * NN - 1024);
      vW[1][i] = vW[0][i] ^ 64;
      if (i < 3) vX[i] = (arow + (lane >> 3)) * KC + ((swz ^ (((a0 + i) & 1) << 2)) << 4) + i * (8 * KC - 1024);
    }
    const unsigned vS0 = (arow + (lane >> 3)) * 128 + ((swz ^ ((a0 & 1) << 2)) << 4), vS1 = vS0 ^ 64;
    unsigned lB = lds0 + LDS_B + (lw >> 1) * 8192, lA = lds0 + LDS_A + a0 * 1024;
    unsigned lB4 = lB + (lw & 1) * 4096, o4 = (lw & 1) * 32 * NN;
    asm volatile("" : "+v"(vW[0][0]), "+v"(vW[0][1]), "+v"(vW[1][2]), "+v"(vW[1][3]), "+v"(vX[0]), "+v"(vX[1]), "+v"(vX[2]), "+s"(lB4), "+s"(o4), "+s"(lA));
    ARGS();
    __builtin_amdgcn_s_setprio(LOADER_PRIO);
    const u32x4 srdW = SRD(ka[2], ka[3], 4096u * NN);
    const u32x4 srdX = SRD(ka[0], ka[1], 4096u * KC);
    const u32x4 srdS = SRD(ka[4], ka[5], CNT_OFF);
    auto W_stage = [&](auto Sc) {
      constexpr int S = Sc.value;
      w_burst<0, 4, w_slot(S)>(srdW, vW, ((unsigned)(src_of(rank, S >> 2) * 4 + q_of(S)) << 19) + o4, lB4);
    };
    auto X_local = [&](auto Sc) {
      constexpr int S = Sc.value;
      const unsigned so = (unsigned)(rank * MS) * KC + q_of(S) * 128;
      set_m0<S * ASLOT>(lA);
      if (a3) dma<3>(srdX, vX[0], vX[1], vX[2], vX[2], so); else dma<2>(srdX, vX[0], vX[1], vX[2], vX[2], so);
    };
    W_stage(IC(0)); X_local(IC(0)); W_stage(IC(4));
    W_stage(IC(1)); X_local(IC(1));
    const unsigned call = sload(cnt) + 1;
    const int Sl = lane & 31;
    const unsigned tabW = (unsigned)(src_of(rank, Sl >> 2) * 4 + q_of(Sl)) << 19;
    const unsigned tabX = src_of(rank, Sl >> 2) * SRC_STRIDE + (call & 1) * XBUF + q_of(Sl) * 65536;
    wait_vm<10>(); BARRIER();
    W_stage(IC(5)); W_stage(IC(2)); X_local(IC(2)); X_local(IC(3));
    wait_vm<12>(); BARRIER();
    W_stage(IC(3)); W_stage(IC(6));
    wait_vm<10>(); BARRIER();
    W_stage(IC(7)); W_stage(IC(9));
    wait_vm<12>(); BARRIER();
    W_stage(IC(10));
    BARRIER();
    W_stage(IC(8));
    if (odd) load_loop<1, 2>(srdW, srdS, vW, vS0, vS1, tabW, tabX, lB, lA);
    else if (a3) load_loop<0, 3>(srdW, srdS, vW, vS0, vS1, tabW, tabX, lB, lA);
    else load_loop<0, 2>(srdW, srdS, vW, vS0, vS1, tabW, tabX, lB, lA);
    if (threadIdx.x == 256) __hip_atomic_store(cnt, call, __ATOMIC_RELAXED, __HIP_MEMORY_SCOPE_AGENT);
    return;
  }

  }

  {
    const int v = b7 + SCU * wave;
    if (b < 7 * SCU && v < 14) {
      unsigned row0[4], q[4], fl[4], so[4]; bool big[4];
#pragma unroll
      for (int n = 0; n < 4; ++n) {
        const int pi = v + 14 * n, h = pi & 1;
        int mt = (pi >> 3) + 1 + dj;
        mt -= mt >= 7 ? 7 : 0;
        q[n] = (pi >> 1) & 3;
        big[n] = mt < 4;
        row0[n] = unit_begin(mt) * 16 + h * (big[n] ? 40 : 32);
        fl[n] = mt * FLAG_LINE + (q[n] * 2 + h) * FLAG_STEP;
        so[n] = row0[n] * KC + q[n] * 128;
      }
      unsigned vL[5];
#pragma unroll
      for (int i = 0; i < 5; ++i) vL[i] = ((lane >> 3) + 8 * i) * KC + (lane & 7) * 16;
      asm volatile("" : "+v"(vL[0]), "+v"(vL[1]), "+v"(vL[2]), "+v"(vL[3]), "+v"(vL[4]), "+s"(so[0]), "+s"(so[1]));
      ARGS();
      const u32x4 srdR = SRD(kl[0], kl[1], (unsigned)MS * KC);
      u32x4 r[4][5];
      auto load = [&](int n) {
#pragma unroll
        for (int i = 0; i < 5; ++i) load16(r[n][i], srdR, vL[i], so[n]);
      };
      STAMP(7);
      load(0);
      load(1);
      BARRIER();
      touch(SRD(kl[2], kl[3], SRC_STRIDE), (lane & 3) * (SRC_STRIDE / 3 - 64 & ~63u), 0, (unsigned)(size_t)(LDSP unsigned char*)smem);
      const unsigned call = sload(cnt) + 1;
      unsigned char* const dreg = ptr_of<unsigned char>(kl[2], kl[3]);
      unsigned char* const dbase = dreg + (call & 1) * XBUF + lane * 16;
      asm volatile("s_waitcnt vmcnt(6)" : "+v"(r[0][0]), "+v"(r[0][1]), "+v"(r[0][2]), "+v"(r[0][3]), "+v"(r[0][4]) :: "memory");
      const unsigned tref = now32();
      STAMP(1);
#pragma unroll
      for (int n = 0; n < 4; ++n) {
        while (now32() - tref < (unsigned)((v + 14 * n) * PACE_TICKS)) __builtin_amdgcn_s_sleep(1);
        unsigned char* d = dbase + q[n] * 65536 + row0[n] * 128;
#pragma unroll
        for (int i = 0; i < 4; ++i) store16_sys(d + i * 1024, r[n][i]);
        if (big[n]) store16_sys(d + 4096, r[n][4]);
        if (n == 0) {
          load(2);
          asm volatile("s_waitcnt vmcnt(5)" : "+v"(r[1][0]), "+v"(r[1][1]), "+v"(r[1][2]), "+v"(r[1][3]), "+v"(r[1][4]) :: "memory");
        } else if (n == 1) {
          load(3);
          asm volatile("s_waitcnt vmcnt(5)" : "+v"(r[2][0]), "+v"(r[2][1]), "+v"(r[2][2]), "+v"(r[2][3]), "+v"(r[2][4]) :: "memory");
        } else if (n == 2) {
          asm volatile("s_waitcnt vmcnt(0)" : "+v"(r[3][0]), "+v"(r[3][1]), "+v"(r[3][2]), "+v"(r[3][3]), "+v"(r[3][4]) :: "memory");
        } else {
          wait_vm<0>();
        }
        if (lane == 0) store4_sys(dreg + SRC_FLAG + fl[n], call);
        STAMP(2 + n);
      }
      BARRIER();
      STAMP(6);
      if (threadIdx.x == 0) __hip_atomic_store(cnt, call, __ATOMIC_RELAXED, __HIP_MEMORY_SCOPE_AGENT);
      if (wave == 0) DUMP(2);
    } else {
    }
  }
}

}

extern "C" void kernel_launch(int rank, int nranks, ncclComm_t comm,
                              hipStream_t stream,
                              void* const* d_in, const int* in_sizes, int n_in,
                              void* d_out, int out_size,
                              const int* in_dtypes, int out_dtype,
                              void* const* d_out_all,
                              void* const* d_ws_all, size_t ws_bytes) {
  if (nranks != NR || (unsigned)rank >= (unsigned)NR || ws_bytes < WS_USED || n_in < 4 || out_size != MS * NN) {
    fprintf(stderr, "unexpected problem geometry\n");
    return;
  }
  static bool init = false;
  if (!init) {
    HIP_CHECK(hipFuncSetAttribute((const void*)a2a_gemm_kernel, hipFuncAttributeMaxDynamicSharedMemorySize, LDS_BYTES));
    init = true;
  }
  Params p;
  p.x = (const signed char*)d_in[0];
  p.w = (const signed char*)d_in[1];
  p.sx = (const float*)d_in[2];
  p.sw = (const float*)d_in[3];
  p.out = (float*)d_out;
  p.myws = (unsigned char*)d_ws_all[rank];
  for (int j = 0; j < NR; ++j) {
    const int dst = (rank + 1 + j) % NR;
    p.link[j].rows = p.x + (size_t)dst * MS * KC;
    p.link[j].region = (unsigned char*)d_ws_all[dst] + (size_t)rank * SRC_STRIDE;
  }
  p.rank = rank;
  p.pad[0] = p.pad[1] = p.pad[2] = 0;
  a2a_gemm_kernel<<<256, 576, LDS_BYTES, stream>>>(p);
  HIP_CHECK(hipGetLastError());
}
```

```cpp
#include <hip/hip_runtime.h>
#include <rccl/rccl.h>
#include <cstdio>
#include <type_traits>

#define HIP_CHECK(expr) do { hipError_t _e = (expr); if (_e != hipSuccess) { \
    fprintf(stderr, "HIP error %s at %s:%d\n", hipGetErrorString(_e), __FILE__, __LINE__); \
    return; } } while (0)

namespace {

typedef int v4i __attribute__((ext_vector_type(4)));
typedef unsigned u32x4 __attribute__((ext_vector_type(4)));
#define LDSP __attribute__((address_space(3)))

constexpr int NR = 8;
constexpr int MS = 512;
constexpr int KC = 512;
constexpr int NN = 4096;
constexpr int NMT = 7;
constexpr int NSEND_WG = 32;

constexpr unsigned XBUF = 4 * 512 * 128;
constexpr unsigned SRC_FLAG = 2 * XBUF;
constexpr unsigned SRC_STRIDE = SRC_FLAG + 16384;
constexpr unsigned BEACON_OFF = NR * SRC_STRIDE;
constexpr unsigned CNT_OFF = BEACON_OFF + 128;
constexpr unsigned TMO_OFF = CNT_OFF + 256 * 128;
constexpr unsigned DIAG_OFF = TMO_OFF + 128;
constexpr unsigned WS_USED = DIAG_OFF + 256 * 128 * 4;

#define PACE_TICKS 8
#define LOADER_PRIO 3
#define SCU 4
#define ROTMASK 0xf8
static_assert(!(ROTMASK & 2), "position 1 is taken in sending order");
constexpr unsigned ORD0 = 0x76543210u;
constexpr unsigned long long DEADLINE_TICKS = 50000000ull;

constexpr int NA = 4, NB = 7;
constexpr unsigned BSLOT = 16384, ASLOT = 10240;
constexpr unsigned LDS_B = 0, LDS_A = NB * BSLOT, LDS_SNAP = LDS_A + NA * ASLOT, LDS_TMO = LDS_SNAP + 256;
constexpr unsigned LDS_JUNK = LDS_TMO + 64, LDS_BYTES = LDS_JUNK + 256;
#define EARLY_0 4
#define EARLY_1 4
#define POLLD 4
#define POLLP 1
#define SPIN_SLEEP 3

struct Link { const signed char* rows; unsigned char* region; };
struct Params {
  const signed char* x;
  const signed char* w;
  unsigned char* myws;
  float* out;
  const float* sx;
  const float* sw;
  int rank;
  int pad[3];
  Link link[NR];
};
typedef unsigned u32x8 __attribute__((ext_vector_type(8)));
typedef unsigned u32x2 __attribute__((ext_vector_type(2)));
template <class T> __device__ __forceinline__ T* ptr_of(unsigned lo, unsigned hi) { return (T*)(((unsigned long long)hi << 32) | lo); }

template <int N> __device__ __forceinline__ void wait_vm() { asm volatile("s_waitcnt vmcnt(%0)" :: "i"(N) : "memory"); }
template <int N, int I = 0, class F> __device__ __forceinline__ void static_for(F&& f) {
  if constexpr (I < N) { f(std::integral_constant<int, I>{}); static_for<N, I + 1>(f); }
}
#define IC(x) std::integral_constant<int, (x)>{}
#define BARRIER() __builtin_amdgcn_s_barrier()

#define XBITS " sc0 sc1"
#define DMA_ONE(n, bits) ".if %[cnt] > " #n "\n\tbuffer_load_dwordx4 %[v" #n "], %[srd], %[so] offen offset:1024*" #n bits " lds\n\t.endif\n\t"
#define DMA_BURST(bits) \
  asm volatile(DMA_ONE(0, bits) DMA_ONE(1, bits) DMA_ONE(2, bits) DMA_ONE(3, bits) \
               :: [v0] "v"(v0), [v1] "v"(v1), [v2] "v"(v2), [v3] "v"(v3), [srd] "s"(srd), [so] "s"(soff), [cnt] "i"(N) : "memory")
template <int N>
__device__ __forceinline__ void dma(u32x4 srd, unsigned v0, unsigned v1, unsigned v2, unsigned v3, unsigned soff) {
  DMA_BURST("");
}
template <int N>
__device__ __forceinline__ void dma_sys(u32x4 srd, unsigned v0, unsigned v1, unsigned v2, unsigned v3, unsigned soff) {
  DMA_BURST(XBITS);
}
#define SET_M0(base, IMM, TAIL) asm volatile("s_add_u32 m0, %0, %1\n\t" TAIL :: "s"(base), "i"(IMM) : "memory", "scc")
template <int IMM> __device__ __forceinline__ void set_m0(unsigned base) { SET_M0(base, IMM, "s_nop 0"); }
template <int IMM> __device__ __forceinline__ void set_m0_barrier(unsigned base) { SET_M0(base, IMM, "s_barrier"); }
template <int IMM> __device__ __forceinline__ void set_m0_nop_barrier(unsigned base) { SET_M0(base, IMM, "s_nop 3\n\ts_barrier"); }
__device__ __forceinline__ void dma4_sys(u32x4 srd, unsigned voff, unsigned lds, unsigned long long mask) {
  asm volatile("s_mov_b32 m0, %2\n\ts_mov_b64 exec, %3\n\tbuffer_load_dword %0, %1, 0 offen sc0 sc1 lds\n\ts_mov_b64 exec, -1"
               :: "v"(voff), "s"(srd), "s"(lds), "s"(mask) : "memory");
}
__device__ __forceinline__ void touch(u32x4 srd, unsigned voff, unsigned soff, unsigned lds) {
  asm volatile("s_mov_b32 m0, %3\n\ts_nop 0\n\tbuffer_load_dword %0, %1, %2 offen lds" :: "v"(voff), "s"(srd), "s"(soff), "s"(lds) : "memory");
}
__device__ __forceinline__ void load16(u32x4& v, u32x4 srd, unsigned voff, unsigned soff) {
  asm volatile("buffer_load_dwordx4 %0, %1, %2, %3 offen" : "=v"(v) : "v"(voff), "s"(srd), "s"(soff) : "memory");
}
__device__ __forceinline__ void store16_sys(void* ptr, u32x4 v) {
  asm volatile("global_store_dwordx4 %0, %1, off sc0 sc1\n\ts_nop 1" :: "v"(ptr), "v"(v) : "memory");
}
__device__ __forceinline__ void store4_sys(void* ptr, unsigned v) {
  asm volatile("global_store_dword %0, %1, off sc0 sc1" :: "v"(ptr), "v"(v) : "memory");
}
__device__ __forceinline__ unsigned sload(const void* p) {
  unsigned v;
  asm volatile("s_load_dword %0, %1, 0x0\n\ts_waitcnt lgkmcnt(0)" : "=s"(v) : "s"(p) : "memory");
  return v;
}
__device__ __forceinline__ unsigned now32() { return (unsigned)__builtin_amdgcn_s_memrealtime(); }
__device__ __forceinline__ unsigned xrow_off(unsigned row) { return (row >> 6) * 16384 + (row & 63) * 128; }
__device__ __forceinline__ unsigned xq_off(unsigned q) { return (q >> 1) * 131072 + (q & 1) * 8192; }
__device__ __forceinline__ int unit_begin(int mt) { return 4 * mt + min(mt, 4); }

constexpr int LEAD = NA - 1;
constexpr int w_lo(int t, int odd) { return t < 8 ? 8 * (t + 4) + 4 * odd : 88 + 7 * (t - 8) + (odd ? 3 : 0); }
constexpr int w_cnt(int t, int odd) { return t == 4 ? 4 : t < 8 || t > 31 ? 0 : odd ? 4 : 3; }
constexpr int in_flight(int t, int odd, int nx) {
  const int S = t + 1 - LEAD;
  int n = 0;
  for (int k = t; k >= S; --k) {
    if (k < t) for (int i = w_cnt(k, odd) - 1; i >= 0; --i) { if (((w_lo(k, odd) + i) >> 3) <= S) return n; ++n; }
    if (k > S && k < 32) n += nx;
  }
  return n;
}
constexpr bool w_schedule_ok() {
  for (int t = 8; t < 32; ++t) {
    if (((w_lo(t, 1) + 3) >> 3) > t - LEAD - 1 + NB) return false;
    if ((w_lo(t, 0) >> 3) <= t + 1 - LEAD) return false;
  }
  return w_lo(32, 0) == 256;
}
static_assert(w_schedule_ok() && NB == 7 && NA == 4, "W schedule");
constexpr int w_slot(int S) { return S < 3 ? 4 + S : S == 3 ? 4 : (S - 4) % NB; }

#define STAMP(i) do { } while (0)
#define STAMPC(i) do { } while (0)
#define DUMP(slot) do { } while (0)

#define WB0(s) "v[" #s "+0:" #s "+3]"
#define WB1(s) "v[" #s "+4:" #s "+7]"
#define XU(s, u) "v[" #s "+8+4*" #u ":" #s "+11+4*" #u "]"
#define MF(acc, w, x) "v_mfma_i32_16x16x64_i8 %[" #acc "], " x ", " w ", %[" #acc "]\n\t"
#define TRL(s, o, p, off) "ds_read_b64_tr_b8 v[" #s "+" #o ":" #s "+" #o "+1], %[" #p "] offset:" #off "\n\t"
#define RX(s, u, p, k) "ds_read_b128 " XU(s, u) ", %[" #p "] offset:2048*" #u "+%[aslot]*" #k "\n\t"
#define IF5(x) ".if %[nu] == 5\n\t" x ".endif\n\t"
#define IF4(x) ".if %[nu] == 4\n\t" x ".endif\n\t"
#define TICK_STAMP
#define DG_OPERAND
#define DG_PARAM
#define DG_ARG
#define LOOP_STAMP

#define HALF_A(k, STAMP) \
    MF(c00, WB0(92), XU(92, 0)) \
    MF(c01, WB1(92), XU(92, 0)) \
    "s_waitcnt lgkmcnt(0)\n\ts_barrier\n\t" STAMP \
    TRL(64, 0, pw0, 0) IF4(TRL(64, 2, pw0, 1024)) \
    MF(c11, WB1(92), XU(92, 1)) IF5(TRL(64, 2, pw0, 1024)) IF4(RX(64, 0, pa0, k)) \
    MF(c10, WB0(92), XU(92, 1)) IF5(RX(64, 0, pa0, k)) IF4(TRL(64, 4, pw1, 0)) \
    MF(c20, WB0(92), XU(92, 2)) IF5(TRL(64, 4, pw1, 0)) IF4(TRL(64, 6, pw1, 1024)) \
    MF(c21, WB1(92), XU(92, 2)) IF5(TRL(64, 6, pw1, 1024)) IF4(RX(64, 1, pa0, k)) \
    MF(c31, WB1(92), XU(92, 3)) IF5(RX(64, 1, pa0, k)) IF4(RX(64, 2, pa0, k)) \
    MF(c30, WB0(92), XU(92, 3)) IF5(RX(64, 2, pa0, k)) IF4(RX(64, 3, pa0, k)) \
    IF5(MF(c40, WB0(92), XU(92, 4)) RX(64, 3, pa0, k) \
    MF(c41, WB1(92), XU(92, 4)) RX(64, 4, pa0, k))
#define HALF_B(k, EXTRA) \
    "s_waitcnt lgkmcnt(%[nu]-1)\n\t" \
    MF(c00, WB0(64), XU(64, 0)) TRL(92, 0, pw0, 8192) TRL(92, 2, pw0, 9216) \
    MF(c01, WB1(64), XU(64, 0)) RX(92, 0, pa1, k) \
    "s_waitcnt lgkmcnt(%[nu])\n\t" \
    MF(c11, WB1(64), XU(64, 1)) TRL(92, 4, pw1, 8192) TRL(92, 6, pw1, 9216) \
    MF(c10, WB0(64), XU(64, 1)) RX(92, 1, pa1, k) "s_bitcmp1_b32 %[wraps], " #k "\n\t" \
    MF(c20, WB0(64), XU(64, 2)) RX(92, 2, pa1, k) "s_cselect_b32 %[dB], %[wrapB], %[stepB]\n\t" \
    MF(c21, WB1(64), XU(64, 2)) RX(92, 3, pa1, k) \
    "s_waitcnt lgkmcnt(8)\n\t" \
    MF(c31, WB1(64), XU(64, 3)) IF5(RX(92, 4, pa1, k)) "v_add_u32 %[pw0], %[dB], %[pw0]\n\t" \
    IF5(MF(c30, WB0(64), XU(64, 3))) "v_add_u32 %[pw1], %[dB], %[pw1]\n\t" \
    IF5(MF(c40, WB0(64), XU(64, 4))) EXTRA \
    IF5(MF(c41, WB1(64), XU(64, 4))) IF4(MF(c30, WB0(64), XU(64, 3))) \
    "s_waitcnt lgkmcnt(%[nu]-1)\n\t"

constexpr unsigned wrap_mask() {
  unsigned m = 0;
  for (int S = 0; S < 31; ++S) if (S != 3 && w_slot(S + 1) < w_slot(S)) m |= 1u << S;
  return m;
}

template <int NU>
__device__ __forceinline__ void k_loop(v4i (&acc)[5][2], unsigned pW0, unsigned pW1, unsigned pA0, unsigned pA1 DG_PARAM) {
  unsigned C = 0, wraps = wrap_mask(), dB, wrapB = -2 * (int)BSLOT;
  asm volatile(
    "s_branch 2f\n\t"
    "1:\n\t"
    HALF_A(0, LOOP_STAMP)
    "3:\n\t"
    HALF_B(0, "")
    HALF_A(1, "") HALF_B(1, "")
    HALF_A(2, "") HALF_B(2, "s_add_u32 %[C], %[C], 1\n\t")
    HALF_A(3, "") HALF_B(3, "s_lshr_b32 %[wraps], %[wraps], 4\n\ts_bitcmp0_b32 %[special], %[C]\n\t")
    "s_cbranch_scc1 1b\n\t"
    MF(c00, WB0(92), XU(92, 0)) MF(c01, WB1(92), XU(92, 0))
    "s_waitcnt lgkmcnt(0)\n\t"
    MF(c11, WB1(92), XU(92, 1)) MF(c10, WB0(92), XU(92, 1))
    MF(c20, WB0(92), XU(92, 2)) MF(c21, WB1(92), XU(92, 2)) MF(c31, WB1(92), XU(92, 3)) MF(c30, WB0(92), XU(92, 3))
    IF5(MF(c40, WB0(92), XU(92, 4)) MF(c41, WB1(92), XU(92, 4)))
    "s_cmp_eq_u32 %[C], 8\n\t"
    "s_cbranch_scc1 5f\n\t"
    "s_mov_b32 %[wrapB], %[back5]\n\t"
    "v_add_u32 %[pw0], %[wrapB], %[pw0]\n\tv_add_u32 %[pw1], %[wrapB], %[pw1]\n\t"
    "s_mov_b32 %[wrapB], %[back6]\n\t"
    ".rept %[nbar]\n\ts_barrier\n\t.endr\n\t"
    "2:\n\t"
    "s_barrier\n\t"
    TICK_STAMP
    TRL(64, 0, pw0, 0) TRL(64, 2, pw0, 1024) RX(64, 0, pa0, 0) TRL(64, 4, pw1, 0) TRL(64, 6, pw1, 1024)
    RX(64, 1, pa0, 0) RX(64, 2, pa0, 0) RX(64, 3, pa0, 0) IF5(RX(64, 4, pa0, 0))
    "s_branch 3b\n\t"
    "5:\n\t"
    "s_nop 7\n\t"
    : [c00] "+v"(acc[0][0]), [c01] "+v"(acc[0][1]), [c10] "+v"(acc[1][0]), [c11] "+v"(acc[1][1]), [c20] "+v"(acc[2][0]),
      [c21] "+v"(acc[2][1]), [c30] "+v"(acc[3][0]), [c31] "+v"(acc[3][1]), [c40] "+v"(acc[4][0]), [c41] "+v"(acc[4][1]),
      [pw0] "+&v"(pW0), [pw1] "+&v"(pW1),
      [C] "+&s"(C), [wraps] "+&s"(wraps), [dB] "=&s"(dB), [wrapB] "+&s"(wrapB) DG_OPERAND
    : [pa0] "v"(pA0), [pa1] "v"(pA1), [nu] "i"(NU), [stepB] "i"(BSLOT), [aslot] "i"(ASLOT),
      [back5] "i"(-5 * (int)BSLOT), [back6] "i"(-(NB - 1) * (int)BSLOT),
      [special] "s"((1u << 1) | (1u << 8)), [nbar] "i"(LEAD)
    : "memory", "scc", "s90", "s91", "s92", "s93",
      "v64", "v65", "v66", "v67", "v68", "v69", "v70", "v71", "v72", "v73", "v74", "v75", "v76", "v77", "v78", "v79",
      "v80", "v81", "v82", "v83", "v84", "v85", "v86", "v87", "v88", "v89", "v90", "v91", "v92", "v93", "v94", "v95",
      "v96", "v97", "v98", "v99", "v100", "v101", "v102", "v103", "v104", "v105", "v106", "v107", "v108", "v109", "v110",
      "v111", "v112", "v113", "v114", "v115", "v116", "v117", "v118", "v119");
}

constexpr int in_flight_loop(int k, int odd, int nx) {
  int m = 63;
  for (int it = 1; it < 3; ++it) m = in_flight(8 + 8 * it + k, odd, nx) < m ? in_flight(8 + 8 * it + k, odd, nx) : m;
  return m;
}

template <int G, int N, int SLOT>
__device__ __forceinline__ void w_burst(u32x4 srdW, const unsigned (&vW)[2][4], unsigned stage_off, unsigned lB) {
  unsigned so;
  asm volatile("s_add_u32 m0, %1, %2\n\ts_add_u32 %0, %3, %4" : "=s"(so) : "s"(lB), "i"(SLOT * BSLOT + G * 1024), "s"(stage_off), "i"(G * 8 * NN) : "memory", "scc");
  dma<N>(srdW, vW[(G >> 1) & 1][0], vW[((G + 1) >> 1) & 1][1], vW[((G + 2) >> 1) & 1][2], vW[((G + 3) >> 1) & 1][3], so);
}

template <int ODD, int NX>
__device__ __forceinline__ void load_loop(u32x4 srdW, u32x4 srdS, const unsigned (&vW)[2][4], const unsigned (&vS)[3],
                                          unsigned tabW, unsigned tabX, unsigned lB, unsigned lA) {
  auto lane_of = [](unsigned tab, int i) { unsigned r = __builtin_amdgcn_readlane(tab, i); asm volatile("" : "+s"(r)); return r; };
  unsigned sx = lane_of(tabX, 4), sw = lane_of(tabW, 11), swn = lane_of(tabW, 12);
  static_for<4>([&](auto kc) {
    constexpr int t = 4 + kc.value;
    if constexpr (t == 4) { asm volatile("s_nop 3"); set_m0<(t % NA) * ASLOT>(lA); } else set_m0_nop_barrier<(t % NA) * ASLOT>(lA);
    dma_sys<NX>(srdS, vS[0], vS[1], vS[2], vS[2], sx);
    sx = lane_of(tabX, t + 1);
    if constexpr (t >= 3 + LEAD) wait_vm<in_flight(t, ODD, NX)>();
    if constexpr (t == 4) w_burst<4 * ODD, 4, w_slot(t + 4)>(srdW, vW, lane_of(tabW, t + 4), lB);
  });
  asm volatile("s_nop 3");
#pragma unroll 1
  for (int it = 0; it < 3; ++it) {
    static_for<8>([&](auto kc) {
      constexpr int k = kc.value, t = 8 + k;
      constexpr int h = w_lo(t, ODD), n = w_cnt(t, ODD), Sc = w_lo(t, 0) >> 3;
      set_m0_barrier<(t % NA) * ASLOT>(lA);
      dma_sys<NX>(srdS, vS[0], vS[1], vS[2], vS[2], sx);
      sx = lane_of(tabX, t + 1 + 8 * it);
      unsigned swnn = 0;
      if constexpr ((w_lo(t + 1, 0) >> 3) != Sc) swnn = lane_of(tabW, Sc + 2 + 7 * it);
      if constexpr (in_flight(t, ODD, NX) < in_flight_loop(k, ODD, NX)) { if (it == 0) wait_vm<in_flight(t, ODD, NX)>(); }
      wait_vm<in_flight_loop(k, ODD, NX)>();
      if constexpr (((h + n - 1) >> 3) == (h >> 3)) {
        w_burst<h & 7, n, w_slot(h >> 3)>(srdW, vW, (h >> 3) == Sc ? sw : swn, lB);
      } else {
        w_burst<h & 7, 8 - (h & 7), w_slot(Sc)>(srdW, vW, sw, lB);
        w_burst<0, n - 8 + (h & 7), w_slot(Sc + 1)>(srdW, vW, swn, lB);
      }
      if constexpr ((w_lo(t + 1, 0) >> 3) != Sc) { sw = swn; swn = swnn; }
    });
  }
  static_for<LEAD>([&](auto kc) {
    constexpr int t = 32 + kc.value;
    BARRIER();
    if constexpr (t < 31 + LEAD) wait_vm<in_flight(t, ODD, NX)>();
  });
}

__global__ __attribute__((flatten)) __launch_bounds__(576, 3) void a2a_gemm_kernel(Params p) {
  extern __shared__ __attribute__((aligned(16))) unsigned char smem[];
  const int b = blockIdx.x;
  const int wave = __builtin_amdgcn_readfirstlane(threadIdx.x >> 6);
  const int lane = threadIdx.x & 63;
  STAMP(0);
  const int b7 = (b & 31) * 37 >> 8, dj = (b & 31) - 7 * b7;
  asm volatile("s_load_dword s100, %0, 0x0\n\ts_load_dword s101, %0, %1"
               :: "s"(__builtin_amdgcn_kernarg_segment_ptr()), "s"(64 + 16 * dj) : "memory", "s100", "s101");
#define ARGS() u32x8 ka, kb; u32x4 kl; \
  asm volatile("s_load_dwordx8 %0, %3, 0x0\n\ts_load_dwordx8 %1, %3, 0x20\n\ts_load_dwordx4 %2, %3, %4\n\ts_waitcnt lgkmcnt(0)" \
               : "=&s"(ka), "=&s"(kb), "=&s"(kl) : "s"(__builtin_amdgcn_kernarg_segment_ptr()), "s"(64 + 16 * dj) : "memory"); \
  const int rank = kb[4]; unsigned char* const myws = ptr_of<unsigned char>(ka[4], ka[5]); \
  unsigned* const cnt = (unsigned*)(myws + CNT_OFF + b * 128); (void)rank; (void)cnt
#define SRD(lo, hi, bytes) u32x4{lo, (hi) & 0xffffu, bytes, 0x00020000u}
  if (wave == 3 && b >= NSEND_WG && b < NSEND_WG + 8 * 4) {
    unsigned long long pc; int len;
    asm volatile("s_getpc_b64 %0\n.Lcode_pf:\n\ts_mov_b32 %1, .Lcode_pf_end-.Lcode_pf" : "=s"(pc), "=s"(len));
    touch(SRD((unsigned)pc, (unsigned)(pc >> 32), (unsigned)max(len, 0)), lane * 128, ((b - NSEND_WG) >> 3) * 8192, (unsigned)(size_t)(LDSP unsigned char*)smem + LDS_JUNK);
  }

  if (b >= NSEND_WG) {
  const int g = (b - NSEND_WG) >> 3, xcd = b & 7;
  const int ntl = g * 37 >> 8, mt = g - NMT * ntl, nt = xcd * 4 + ntl;
  const int ub = unit_begin(mt), nu = mt < 4 ? 5 : 4;
  volatile LDSP unsigned* const ltmo = (volatile LDSP unsigned*)(smem + LDS_TMO);
  const unsigned lds0 = (unsigned)(size_t)(LDSP unsigned char*)smem;
  auto src_of = [&](int rank, int js) { int sj = js - 1 - mt; sj += sj < 0 ? 7 : 0; return js == 0 ? rank : (rank + 1 + sj) & 7; };
  auto q_of = [&](int S) { return (S + (S < 4 ? mt + ntl : ((ROTMASK >> (S >> 2)) & 1) ? ntl + (S >> 2) : 0)) & 3; };

  if (wave < 4) {
  const int fg = lane >> 4, fq = (lane & 15) >> 1, fp = lane & 1;
  const unsigned pW0 = lds0 + LDS_B + w_slot(0) * BSLOT + (16 * fg + fq) * 128 + (((2 * wave) ^ (((fq >> 1) & 3) | ((fg & 1) << 2))) << 4) + 8 * fp;
  const unsigned pA0 = lds0 + LDS_A + (lane & 15) * 128 + ((fg ^ fq) << 4);
  unsigned vO[8];
#pragma unroll
  for (int i = 0; i < 8; ++i) {
    vO[i] = ((ub * 16 + (lane >> 5) * 8 + i) * NN + nt * 128 + wave * 32 + (lane & 31)) * 4;
    asm volatile("" : "+v"(vO[i]));
  }
  ARGS();
  const u32x4 srdO = SRD(ka[6], ka[7], (unsigned)MS * NN * 4);
  {
    const int slow = (3 - (int)(ka[2] >> 7) - xcd * 4) & 7;
    if (wave == 0) {
      touch(srdO, (ub * 16 + (lane >> 5) * (nu * 16 - 1)) * NN * 4 + nt * 512, 0, lds0 + LDS_JUNK);
    } else if (wave < 3 && slow < 4 && slow != ntl) {
      const int part = ((ntl - slow - 1) & 3) * 2 + wave - 1;
      const unsigned row = min(part * 43 + min(lane, 42), 255);
      touch(SRD(ka[2], ka[3], 4096u * NN), row * NN, (unsigned)(src_of(rank, 1) * KC + 256) * NN + (xcd * 4 + slow) * 128, lds0 + LDS_JUNK);
    }
  }
  float sx, sw;
  asm volatile("s_load_dword %0, %2, 0x0\n\ts_load_dword %1, %3, 0x0\n\ts_waitcnt lgkmcnt(0)"
               : "=&s"(sx), "=&s"(sw) : "s"(ptr_of<float>(kb[0], kb[1])), "s"(ptr_of<float>(kb[2], kb[3])) : "memory");
  float scale = sx * sw;
  asm volatile("" : "+v"(scale));
  v4i acc[5][2];
#pragma unroll
  for (int u = 0; u < 5; ++u) { acc[u][0] = v4i{0, 0, 0, 0}; acc[u][1] = v4i{0, 0, 0, 0}; }
  if (nu == 5) k_loop<5>(acc, pW0, pW0 ^ 16, pA0, pA0 ^ 64 DG_ARG);
  else k_loop<4>(acc, pW0, pW0 ^ 16, pA0, pA0 ^ 64 DG_ARG);
  if (*ltmo) scale = __builtin_nanf("");
  STAMP(11);
  STAMPC(17);

#pragma unroll
  for (int u = 0; u < 5; ++u) {
    if (u < nu) {
#pragma unroll
      for (int r = 0; r < 4; ++r) {
        const float y0 = __builtin_elementwise_maximum((float)acc[u][0][r] * scale, 0.f);
        const float y1 = __builtin_elementwise_maximum((float)acc[u][1][r] * scale, 0.f);
        const auto y = __builtin_amdgcn_permlane16_swap(__builtin_bit_cast(unsigned, y0), __builtin_bit_cast(unsigned, y1), false, false);
#define OUT_BITS " sc1"
        asm volatile("buffer_store_dword %0, %1, %2, %3 offen" OUT_BITS :: "v"(y[0]), "v"(vO[r]), "s"(srdO), "s"(u * 16 * NN * 4) : "memory");
        asm volatile("buffer_store_dword %0, %1, %2, %3 offen" OUT_BITS :: "v"(y[1]), "v"(vO[r + 4]), "s"(srdO), "s"(u * 16 * NN * 4) : "memory");
      }
    }
  }
  asm volatile(".Lcode_pf_end:");
  return;
  }

  if (wave == 8) {
    volatile LDSP unsigned* const snap = (volatile LDSP unsigned*)(smem + LDS_SNAP);
    snap[lane] = 0;
    if (lane < 2) ltmo[lane] = lane ? ORD0 : 0;
    ARGS();
    __builtin_amdgcn_s_setprio(3);
    const unsigned call = sload(cnt) + 1;
    unsigned* const tmo = (unsigned*)(myws + TMO_OFF);
    const u32x4 srdS = SRD(ka[4], ka[5], CNT_OFF);
    const int srcl = src_of(rank, lane < 8 ? lane : lane >> 3);
    const unsigned vF = lane < 8 ? BEACON_OFF + srcl * 4 : srcl * SRC_STRIDE + SRC_FLAG + (mt * 4 + ((lane >> 1) & 3)) * 128 + (lane & 1) * 4;
    touch(srdS, srcl * SRC_STRIDE + ((lane & 1) ? SRC_FLAG + 8192 : (call & 1) * XBUF), 0, lds0 + LDS_JUNK);
    bool dead = false;
    unsigned long long t0 = 0;
    auto expired = [&](unsigned it) {
      if (it == 0) t0 = __builtin_amdgcn_s_memrealtime();
      if ((it & 63) != 63) return false;
      const unsigned t = __builtin_amdgcn_readfirstlane(__hip_atomic_load(tmo, __ATOMIC_RELAXED, __HIP_MEMORY_SCOPE_AGENT));
      if (t != call && __builtin_amdgcn_s_memrealtime() - t0 <= DEADLINE_TICKS) return false;
      __hip_atomic_store(tmo, call, __ATOMIC_RELAXED, __HIP_MEMORY_SCOPE_AGENT);
      ltmo[0] = 1;
      asm volatile("s_waitcnt lgkmcnt(0)" ::: "memory");
      return true;
    };
    constexpr unsigned long long FIRST = 0x030303030303fffeull;
    const unsigned char* const f1 = myws + (unsigned)__builtin_amdgcn_readlane(vF, 8);
    unsigned early = 0;
    auto early_poll = [&](unsigned copy) {
      u32x8 be; u32x2 f[4];
      asm volatile("s_load_dwordx2 %1, %6, 0x0\n\ts_load_dwordx8 %0, %5, 0x0\n\ts_load_dwordx2 %2, %6, 0x80\n\t"
                   "s_load_dwordx2 %3, %6, 0x100\n\ts_load_dwordx2 %4, %6, 0x180\n\ts_waitcnt lgkmcnt(0)"
                   : "=&s"(be), "=&s"(f[0]), "=&s"(f[1]), "=&s"(f[2]), "=&s"(f[3])
                   : "s"(myws + BEACON_OFF + copy), "s"(f1 + copy) : "memory");
      unsigned bm = 0;
#pragma unroll
      for (int r = 0; r < 8; ++r) bm |= be[r] >= call ? 1u << r : 0u;
      early |= (unsigned)__ballot(lane < 8 && ((bm >> srcl) & 1));
#pragma unroll
      for (int q = 0; q < 4; ++q) early |= min(f[q][0], f[q][1]) >= call ? 0x300u << (2 * q) : 0u;
    };
    BARRIER(); BARRIER(); BARRIER(); BARRIER();
    STAMP(9);
    __builtin_amdgcn_s_sleep(EARLY_0);
    early_poll(0);
    if (!(early & 0x100)) {
      dma4_sys(srdS, vF, lds0 + LDS_SNAP, FIRST);
      __builtin_amdgcn_s_sleep(EARLY_1);
      early_poll(64);
    }
    unsigned ord = ORD0;
    for (unsigned it = 0;; ++it) {
      const unsigned long long m = __ballot(snap[lane] >= call) | early;
      const unsigned long long first = m & (m >> 1) & 0x0101010101010100ull;
      if (first) {
        const int j0 = __builtin_ctzll(first) >> 3;
        const unsigned good = (((unsigned)m & 0xfe) | (1u << j0)) & (~0u << j0);
        if (good != 0xfe) {
          ord = 0;
          int p = 1;
          for (int pass = 0; pass < 2; ++pass)
            for (int j = 1; j < 8; ++j)
              if ((((good >> j) & 1) ^ pass) == 1) ord |= (unsigned)j << (4 * p++);
          ltmo[1] = ord;
          asm volatile("s_waitcnt lgkmcnt(0)" ::: "memory");
        }
        break;
      }
      if (expired(it)) { dead = true; break; }
      dma4_sys(srdS, vF, lds0 + LDS_SNAP, FIRST);
      __builtin_amdgcn_s_sleep(SPIN_SLEEP);
      wait_vm<8>();
    }
    volatile LDSP unsigned* const mine = snap + ((ord >> ((lane >> 3) * 4)) & 7) * 8 + q_of(lane >> 1) * 2 + (lane & 1);
    auto poll = [&](int from) {
      const unsigned o = ord >> (from & 28);
      dma4_sys(srdS, vF, lds0 + LDS_SNAP, (0xffull << ((o & 7) * 8)) | ((0xffull << ((o >> 4 & 7) * 8)) & ~0xffull));
    };
    if ((ord & 0xf0) != 0x10) early = 0;
    auto arrived = [&]() { const unsigned long long m = ~(__ballot(*mine >= call) | early | 0xff); return m ? (int)__builtin_ctzll(m) >> 1 : 32; };
    int avail = dead ? 32 : 4, next = 0;
    for (int S = 4; S < 32; ++S) {
      if (avail < 32 && avail <= S + POLLD) {
        avail = arrived();
        if (avail <= S) {
          for (unsigned it = 0;; ++it) {
            if (it) poll(avail);
            wait_vm<0>();
            avail = arrived();
            if (avail > S) break;
            if (expired(it)) { avail = 32; break; }
          }
        }
        if (avail < 32 && avail <= S + POLLD && S >= next) { poll(avail); next = S + POLLP; }
      }
      if ((S & 3) == 0) STAMP(S >> 2);
      BARRIER();
    }
    STAMP(8);
    DUMP(1);
    return;
  }

  {
    const int lw = wave - 4;
    const int a0 = nu == 5 ? (lw >> 1) * 5 + (lw & 1) * 3 : lw * 2;
    const bool odd = lw & 1, a3 = nu == 5 && !odd;
    const int arow = ub * 16 + a0 * 8;
    const unsigned swz = (lane & 7) ^ ((lane >> 4) & 3);
    unsigned vW[2][4], vX[3], vS[3];
#pragma unroll
    for (int i = 0; i < 4; ++i) {
      vW[0][i] = ((lw >> 1) * 64 + (lane >> 3)) * NN + nt * 128 + (swz << 4) + i * (8 * NN - 1024);
      vW[1][i] = vW[0][i] ^ 64;
      if (i < 3) vX[i] = (arow + (lane >> 3)) * KC + ((swz ^ (((a0 + i) & 1) << 2)) << 4) + i * (8 * KC - 1024);
      if (i < 3) vS[i] = xrow_off(arow + 8 * i) + (lane >> 3) * 128 + ((swz ^ (((a0 + i) & 1) << 2)) << 4) - i * 1024;
    }
    unsigned lB = lds0 + LDS_B + (lw >> 1) * 8192, lA = lds0 + LDS_A + a0 * 1024;
    unsigned lB4 = lB + (lw & 1) * 4096, o4 = (lw & 1) * 32 * NN;
    asm volatile("" : "+v"(vW[0][0]), "+v"(vW[0][1]), "+v"(vW[1][2]), "+v"(vW[1][3]), "+v"(vX[0]), "+v"(vX[1]), "+v"(vX[2]), "+s"(lB4), "+s"(o4), "+s"(lA));
    ARGS();
    __builtin_amdgcn_s_setprio(LOADER_PRIO);
    const u32x4 srdW = SRD(ka[2], ka[3], 4096u * NN);
    const u32x4 srdX = SRD(ka[0], ka[1], 4096u * KC);
    const u32x4 srdS = SRD(ka[4], ka[5], CNT_OFF);
    auto W_stage = [&](auto Sc) {
      constexpr int S = Sc.value;
      w_burst<0, 4, w_slot(S)>(srdW, vW, ((unsigned)(src_of(rank, S >> 2) * 4 + q_of(S)) << 19) + o4, lB4);
    };
    auto X_local = [&](auto Sc) {
      constexpr int S = Sc.value;
      const unsigned so = (unsigned)(rank * MS) * KC + q_of(S) * 128;
      set_m0<S * ASLOT>(lA);
      if (a3) dma<3>(srdX, vX[0], vX[1], vX[2], vX[2], so); else dma<2>(srdX, vX[0], vX[1], vX[2], vX[2], so);
    };
    W_stage(IC(0)); X_local(IC(0));
    W_stage(IC(4)); W_stage(IC(1)); X_local(IC(1));
    const unsigned call = sload(cnt) + 1;
    const int Sl = lane & 31;
    unsigned tabW, tabX;
    auto tables = [&](unsigned ord) {
      const int src = src_of(rank, (ord >> (Sl & 28)) & 7);
      tabW = (unsigned)(src * 4 + q_of(Sl)) << 19;
      tabX = src * SRC_STRIDE + (call & 1) * XBUF + xq_off(q_of(Sl));
    };
    tables(ORD0);
    wait_vm<10>(); BARRIER();
    W_stage(IC(5)); W_stage(IC(2)); X_local(IC(2)); X_local(IC(3));
    wait_vm<12>(); BARRIER();
    W_stage(IC(3)); W_stage(IC(6));
    wait_vm<10>(); BARRIER();
    W_stage(IC(7)); W_stage(IC(9));
    wait_vm<12>(); BARRIER();
    W_stage(IC(10));
    BARRIER();
    const unsigned ord = __builtin_amdgcn_readfirstlane(ltmo[1]);
    if (ord != ORD0) {
      tables(ord);
      static_for<7>([&](auto kc) {
        constexpr int S = 4 + kc.value;
        if (S != 8 && ((ord ^ ORD0) >> (S & 28) & 15)) w_burst<0, 4, w_slot(S)>(srdW, vW, __builtin_amdgcn_readlane(tabW, S) + o4, lB4);
      });
    }
    if (odd) load_loop<1, 2>(srdW, srdS, vW, vS, tabW, tabX, lB, lA);
    else if (a3) load_loop<0, 3>(srdW, srdS, vW, vS, tabW, tabX, lB, lA);
    else load_loop<0, 2>(srdW, srdS, vW, vS, tabW, tabX, lB, lA);
    if (threadIdx.x == 256) __hip_atomic_store(cnt, call, __ATOMIC_RELAXED, __HIP_MEMORY_SCOPE_AGENT);
    return;
  }

  }

  {
    const int v = b7 + SCU * wave;
    if (b < 7 * SCU && v < 14) {
      unsigned row0[4], q[4], fl[4], so[4]; bool big[4];
#pragma unroll
      for (int n = 0; n < 4; ++n) {
        const int pi = v + 14 * n, h = pi & 1;
        int mt = (pi >> 3) + 1 + dj;
        mt -= mt >= 7 ? 7 : 0;
        q[n] = (pi >> 1) & 3;
        big[n] = mt < 4;
        row0[n] = unit_begin(mt) * 16 + h * (big[n] ? 40 : 32);
        fl[n] = (mt * 4 + q[n]) * 128 + h * 4;
        so[n] = row0[n] * KC + q[n] * 128;
      }
      unsigned vL[5];
#pragma unroll
      for (int i = 0; i < 5; ++i) vL[i] = ((lane >> 3) + 8 * i) * KC + (lane & 7) * 16;
      asm volatile("" : "+v"(vL[0]), "+v"(vL[1]), "+v"(vL[2]), "+v"(vL[3]), "+v"(vL[4]), "+s"(so[0]), "+s"(so[1]));
      ARGS();
      const u32x4 srdR = SRD(kl[0], kl[1], (unsigned)MS * KC);
      u32x4 r[4][5];
      auto load = [&](int n) {
#pragma unroll
        for (int i = 0; i < 5; ++i) load16(r[n][i], srdR, vL[i], so[n]);
      };
      STAMP(7);
      load(0);
      load(1);
      BARRIER();
      touch(SRD(kl[2], kl[3], SRC_STRIDE), (lane & 3) * (SRC_STRIDE / 3 - 64 & ~63u), 0, (unsigned)(size_t)(LDSP unsigned char*)smem);
      const unsigned call = sload(cnt) + 1;
      unsigned char* const dreg = ptr_of<unsigned char>(kl[2], kl[3]);
      unsigned char* const dbase = dreg + (call & 1) * XBUF + lane * 16;
      asm volatile("s_waitcnt vmcnt(6)" : "+v"(r[0][0]), "+v"(r[0][1]), "+v"(r[0][2]), "+v"(r[0][3]), "+v"(r[0][4]) :: "memory");
      const unsigned tref = now32();
      if (v == 0 && lane < 2) store4_sys(dreg - rank * SRC_STRIDE + BEACON_OFF + rank * 4 + lane * 64, call);
      STAMP(1);
#pragma unroll
      for (int n = 0; n < 4; ++n) {
        while (now32() - tref < (unsigned)((v + 14 * n) * PACE_TICKS)) __builtin_amdgcn_s_sleep(1);
        unsigned char* d = dbase + xq_off(q[n]);
#pragma unroll
        for (int i = 0; i < 4; ++i) store16_sys(d + xrow_off(row0[n] + 8 * i), r[n][i]);
        if (big[n]) store16_sys(d + xrow_off(row0[n] + 32), r[n][4]);
        if (n == 0) {
          load(2);
          asm volatile("s_waitcnt vmcnt(5)" : "+v"(r[1][0]), "+v"(r[1][1]), "+v"(r[1][2]), "+v"(r[1][3]), "+v"(r[1][4]) :: "memory");
        } else if (n == 1) {
          load(3);
          asm volatile("s_waitcnt vmcnt(5)" : "+v"(r[2][0]), "+v"(r[2][1]), "+v"(r[2][2]), "+v"(r[2][3]), "+v"(r[2][4]) :: "memory");
        } else if (n == 2) {
          asm volatile("s_waitcnt vmcnt(0)" : "+v"(r[3][0]), "+v"(r[3][1]), "+v"(r[3][2]), "+v"(r[3][3]), "+v"(r[3][4]) :: "memory");
        } else {
          wait_vm<0>();
        }
        if (lane < (n == 0 ? 2 : 1)) store4_sys(dreg + SRC_FLAG + fl[n] + lane * 64, call);
        STAMP(2 + n);
      }
      BARRIER();
      STAMP(6);
      if (threadIdx.x == 0) __hip_atomic_store(cnt, call, __ATOMIC_RELAXED, __HIP_MEMORY_SCOPE_AGENT);
      if (wave == 0) DUMP(2);
    } else {
    }
  }
}

}

extern "C" void kernel_launch(int rank, int nranks, ncclComm_t comm,
                              hipStream_t stream,
                              void* const* d_in, const int* in_sizes, int n_in,
                              void* d_out, int out_size,
                              const int* in_dtypes, int out_dtype,
                              void* const* d_out_all,
                              void* const* d_ws_all, size_t ws_bytes) {
  if (nranks != NR || (unsigned)rank >= (unsigned)NR || ws_bytes < WS_USED || n_in < 4 || out_size != MS * NN) {
    fprintf(stderr, "unexpected problem geometry\n");
    return;
  }
  static bool init = false;
  if (!init) {
    HIP_CHECK(hipFuncSetAttribute((const void*)a2a_gemm_kernel, hipFuncAttributeMaxDynamicSharedMemorySize, LDS_BYTES));
    init = true;
  }
  Params p;
  p.x = (const signed char*)d_in[0];
  p.w = (const signed char*)d_in[1];
  p.sx = (const float*)d_in[2];
  p.sw = (const float*)d_in[3];
  p.out = (float*)d_out;
  p.myws = (unsigned char*)d_ws_all[rank];
  for (int j = 0; j < NR; ++j) {
    const int dst = (rank + 1 + j) % NR;
    p.link[j].rows = p.x + (size_t)dst * MS * KC;
    p.link[j].region = (unsigned char*)d_ws_all[dst] + (size_t)rank * SRC_STRIDE;
  }
  p.rank = rank;
  p.pad[0] = p.pad[1] = p.pad[2] = 0;
  a2a_gemm_kernel<<<256, 576, LDS_BYTES, stream>>>(p);
  HIP_CHECK(hipGetLastError());
}
```

```cpp
#include <hip/hip_runtime.h>
#include <rccl/rccl.h>
#include <cstdio>
#include <type_traits>

#define HIP_CHECK(expr) do { hipError_t _e = (expr); if (_e != hipSuccess) { \
    fprintf(stderr, "HIP error %s at %s:%d\n", hipGetErrorString(_e), __FILE__, __LINE__); \
    return; } } while (0)

namespace {

typedef int v4i __attribute__((ext_vector_type(4)));
typedef unsigned u32x4 __attribute__((ext_vector_type(4)));
#define LDSP __attribute__((address_space(3)))

constexpr int NR = 8;
constexpr int MS = 512;
constexpr int KC = 512;
constexpr int NN = 4096;
constexpr int NMT = 7;
constexpr int NSEND_WG = 32;

#define XP 128
#define XQ 65536
constexpr unsigned XBUF = XP == 128 ? 4 * 512 * 128 : 512 * XP;
constexpr unsigned SRC_FLAG = 2 * XBUF;
constexpr unsigned SRC_STRIDE = SRC_FLAG + 16384;
constexpr unsigned BEACON_OFF = NR * SRC_STRIDE;
constexpr unsigned CNT_OFF = BEACON_OFF + 128;
constexpr unsigned TMO_OFF = CNT_OFF + 256 * 128;
constexpr unsigned DIAG_OFF = TMO_OFF + 128;
constexpr unsigned WS_USED = DIAG_OFF + 256 * 128 * 4;

#define PACE_TICKS 8
#define NTHREADS 512
#define LOADER_PRIO 3
#define SCU 4
#define ROTMASK 0xf0
constexpr unsigned ORD0 = 0x76543210u;
constexpr int PLAN[7][7] = {{0,3,2,5,1,6,4},{1,0,3,6,2,5,4},{2,3,0,4,1,5,6},{3,1,2,6,0,4,5},{0,5,4,1,6,2,3},{1,4,5,0,2,6,3},{2,4,6,3,5,0,1}};
constexpr unsigned plan_word(int mt) {
  unsigned w = 0;
  for (int j = 0; j < 7; ++j) w |= (unsigned)(PLAN[mt][j] + 1) << (4 * j + 4);
  return w;
}
constexpr int due(int mt, int c) {
  int j = 0;
  while (PLAN[mt][j] != c) ++j;
  return (j ? 40 + (j - 1) * (mt < 4 ? 80 : 68) : -48) + (mt < 4 ? 100 : 160);
}
constexpr unsigned send_word(int c) {
  unsigned w = 0;
  for (int mt = 0; mt < 7; ++mt) {
    int k = 0;
    for (int o = 0; o < 7; ++o) k += due(o, c) < due(mt, c) || (due(o, c) == due(mt, c) && o < mt);
    w |= (unsigned)mt << (4 * k);
  }
  return w;
}
template <unsigned (*F)(int)> __device__ __forceinline__ unsigned pick7(int i) {
  constexpr unsigned a = F(0), b = F(1), c = F(2), d = F(3), e = F(4), f = F(5), g = F(6);
  unsigned r = i == 1 ? b : a;
  r = i == 2 ? c : r; r = i == 3 ? d : r; r = i == 4 ? e : r; r = i == 5 ? f : r;
  return i == 6 ? g : r;
}
constexpr unsigned long long DEADLINE_TICKS = 50000000ull;

constexpr int NA = 6, NB = 6;
constexpr unsigned BSLOT = 16384, ASLOT = 10240;
constexpr unsigned LDS_B = 0, LDS_A = NB * BSLOT, LDS_SNAP = LDS_A + NA * ASLOT, LDS_TMO = LDS_SNAP + 512;
constexpr unsigned LDS_JUNK = LDS_TMO + 64, LDS_BYTES = LDS_JUNK + 256;
#define EARLY_AT 11
#define POLLW 8
#define SPIN_SLEEP 1

struct Link { const signed char* rows; unsigned char* region; };
struct Params {
  const signed char* x;
  const signed char* w;
  unsigned char* myws;
  float* out;
  const float* sx;
  const float* sw;
  int rank;
  int pad[3];
  Link link[NR];
};
typedef unsigned u32x8 __attribute__((ext_vector_type(8)));
typedef unsigned u32x2 __attribute__((ext_vector_type(2)));
template <class T> __device__ __forceinline__ T* ptr_of(unsigned lo, unsigned hi) {
  return (T*)(__attribute__((address_space(1))) T*)(((unsigned long long)hi << 32) | lo);
}

template <int N> __device__ __forceinline__ void wait_vm() { asm volatile("s_waitcnt vmcnt(%0)" :: "i"(N) : "memory"); }
template <int N, int I = 0, class F> __device__ __forceinline__ void static_for(F&& f) {
  if constexpr (I < N) { f(std::integral_constant<int, I>{}); static_for<N, I + 1>(f); }
}
#define IC(x) std::integral_constant<int, (x)>{}
#define BARRIER() __builtin_amdgcn_s_barrier()

#define XBITS " sc0 sc1"
#define DMA_ONE(n, bits) ".if %[cnt] > " #n "\n\tbuffer_load_dwordx4 %[v" #n "], %[srd], %[so] offen offset:1024*" #n bits " lds\n\t.endif\n\t"
#define DMA_BURST(bits) \
  asm volatile(DMA_ONE(0, bits) DMA_ONE(1, bits) DMA_ONE(2, bits) DMA_ONE(3, bits) \
               :: [v0] "v"(v0), [v1] "v"(v1), [v2] "v"(v2), [v3] "v"(v3), [srd] "s"(srd), [so] "s"(soff), [cnt] "i"(N) : "memory")
template <int N>
__device__ __forceinline__ void dma(u32x4 srd, unsigned v0, unsigned v1, unsigned v2, unsigned v3, unsigned soff) {
  DMA_BURST("");
}
template <int N>
__device__ __forceinline__ void dma_sys(u32x4 srd, unsigned v0, unsigned v1, unsigned v2, unsigned v3, unsigned soff) {
  DMA_BURST(XBITS);
}
#define SET_M0(base, IMM, TAIL) asm volatile("s_add_u32 m0, %0, %1\n\t" TAIL :: "s"(base), "i"(IMM) : "memory", "scc")
template <int IMM> __device__ __forceinline__ void set_m0(unsigned base) { SET_M0(base, IMM, "s_nop 0"); }
template <int IMM> __device__ __forceinline__ void set_m0_barrier(unsigned base) { SET_M0(base, IMM, "s_barrier"); }
template <int IMM> __device__ __forceinline__ void set_m0_nop_barrier(unsigned base) { SET_M0(base, IMM, "s_nop 3\n\ts_barrier"); }
__device__ __forceinline__ void touch(u32x4 srd, unsigned voff, unsigned soff, unsigned lds) {
  asm volatile("s_mov_b32 m0, %3\n\ts_nop 0\n\tbuffer_load_dword %0, %1, %2 offen lds" :: "v"(voff), "s"(srd), "s"(soff), "s"(lds) : "memory");
}
__device__ __forceinline__ void load16(u32x4& v, u32x4 srd, unsigned voff, unsigned soff) {
  asm volatile("buffer_load_dwordx4 %0, %1, %2, %3 offen" : "=v"(v) : "v"(voff), "s"(srd), "s"(soff) : "memory");
}
__device__ __forceinline__ void store16_sys(void* ptr, u32x4 v) {
  asm volatile("global_store_dwordx4 %0, %1, off sc0 sc1\n\ts_nop 1" :: "v"(ptr), "v"(v) : "memory");
}
__device__ __forceinline__ void store4_sys(void* ptr, unsigned v) {
  asm volatile("global_store_dword %0, %1, off sc0 sc1" :: "v"(ptr), "v"(v) : "memory");
}
__device__ __forceinline__ unsigned sload(const void* p) {
  unsigned v;
  asm volatile("s_load_dword %0, %1, 0x0\n\ts_waitcnt lgkmcnt(0)" : "=s"(v) : "s"(p) : "memory");
  return v;
}
__device__ __forceinline__ unsigned now32() { return (unsigned)__builtin_amdgcn_s_memrealtime(); }
__device__ __forceinline__ unsigned xrow_off(unsigned row) { return row * XP; }
__device__ __forceinline__ unsigned xq_off(unsigned q) { return q * XQ; }
__device__ __forceinline__ int unit_begin(int mt) { return 4 * mt + min(mt, 4); }

constexpr int stage_of(int r) { return r < 8 ? r + 4 : r < 12 ? r + 8 : r + 12; }
constexpr int w_ring(int T) { return T < 5 ? 5 : T < 7 ? 2 * T - 4 : T + 3; }
constexpr int w_cnt(int T) { return T > 16 ? 0 : T == 5 || T == 6 ? 2 : 1; }
constexpr int in_flight(int T, int nx) {
  int n = 0;
  for (int k = T; k >= T - 1; --k) {
    for (int i = w_cnt(k) - 1; i >= 0; --i) { if (stage_of(w_ring(k) + i) <= 2 * T - 1) return n; n += 4; }
    if (k == T && k < 16) n += 2 * nx;
  }
  return n;
}
constexpr int in_flight_loop(int nx) {
  int m = 63;
  for (int T = 7; T < 15; ++T) m = in_flight(T, nx) < m ? in_flight(T, nx) : m;
  return m;
}
constexpr bool w_schedule_ok() {
  for (int T = 5; T < 17; ++T) {
    if (stage_of(w_ring(T) + w_cnt(T) - 1 - NB) / 2 + 2 >= T) return false;
    if (stage_of(w_ring(T)) / 2 + 2 <= T) return false;
    if (w_ring(T) != w_ring(T - 1) + w_cnt(T - 1)) return false;
  }
  return w_ring(16) == 19;
}
static_assert(w_schedule_ok() && NB == 6 && NA == 6, "W schedule");

#define STAMP(i) do { } while (0)
#define STAMPC(i) do { } while (0)
#define DUMP(slot) do { } while (0)

#define WB0(s) "v[" #s "+0:" #s "+3]"
#define WB1(s) "v[" #s "+4:" #s "+7]"
#define XU(s, u) "v[" #s "+8+4*" #u ":" #s "+11+4*" #u "]"
#define MF(acc, w, x) "v_mfma_i32_16x16x64_i8 %[" #acc "], " x ", " w ", %[" #acc "]\n\t"
#define TRL(s, o, p, off, k) "ds_read_b64_tr_b8 v[" #s "+" #o ":" #s "+" #o "+1], %[" #p "] offset:" #off "+%[stepB]*(" #k "&1)\n\t"
#define RX(s, u, p, k) "ds_read_b128 " XU(s, u) ", %[" #p "] offset:2048*" #u "+%[aslot]*(" #k "&1)\n\t"
#define IF5(x) ".if %[nu] == 5\n\t" x ".endif\n\t"
#define IF4(x) ".if %[nu] == 4\n\t" x ".endif\n\t"
#define TICK_STAMP
#define DG_OPERAND
#define DG_PARAM
#define DG_ARG
#define LOOP_STAMP

#define PICK(d, m, i, wrap, step) "s_bitcmp1_b32 %[" #m "], " #i "\n\t", "s_cselect_b32 %[" #d "], %[" #wrap "], 2*%[" #step "]\n\t"
#define ADV(p, d) "v_add_u32 %[" #p "], %[" #d "], %[" #p "]\n\t"
#define BAR(k) ".if (" #k "&1) == 0\n\ts_barrier\n\t.endif\n\t"
#define HALF_A(k, STAMP) \
    MF(c00, WB0(92), XU(92, 0)) \
    MF(c01, WB1(92), XU(92, 0)) \
    "s_waitcnt lgkmcnt(0)\n\t" BAR(k) STAMP \
    TRL(64, 0, pw0, 0, k) TRL(64, 2, pw0, 1024, k) \
    MF(c11, WB1(92), XU(92, 1)) RX(64, 0, pa0, k) \
    MF(c10, WB0(92), XU(92, 1)) TRL(64, 4, pw1, 0, k) TRL(64, 6, pw1, 1024, k) \
    MF(c20, WB0(92), XU(92, 2)) RX(64, 1, pa0, k) \
    MF(c21, WB1(92), XU(92, 2)) RX(64, 2, pa0, k) \
    MF(c31, WB1(92), XU(92, 3)) RX(64, 3, pa0, k) \
    MF(c30, WB0(92), XU(92, 3)) IF5(RX(64, 4, pa0, k)) \
    IF5(MF(c40, WB0(92), XU(92, 4)) MF(c41, WB1(92), XU(92, 4)))
#define HALF_B_(k, E1, E2, E3, E4, E5) \
    "s_waitcnt lgkmcnt(%[nu]-1)\n\t" \
    MF(c00, WB0(64), XU(64, 0)) TRL(92, 0, pw0, 8192, k) TRL(92, 2, pw0, 9216, k) \
    MF(c01, WB1(64), XU(64, 0)) RX(92, 0, pa1, k) \
    "s_waitcnt lgkmcnt(%[nu])\n\t" \
    MF(c11, WB1(64), XU(64, 1)) TRL(92, 4, pw1, 8192, k) TRL(92, 6, pw1, 9216, k) \
    MF(c10, WB0(64), XU(64, 1)) RX(92, 1, pa1, k) E1 \
    MF(c20, WB0(64), XU(64, 2)) RX(92, 2, pa1, k) E2 \
    MF(c21, WB1(64), XU(64, 2)) RX(92, 3, pa1, k) \
    "s_waitcnt lgkmcnt(8)\n\t" \
    MF(c31, WB1(64), XU(64, 3)) IF5(RX(92, 4, pa1, k)) E3 \
    IF5(MF(c30, WB0(64), XU(64, 3))) E4 \
    IF5(MF(c40, WB0(64), XU(64, 4))) E5 \
    IF5(MF(c41, WB1(64), XU(64, 4))) IF4(MF(c30, WB0(64), XU(64, 3))) \
    "s_waitcnt lgkmcnt(%[nu]-1)\n\t"
#define HALF_B(...) HALF_B_(__VA_ARGS__)

#define RW(b, k, o) "v[" #b "+16*" #k "+" #o ":" #b "+16*" #k "+" #o "+3]"
#define REG_A(k, P0, P1) \
    MF(c00, P0, XU(92, 0)) \
    MF(c01, P1, XU(92, 0)) \
    "s_waitcnt lgkmcnt(0)\n\t" BAR(k) \
    RX(64, 0, pa0, k) \
    MF(c11, P1, XU(92, 1)) RX(64, 1, pa0, k) \
    MF(c10, P0, XU(92, 1)) RX(64, 2, pa0, k) \
    MF(c20, P0, XU(92, 2)) RX(64, 3, pa0, k) \
    MF(c21, P1, XU(92, 2)) IF5(RX(64, 4, pa0, k)) \
    MF(c31, P1, XU(92, 3)) \
    MF(c30, P0, XU(92, 3)) \
    IF5(MF(c40, P0, XU(92, 4)) MF(c41, P1, XU(92, 4)))
#define REG_B_(k, C0, C1, E1, E2, E3) \
    "s_waitcnt lgkmcnt(0)\n\t" \
    MF(c00, C0, XU(64, 0)) RX(92, 0, pa1, k) \
    MF(c01, C1, XU(64, 0)) RX(92, 1, pa1, k) \
    MF(c11, C1, XU(64, 1)) RX(92, 2, pa1, k) \
    MF(c10, C0, XU(64, 1)) RX(92, 3, pa1, k) \
    MF(c20, C0, XU(64, 2)) IF5(RX(92, 4, pa1, k)) E1 \
    MF(c21, C1, XU(64, 2)) E2 \
    MF(c31, C1, XU(64, 3)) E3 \
    MF(c30, C0, XU(64, 3)) \
    IF5(MF(c40, C0, XU(64, 4)) MF(c41, C1, XU(64, 4))) \
    "s_waitcnt lgkmcnt(%[nu]-1)\n\t"
#define REG_B(...) REG_B_(__VA_ARGS__)
#define MOV2(d, s) "v_mov_b64 v[" #d ":" #d "+1], v[" #s ":" #s "+1]\n\tv_mov_b64 v[" #d "+2:" #d "+3], v[" #s "+2:" #s "+3]\n\t"
#define REG_CHUNK(b) \
    REG_A(0, WB0(92), WB1(92)) REG_B(0, RW(b, 0, 0), RW(b, 0, 4), PICK(dA, wrapsA, 0, wrapA, aslot), "s_add_u32 %[C], %[C], 1\n\t") \
    REG_A(1, RW(b, 0, 8), RW(b, 0, 12)) REG_B(1, RW(b, 1, 0), RW(b, 1, 4), ADV(pa0, dA), ADV(pa1, dA), "") \
    REG_A(2, RW(b, 1, 8), RW(b, 1, 12)) REG_B(2, RW(b, 2, 0), RW(b, 2, 4), PICK(dA, wrapsA, 1, wrapA, aslot), "s_lshr_b32 %[wrapsA], %[wrapsA], 2\n\t") \
    REG_A(3, RW(b, 2, 8), RW(b, 2, 12)) REG_B(3, RW(b, 3, 0), RW(b, 3, 4), MOV2(92, b+56), MOV2(96, b+60), ADV(pa0, dA) ADV(pa1, dA))
#define TRR(d, p, off) "ds_read_b64_tr_b8 v[" d ":" d "+1], %[" #p "] offset:" off "\n\t"
#define PASS(n) \
    "s_barrier\n\t" \
    TRR("56+16*" #n, pw0, "(" #n "&3)*%[stepB]") TRR("58+16*" #n, pw0, "(" #n "&3)*%[stepB]+1024") \
    TRR("60+16*" #n, pw1, "(" #n "&3)*%[stepB]") TRR("62+16*" #n, pw1, "(" #n "&3)*%[stepB]+1024") \
    TRR("64+16*" #n, pw0, "(" #n "&3)*%[stepB]+8192") TRR("66+16*" #n, pw0, "(" #n "&3)*%[stepB]+9216") \
    TRR("68+16*" #n, pw1, "(" #n "&3)*%[stepB]+8192") TRR("70+16*" #n, pw1, "(" #n "&3)*%[stepB]+9216") \
    "s_waitcnt lgkmcnt(0)\n\t"
#define V8(a) "v" #a "0", "v" #a "1", "v" #a "2", "v" #a "3", "v" #a "4", "v" #a "5", "v" #a "6", "v" #a "7"
#define V10(a) V8(a), "v" #a "8", "v" #a "9"

template <int NU>
__device__ __forceinline__ void k_loop(v4i (&acc)[5][2], unsigned pW0, unsigned pW1, unsigned pA0, unsigned pA1 DG_PARAM) {
  unsigned C = 0, wrapsA = 0x4924u, wrapsB = 0x492u, dA, dB;
  asm volatile(
    "s_branch 2f\n\t"
    "1:\n\t"
    HALF_A(0, LOOP_STAMP)
    "3:\n\t"
    HALF_B(0, PICK(dB, wrapsB, 0, wrapB, stepB), PICK(dA, wrapsA, 0, wrapA, aslot), "")
    HALF_A(1, "") HALF_B(1, ADV(pw0, dB), ADV(pw1, dB), ADV(pa0, dA), ADV(pa1, dA), "s_add_u32 %[C], %[C], 1\n\t")
    HALF_A(2, "") HALF_B(2, PICK(dB, wrapsB, 1, wrapB, stepB), PICK(dA, wrapsA, 1, wrapA, aslot),
                         "s_lshr_b32 %[wrapsA], %[wrapsA], 2\n\ts_lshr_b32 %[wrapsB], %[wrapsB], 2\n\t")
    HALF_A(3, "") HALF_B(3, ADV(pw0, dB), ADV(pw1, dB), ADV(pa0, dA), ADV(pa1, dA), "s_bitcmp0_b32 %[special], %[C]\n\t")
    "s_cbranch_scc1 1b\n\t"
    "s_cmp_eq_u32 %[C], 5\n\t"
    "s_cbranch_scc1 6f\n\t"
    "s_cmp_lg_u32 %[C], 3\n\t"
    "s_cbranch_scc1 4f\n\t"
    REG_CHUNK(120)
    "s_branch 1b\n\t"
    "6:\n\t"
    REG_CHUNK(184)
    "s_branch 1b\n\t"
    "4:\n\t"
    MF(c00, WB0(92), XU(92, 0)) MF(c01, WB1(92), XU(92, 0))
    "s_waitcnt lgkmcnt(0)\n\t"
    MF(c11, WB1(92), XU(92, 1)) MF(c10, WB0(92), XU(92, 1))
    MF(c20, WB0(92), XU(92, 2)) MF(c21, WB1(92), XU(92, 2)) MF(c31, WB1(92), XU(92, 3)) MF(c30, WB0(92), XU(92, 3))
    IF5(MF(c40, WB0(92), XU(92, 4)) MF(c41, WB1(92), XU(92, 4)))
    "s_cmp_eq_u32 %[C], 8\n\t"
    "s_cbranch_scc1 5f\n\t"
    "v_add_u32 %[pw0], 2*%[stepB], %[pw0]\n\tv_add_u32 %[pw1], 2*%[stepB], %[pw1]\n\t"
    PASS(4) PASS(5) PASS(6) PASS(7) PASS(8) PASS(9) PASS(10) PASS(11)
    "v_subrev_u32 %[pw0], 2*%[stepB], %[pw0]\n\tv_subrev_u32 %[pw1], 2*%[stepB], %[pw1]\n\t"
    "s_barrier\n\t"
    "2:\n\t"
    "s_barrier\n\t"
    TICK_STAMP
    TRL(64, 0, pw0, 0, 0) TRL(64, 2, pw0, 1024, 0) RX(64, 0, pa0, 0) TRL(64, 4, pw1, 0, 0) TRL(64, 6, pw1, 1024, 0)
    RX(64, 1, pa0, 0) RX(64, 2, pa0, 0) RX(64, 3, pa0, 0) IF5(RX(64, 4, pa0, 0))
    "s_branch 3b\n\t"
    "5:\n\t"
    "s_nop 7\n\t"
    : [c00] "+v"(acc[0][0]), [c01] "+v"(acc[0][1]), [c10] "+v"(acc[1][0]), [c11] "+v"(acc[1][1]), [c20] "+v"(acc[2][0]),
      [c21] "+v"(acc[2][1]), [c30] "+v"(acc[3][0]), [c31] "+v"(acc[3][1]), [c40] "+v"(acc[4][0]), [c41] "+v"(acc[4][1]),
      [pw0] "+v"(pW0), [pw1] "+v"(pW1), [pa0] "+v"(pA0), [pa1] "+v"(pA1),
      [C] "+&s"(C), [wrapsA] "+&s"(wrapsA), [wrapsB] "+&s"(wrapsB), [dA] "=&s"(dA), [dB] "=&s"(dB) DG_OPERAND
    : [nu] "i"(NU), [stepB] "i"(BSLOT), [aslot] "i"(ASLOT), [wrapA] "s"(-4 * (int)ASLOT), [wrapB] "s"(-4 * (int)BSLOT),
      [special] "s"((1u << 1) | (1u << 3) | (1u << 5) | (1u << 8))
    : "memory", "scc", "s90", "s91", "s92", "s93", "v64", "v65", "v66", "v67", "v68", "v69",
      V10(7), V10(8), V10(9), V10(10), V10(11), V10(12), V10(13), V10(14), V10(15), V10(16), V10(17), V10(18), V10(19),
      V10(20), V10(21), V10(22), V10(23), V8(24));
}

struct Flags {
  u32x4 srd;
  unsigned vF;
  unsigned vSnap, snap;
  unsigned vTmo;
  unsigned call, t0;
  unsigned long long seen;
};
__device__ __forceinline__ unsigned lds_word(unsigned addr) { return *(volatile LDSP unsigned*)(size_t)addr; }
__device__ __forceinline__ void poll(const Flags& f, unsigned long long want) {
  asm volatile("s_andn2_b64 exec, %3, %4\n\ts_cbranch_scc0 9f\n\ts_mov_b32 m0, %2\n\ts_nop 0\n\t"
               "buffer_load_dword %0, %1, 0 offen sc0 sc1 lds\n\t9:\n\ts_mov_b64 exec, -1"
               :: "v"(f.vF), "s"(f.srd), "s"(f.snap), "s"(want), "s"(f.seen) : "memory", "scc");
}
template <int IMM, int BAR>
__device__ __forceinline__ void gate(Flags& f, unsigned base, unsigned bit) {
  unsigned long long ts;
  asm volatile(
    "s_add_u32 m0, %[base], %[imm]\n\t"
    "s_bitcmp1_b64 %[seen], %[bit]\n\t"
    ".if %[bar]\n\ts_barrier\n\t.endif\n\t"
    "s_cbranch_scc0 7f\n\t"
    "6:\n\t"
    ".subsection 1\n\t"
    "7:\n\t"
    "s_mov_b32 m0, %[snap]\n\t"
    "8:\n\t"
    "ds_read_b32 v63, %[vsnap]\n\t"
    "s_lshl_b64 exec, 0x55, %[bit]\n\t"
    "buffer_load_dword %[vf], %[srd], 0 offen sc0 sc1 lds\n\t"
    "s_mov_b64 exec, -1\n\t"
    "s_memrealtime %[ts]\n\t"
    "s_waitcnt lgkmcnt(0)\n\t"
    "v_cmp_le_u32 %[seen], %[call], v63\n\t"
    "s_bitcmp1_b64 %[seen], %[bit]\n\t"
    "s_cbranch_scc1 9f\n\t"
    "s_waitcnt vmcnt(1)\n\t"
    "s_lshl_b64 %[ts], %[ts], 32\n\ts_lshr_b64 vcc, %[ts], 32\n\t"
    "s_sub_u32 vcc_lo, vcc_lo, %[t0]\n\t"
    "s_cmp_lt_u32 vcc_lo, %[dl]\n\t"
    "s_cbranch_scc1 8b\n\t"
    "v_mov_b32 v63, 1\n\t"
    "ds_write_b32 %[vtmo], v63\n\t"
    "s_waitcnt lgkmcnt(0)\n\t"
    "9:\n\t"
    "s_add_u32 m0, %[base], %[imm]\n\t"
    "s_branch 6b\n\t"
    ".subsection 0"
    : [seen] "+&s"(f.seen), [ts] "=&s"(ts)
    : [base] "s"(base), [imm] "i"(IMM), [bar] "i"(BAR), [bit] "s"(bit), [snap] "s"(f.snap), [vsnap] "v"(f.vSnap),
      [vf] "v"(f.vF), [srd] "s"(f.srd), [call] "s"(f.call), [t0] "s"(f.t0), [dl] "i"((unsigned)DEADLINE_TICKS), [vtmo] "v"(f.vTmo)
    : "memory", "scc", "vcc", "v63");
}

template <int SLOT>
__device__ __forceinline__ void w_burst(u32x4 srdW, const unsigned (&vW)[4], unsigned stage_off, unsigned lB) {
  unsigned so;
  asm volatile("s_add_u32 m0, %1, %2\n\ts_mov_b32 %0, %3" : "=s"(so) : "s"(lB), "i"(SLOT * BSLOT), "s"(stage_off) : "memory", "scc");
  dma<4>(srdW, vW[0], vW[1], vW[2], vW[3], so);
}

__device__ __forceinline__ unsigned lane_of(unsigned tab, int i) {
  unsigned r = __builtin_amdgcn_readlane(tab, i);
  asm volatile("" : "+s"(r));
  return r;
}
constexpr unsigned long long WINDOW = 0x5555555555555555ull >> (64 - 2 * POLLW);

template <int NX>
__device__ __forceinline__ void load_loop(u32x4 srdW, const unsigned (&vW)[4], const unsigned (&vS)[3], unsigned tabW, unsigned tabX,
                                          unsigned tabL, unsigned lA, unsigned h, unsigned long long window, Flags& f DG_PARAM) {
  unsigned sx0 = lane_of(tabX, 8), sx1 = lane_of(tabX, 9), bit = 16 + h, la = lA + 2 * ASLOT;
  asm volatile("s_nop 1");
  auto tick = [&](auto Tc, int it) {
    constexpr int T = Tc.value;
    gate<0, 1>(f, la, bit);
    dma_sys<NX>(f.srd, vS[0], vS[1], vS[2], vS[2], sx0);
    if constexpr (T != 4) poll(f, window << bit);
    gate<ASLOT, 0>(f, la, bit + 2);
    unsigned wv;
    asm volatile("ds_read_b32 %0, %1" : "=v"(wv) : "v"(f.vSnap) : "memory");
    dma_sys<NX>(f.srd, vS[0], vS[1], vS[2], vS[2], sx1);
    __builtin_amdgcn_sched_barrier(0);
    bit += 4;
    sx0 = lane_of(tabX, 2 * (T + it) + 2); sx1 = lane_of(tabX, 2 * (T + it) + 3);
    const unsigned sw = lane_of(tabW, w_ring(T) + it), sl = lane_of(tabL, w_ring(T) + it);
    la = la == lA + 4 * ASLOT ? lA : la + 2 * ASLOT;
    asm volatile("" : "+s"(la), "+s"(bit));
    asm volatile("s_waitcnt lgkmcnt(0)\n\tv_cmp_le_u32 %0, %1, %2" : "=s"(f.seen) : "s"(f.call), "v"(wv) : "memory");
    __builtin_amdgcn_sched_barrier(0);
    w_burst<0>(srdW, vW, sw, sl);
    if constexpr (w_cnt(T) == 2) w_burst<0>(srdW, vW, lane_of(tabW, w_ring(T) + 1), lane_of(tabL, w_ring(T) + 1));
    if constexpr (T == 4) wait_vm<2 * NX + 4>();
    else if constexpr (T < 7) wait_vm<in_flight(T, NX)>();
    else wait_vm<in_flight_loop(NX)>();
    __builtin_amdgcn_sched_barrier(0);
  };
  tick(IC(4), 0); tick(IC(5), 0); tick(IC(6), 0);
#pragma unroll 1
  for (int it = 0; it < 9; ++it) tick(IC(7), it);
  wait_vm<in_flight(15, NX)>();
  STAMP(8);
  const unsigned sw = lane_of(tabW, 19), sl = lane_of(tabL, 19);
  BARRIER();
  w_burst<0>(srdW, vW, sw, sl);
  wait_vm<0>();
  BARRIER();
}

__global__ __attribute__((flatten)) __launch_bounds__(NTHREADS, 2) void a2a_gemm_kernel(Params p) {
  extern __shared__ __attribute__((aligned(16))) unsigned char smem[];
  const int b = blockIdx.x;
  const int wave = __builtin_amdgcn_readfirstlane(threadIdx.x >> 6);
  const int lane = threadIdx.x & 63;
  STAMP(0);
  const int b7 = (b & 31) * 37 >> 8, dj = (b & 31) - 7 * b7;
  asm volatile("s_load_dword s100, %0, 0x0\n\ts_load_dword s101, %0, %1"
               :: "s"(__builtin_amdgcn_kernarg_segment_ptr()), "s"(64 + 16 * dj) : "memory", "s100", "s101");
#define ARGS() u32x8 ka, kb; u32x4 kl; \
  asm volatile("s_load_dwordx8 %0, %3, 0x0\n\ts_load_dwordx8 %1, %3, 0x20\n\ts_load_dwordx4 %2, %3, %4\n\ts_waitcnt lgkmcnt(0)" \
               : "=&s"(ka), "=&s"(kb), "=&s"(kl) : "s"(__builtin_amdgcn_kernarg_segment_ptr()), "s"(64 + 16 * dj) : "memory"); \
  const int rank = kb[4]; unsigned char* const myws = ptr_of<unsigned char>(ka[4], ka[5]); \
  unsigned* const cnt = (unsigned*)(myws + CNT_OFF + b * 128); (void)rank; (void)cnt
#define SRD(lo, hi, bytes) u32x4{lo, (hi) & 0xffffu, bytes, 0x00020000u}
  if (wave == 3 && b >= NSEND_WG && b < NSEND_WG + 8 * 4) {
    unsigned long long pc; int len;
    asm volatile("s_getpc_b64 %0\n.Lcode_pf:\n\ts_mov_b32 %1, .Lcode_pf_end-.Lcode_pf\n\t.subsection 2\n.Lcode_pf_end:\n\t.subsection 0" : "=s"(pc), "=s"(len));
    touch(SRD((unsigned)pc, (unsigned)(pc >> 32), (unsigned)max(len, 0)), lane * 128, ((b - NSEND_WG) >> 3) * 8192, (unsigned)(size_t)(LDSP unsigned char*)smem + LDS_JUNK);
  }

  if (b >= NSEND_WG) {
  const int g = (b - NSEND_WG) >> 3, xcd = b & 7;
  const int ntl = g * 37 >> 8, mt = g - NMT * ntl, nt = xcd * 4 + ntl;
  const int ub = unit_begin(mt), nu = mt < 4 ? 5 : 4;
  volatile LDSP unsigned* const ltmo = (volatile LDSP unsigned*)(smem + LDS_TMO);
  const unsigned lds0 = (unsigned)(size_t)(LDSP unsigned char*)smem;
  const unsigned plan = pick7<plan_word>(mt);
  auto src_of = [&](int rank, int js) { return (int)(rank + (plan >> (4 * js))) & 7; };
  auto q_of = [&](int S) { return (S + (S < 4 ? mt + ntl : S < 8 ? (nu == 5 ? 0 : 2) : ((ROTMASK >> (S >> 2)) & 1) ? ntl : 0)) & 3; };

  if (wave < 4) {
  const int fg = lane >> 4, fq = (lane & 15) >> 1, fp = lane & 1;
  const unsigned pW0 = lds0 + LDS_B + 2 * BSLOT + (16 * fg + fq) * 128 + (((2 * wave) ^ (((fq >> 1) & 3) | ((fg & 1) << 2))) << 4) + 8 * fp;
  const unsigned pA0 = lds0 + LDS_A + (lane & 15) * 128 + ((fg ^ fq) << 4);
  unsigned vO[8];
#pragma unroll
  for (int i = 0; i < 8; ++i) {
    vO[i] = ((ub * 16 + (lane >> 5) * 8 + i) * NN + nt * 128 + wave * 32 + (lane & 31)) * 4;
    asm volatile("" : "+v"(vO[i]));
  }
  ARGS();
  const u32x4 srdO = SRD(ka[6], ka[7], (unsigned)MS * NN * 4);
  {
    const u32x4 srdW = SRD(ka[2], ka[3], 4096u * NN);
    const int slow = (3 - (int)(ka[2] >> 7) - xcd * 4) & 7;
    if (wave == 0) {
      touch(srdO, (ub * 16 + (lane >> 5) * (nu * 16 - 1)) * NN * 4 + nt * 512, 0, lds0 + LDS_JUNK);
    } else if (wave == 1) {
      if (slow < 4 && slow != ntl) {
        const int part = (ntl - slow - 1) & 3;
        const unsigned col = (xcd * 4 + slow) * 128;
        touch(srdW, min(part * 43 + min(lane, 42), 127) * NN, (unsigned)(src_of(rank, 1) * KC + q_of(5) * 128) * NN + col, lds0 + LDS_JUNK);
        if (nu == 4) {
          touch(srdW, lane * NN, (unsigned)(src_of(rank, 3) * KC + 128 + part * 128) * NN + col, lds0 + LDS_JUNK);
          touch(srdW, lane * NN, (unsigned)(src_of(rank, 3) * KC + 192 + part * 128) * NN + col, lds0 + LDS_JUNK);
        }
      }
    } else if (nu == 4) {
      const unsigned row = min(((mt - 4) * 2 + wave - 2) * 43 + min(lane, 42), 255);
      touch(srdW, row * NN, (unsigned)(((rank + 4) & 7) * KC + 256) * NN + nt * 128, lds0 + LDS_JUNK);
    }
  }
  if (wave < 2) ((volatile LDSP unsigned*)(smem + LDS_SNAP))[wave * 64 + lane] = 0;
  else if (wave == 2 && lane < 2) ltmo[lane] = lane ? ORD0 : 0;
  float sx, sw;
  asm volatile("s_load_dword %0, %2, 0x0\n\ts_load_dword %1, %3, 0x0\n\ts_waitcnt lgkmcnt(0)"
               : "=&s"(sx), "=&s"(sw) : "s"(ptr_of<float>(kb[0], kb[1])), "s"(ptr_of<float>(kb[2], kb[3])) : "memory");
  float scale = sx * sw;
  asm volatile("" : "+v"(scale));
  v4i acc[5][2];
#pragma unroll
  for (int u = 0; u < 5; ++u) { acc[u][0] = v4i{0, 0, 0, 0}; acc[u][1] = v4i{0, 0, 0, 0}; }
  if (nu == 5) k_loop<5>(acc, pW0, pW0 ^ 16, pA0, pA0 ^ 64 DG_ARG);
  else k_loop<4>(acc, pW0, pW0 ^ 16, pA0, pA0 ^ 64 DG_ARG);
  const float sc = *ltmo ? __builtin_nanf("") : scale;
  STAMP(11);
  STAMPC(17);

#pragma unroll
  for (int u = 0; u < 5; ++u) {
    if (u < nu) {
#pragma unroll
      for (int r = 0; r < 4; ++r) {
        const float y0 = __builtin_elementwise_maximum((float)acc[u][0][r] * sc, 0.f);
        const float y1 = __builtin_elementwise_maximum((float)acc[u][1][r] * sc, 0.f);
        const auto y = __builtin_amdgcn_permlane16_swap(__builtin_bit_cast(unsigned, y0), __builtin_bit_cast(unsigned, y1), false, false);
#define OUT_BITS " sc1"
        asm volatile("buffer_store_dword %0, %1, %2, %3 offen" OUT_BITS :: "v"(y[0]), "v"(vO[r]), "s"(srdO), "s"(u * 16 * NN * 4) : "memory");
        asm volatile("buffer_store_dword %0, %1, %2, %3 offen" OUT_BITS :: "v"(y[1]), "v"(vO[r + 4]), "s"(srdO), "s"(u * 16 * NN * 4) : "memory");
      }
    }
  }
  return;
  }

  {
    const int lw = wave - 4;
    const unsigned swz = (lane & 7) ^ ((lane >> 4) & 3);
    unsigned vW[4];
#pragma unroll
    for (int i = 0; i < 4; ++i) vW[i] = (lw * 32 + (lane >> 3)) * NN + nt * 128 + ((swz ^ ((i & 2) << 1)) << 4) + i * (8 * NN - 1024);
    unsigned lB = lds0 + LDS_B + lw * 4096, q0 = q_of(0) << 19;
    asm volatile("" : "+v"(vW[0]), "+v"(vW[1]), "+v"(vW[2]), "+v"(vW[3]));
    asm volatile("" : "+s"(lB), "+s"(q0));
    ARGS();
    const u32x4 srdW = SRD(ka[2], ka[3], 4096u * NN);
    STAMP(10);
    w_burst<2>(srdW, vW, ((unsigned)rank << 21) + q0, lB);
    __builtin_amdgcn_sched_barrier(0);
    __builtin_amdgcn_s_setprio(LOADER_PRIO);
    const bool odd = lw & 1, a3 = nu == 5 && !odd;
    const int a0 = lw * 2 + (nu == 5 ? (lw + 1) >> 1 : 0);
    const int arow = ub * 16 + a0 * 8;
    unsigned vX[3], vS[3];
#pragma unroll
    for (int i = 0; i < 3; ++i) {
      vX[i] = (arow + (lane >> 3)) * KC + ((swz ^ (((a0 + i) & 1) << 2)) << 4) + i * (8 * KC - 1024);
      vS[i] = xrow_off(arow + 8 * i + (lane >> 3)) + ((swz ^ (((a0 + i) & 1) << 2)) << 4) - i * 1024;
    }
    const unsigned lA = lds0 + LDS_A + a0 * 1024;
    const u32x4 srdX = SRD(ka[0], ka[1], 4096u * KC);
    const u32x4 srdS = SRD(ka[4], ka[5], CNT_OFF);
    auto w_off = [&](int S) { return (unsigned)(src_of(rank, S >> 2) * 4 + q_of(S)) << 19; };
    auto W_load = [&](auto Sc, auto slot) { w_burst<slot.value>(srdW, vW, w_off(Sc.value), lB); };
    auto X_local = [&](auto Sc) {
      constexpr int S = Sc.value;
      const unsigned so = (unsigned)(rank * MS) * KC + q_of(S) * 128;
      set_m0<S * ASLOT>(lA);
      if (a3) dma<3>(srdX, vX[0], vX[1], vX[2], vX[2], so); else dma<2>(srdX, vX[0], vX[1], vX[2], vX[2], so);
    };
    X_local(IC(0)); W_load(IC(4), IC(0)); W_load(IC(1), IC(3)); X_local(IC(1));
    asm volatile("s_load_dword s100, %0, 0x0" :: "s"(cnt) : "memory", "s100");
    W_load(IC(2), IC(4)); X_local(IC(2));
    wait_vm<6>(); BARRIER();
    W_load(IC(3), IC(5)); X_local(IC(3)); W_load(IC(5), IC(1));
    const unsigned call = sload(cnt) + 1;
    const int Sl = lane & 31;
    unsigned tabW, tabX;
    Flags f;
    f.srd = srdS; f.call = call; f.t0 = now32(); f.seen = 0;
    f.snap = lds0 + LDS_SNAP; f.vSnap = f.snap + lane * 4; f.vTmo = lds0 + LDS_TMO;
    auto tables = [&](unsigned ord) {
      const int srcf = src_of(rank, lane < 8 ? lane : (ord >> ((lane >> 3) * 4)) & 7);
      f.vF = lane < 8 ? BEACON_OFF + srcf * 4 : srcf * SRC_STRIDE + SRC_FLAG + (mt * 4 + q_of(lane >> 1)) * 128 + (lane & 1) * 4;
      const int src = src_of(rank, (ord >> (Sl & 28)) & 7);
      const int Sr = min(Sl < 8 ? Sl + 4 : Sl < 12 ? Sl + 8 : Sl + 12, 31);
      tabW = (unsigned)(src_of(rank, (ord >> (Sr & 28)) & 7) * 4 + q_of(Sr)) << 19;
      tabX = src * SRC_STRIDE + (call & 1) * XBUF + xq_off(q_of(Sl));
    };
    tables(ORD0);
    const unsigned tabL = lB + (Sl % NB) * BSLOT;
    constexpr unsigned long long FIRST = 0xfffffeull;
    const unsigned h = lw >> 1;
    auto X_first = [&]() {
      static_for<4>([&](auto kc) {
        constexpr int S = 4 + kc.value;
        const unsigned sx = lane_of(tabX, S);
        asm volatile("s_nop 1");
        gate<(S % NA) * ASLOT, 0>(f, lA, 2 * S + h);
        if (a3) dma_sys<3>(srdS, vS[0], vS[1], vS[2], vS[2], sx); else dma_sys<2>(srdS, vS[0], vS[1], vS[2], vS[2], sx);
      });
    };
    auto look = [&]() {
      f.seen = __ballot(lds_word(f.vSnap) >= call);
    };
    wait_vm<4>(); BARRIER();
    if (threadIdx.x == 256) __hip_atomic_store(cnt, call, __ATOMIC_RELAXED, __HIP_MEMORY_SCOPE_AGENT);
    if (lw == 3 && !(lane & 6)) touch(srdS, (lane & 1) ? f.vF : tabX, 0, lds0 + LDS_JUNK);
    W_load(IC(12), IC(2)); W_load(IC(13), IC(3));
    static_assert(EARLY_AT >= 10 && EARLY_AT <= 11, "");
    bool early = false;
    static_for<8>([&](auto kc) {
      constexpr int n = 7 + kc.value;
      unsigned so = w_off(n < 8 ? n + 8 : n < 12 ? n + 12 : n - 6);
      asm volatile("" : "+s"(so));
      if constexpr (n > EARLY_AT && n <= EARLY_AT + 3) { if (early) wait_vm<16>(); else wait_vm<8>(); }
      else wait_vm<n == 7 ? 4 : 8>();
      if constexpr (n == 7) { BARRIER(); W_load(IC(14), IC(4)); set_m0<(2 + n % 4) * BSLOT>(lB); }
      else set_m0_barrier<(2 + n % 4) * BSLOT>(lB);
      dma<4>(srdW, vW[0], vW[1], vW[2], vW[3], so);
      if constexpr (n == EARLY_AT - 3 || n == 13) { if (lw == 1) poll(f, FIRST); }
      if constexpr (n == EARLY_AT) {
        look();
        if (((unsigned)f.seen & 0xfffe) == 0xfffe) { X_first(); early = true; }
      }
    });
    STAMP(9);
    if (lw == 1 && !early) {
      for (;;) {
        poll(f, FIRST);
        const unsigned m = (unsigned)__ballot(lds_word(f.vSnap) >= call);
        const unsigned first = m & (m >> 1) & ((m & 2) ? 0x100u : 0x10100u);
        if (first) {
          const int j0 = __builtin_ctz(first) >> 3;
          const unsigned good = ((m & 0xfe) | (1u << j0)) & (~0u << j0);
          if (good != 0xfe) {
            unsigned ord = 0x00503000u, free = 0x76421u;
            for (int pass = 0; pass < 2; ++pass)
              for (int j = 1; j < 8; ++j)
                if (j != 3 && j != 5 && (((good >> j) & 1) ^ pass) == 1) { ord |= (unsigned)j << (4 * (free & 7)); free >>= 4; }
            ltmo[1] = ord;
          }
          break;
        }
        if (now32() - f.t0 >= (unsigned)DEADLINE_TICKS) { ltmo[0] = 1; break; }
        __builtin_amdgcn_s_sleep(SPIN_SLEEP);
        wait_vm<8>();
      }
      asm volatile("s_waitcnt lgkmcnt(0)" ::: "memory");
    }
    BARRIER();
    const unsigned ord = __builtin_amdgcn_readfirstlane(ltmo[1]);
    if (ord != ORD0) {
      tables(ord);
      f.snap += 256; f.vSnap += 256;
      early = false;
#pragma unroll 1
      for (int r = 0; r < NB - 1; ++r)
        if ((ord ^ ORD0) >> ((r + 4) & 28) & 15) w_burst<0>(srdW, vW, __builtin_amdgcn_readlane(tabW, r), lB + r * BSLOT);
    }
    STAMP(1);
    look();
    const unsigned long long window = odd ? WINDOW << 4 : 0;
    poll(f, (window | window >> 4) << (16 + h));
    if (early) {
      wait_vm<12>();
    } else {
      X_first();
      if (a3) wait_vm<6>(); else wait_vm<4>();
      look();
    }
    if (a3) load_loop<3>(srdW, vW, vS, tabW, tabX, tabL, lA, h, window, f DG_ARG);
    else load_loop<2>(srdW, vW, vS, tabW, tabX, tabL, lA, h, window, f DG_ARG);
    if (lw == 1) DUMP(1);
    return;
  }

  }

  {
    const int v = b7 + SCU * wave;
    if (b < 7 * SCU && v < 14) {
      const unsigned serve = pick7<send_word>(6 - dj);
      unsigned row0[4], q[4], fl[4], so[4]; bool big[4];
#pragma unroll
      for (int n = 0; n < 4; ++n) {
        const int pi = v + 14 * n, h = pi & 1;
        const int mt = (serve >> (4 * (pi >> 3))) & 7;
        q[n] = (pi >> 1) & 3;
        big[n] = mt < 4;
        row0[n] = unit_begin(mt) * 16 + h * (big[n] ? 40 : 32);
        fl[n] = (mt * 4 + q[n]) * 128 + h * 4;
        so[n] = row0[n] * KC + q[n] * 128;
      }
      unsigned vL[5];
#pragma unroll
      for (int i = 0; i < 5; ++i) vL[i] = ((lane >> 3) + 8 * i) * KC + (lane & 7) * 16;
      asm volatile("" : "+v"(vL[0]), "+v"(vL[1]), "+v"(vL[2]), "+v"(vL[3]), "+v"(vL[4]));
      asm volatile("" : "+s"(so[0]), "+s"(so[1]));
      ARGS();
      const u32x4 srdR = SRD(kl[0], kl[1], (unsigned)MS * KC);
      u32x4 r[4][5];
      auto load = [&](int n) {
#pragma unroll
        for (int i = 0; i < 5; ++i) load16(r[n][i], srdR, vL[i], so[n]);
      };
      STAMP(7);
      load(0);
      load(1);
      BARRIER();
      touch(SRD(kl[2], kl[3], SRC_STRIDE), (lane & 3) * (SRC_STRIDE / 3 - 64 & ~63u), 0, (unsigned)(size_t)(LDSP unsigned char*)smem);
      const unsigned call = sload(cnt) + 1;
      unsigned char* const dreg = ptr_of<unsigned char>(kl[2], kl[3]);
      unsigned char* const dbase = dreg + (call & 1) * XBUF + xrow_off(lane >> 3) + (lane & 7) * 16;
      asm volatile("s_waitcnt vmcnt(6)" : "+v"(r[0][0]), "+v"(r[0][1]), "+v"(r[0][2]), "+v"(r[0][3]), "+v"(r[0][4]) :: "memory");
      const unsigned tref = now32();
      if (v == 0 && lane == 0) store4_sys(dreg - rank * SRC_STRIDE + BEACON_OFF + rank * 4, call);
      STAMP(1);
#pragma unroll
      for (int n = 0; n < 4; ++n) {
        while (now32() - tref < (unsigned)((v + 14 * n) * PACE_TICKS)) __builtin_amdgcn_s_sleep(1);
        unsigned char* d = dbase + xq_off(q[n]);
#pragma unroll
        for (int i = 0; i < 4; ++i) store16_sys(d + xrow_off(row0[n] + 8 * i), r[n][i]);
        if (big[n]) store16_sys(d + xrow_off(row0[n] + 32), r[n][4]);
        if (n == 0) {
          load(2);
          asm volatile("s_waitcnt vmcnt(5)" : "+v"(r[1][0]), "+v"(r[1][1]), "+v"(r[1][2]), "+v"(r[1][3]), "+v"(r[1][4]) :: "memory");
        } else if (n == 1) {
          load(3);
          asm volatile("s_waitcnt vmcnt(5)" : "+v"(r[2][0]), "+v"(r[2][1]), "+v"(r[2][2]), "+v"(r[2][3]), "+v"(r[2][4]) :: "memory");
        } else if (n == 2) {
          asm volatile("s_waitcnt vmcnt(0)" : "+v"(r[3][0]), "+v"(r[3][1]), "+v"(r[3][2]), "+v"(r[3][3]), "+v"(r[3][4]) :: "memory");
        } else {
          wait_vm<0>();
        }
        if (lane == 0) store4_sys(dreg + SRC_FLAG + fl[n], call);
        STAMP(2 + n);
      }
      BARRIER();
      STAMP(6);
      if (threadIdx.x == 0) __hip_atomic_store(cnt, call, __ATOMIC_RELAXED, __HIP_MEMORY_SCOPE_AGENT);
      if (wave == 0) DUMP(2);
    } else {
    }
  }
}

}

extern "C" void kernel_launch(int rank, int nranks, ncclComm_t comm,
                              hipStream_t stream,
                              void* const* d_in, const int* in_sizes, int n_in,
                              void* d_out, int out_size,
                              const int* in_dtypes, int out_dtype,
                              void* const* d_out_all,
                              void* const* d_ws_all, size_t ws_bytes) {
  if (nranks != NR || (unsigned)rank >= (unsigned)NR || ws_bytes < WS_USED || n_in < 4 || out_size != MS * NN) {
    fprintf(stderr, "unexpected problem geometry\n");
    return;
  }
  static bool init = false;
  if (!init) {
    HIP_CHECK(hipFuncSetAttribute((const void*)a2a_gemm_kernel, hipFuncAttributeMaxDynamicSharedMemorySize, LDS_BYTES));
    init = true;
  }
  Params p;
  p.x = (const signed char*)d_in[0];
  p.w = (const signed char*)d_in[1];
  p.sx = (const float*)d_in[2];
  p.sw = (const float*)d_in[3];
  p.out = (float*)d_out;
  p.myws = (unsigned char*)d_ws_all[rank];
  for (int j = 0; j < NR; ++j) {
    const int dst = (rank + 1 + j) % NR;
    p.link[j].rows = p.x + (size_t)dst * MS * KC;
    p.link[j].region = (unsigned char*)d_ws_all[dst] + (size_t)rank * SRC_STRIDE;
  }
  p.rank = rank;
  p.pad[0] = p.pad[1] = p.pad[2] = 0;
  a2a_gemm_kernel<<<256, NTHREADS, LDS_BYTES, stream>>>(p);
  HIP_CHECK(hipGetLastError());
}
```

```cpp
#include <hip/hip_runtime.h>
#include <rccl/rccl.h>
#include <cstdio>
#include <type_traits>

#define HIP_CHECK(expr) do { hipError_t _e = (expr); if (_e != hipSuccess) { \
    fprintf(stderr, "HIP error %s at %s:%d\n", hipGetErrorString(_e), __FILE__, __LINE__); \
    return; } } while (0)

namespace {

typedef unsigned u32x4 __attribute__((ext_vector_type(4)));
#define LDSP __attribute__((address_space(3)))

constexpr int NR = 8;
constexpr int MS = 512;
constexpr int KC = 512;
constexpr int NN = 4096;
constexpr int NMT = 7;
constexpr int NSEND_WG = 32;

#define XP 128
#define XQ 65536
constexpr unsigned XBUF = XP == 128 ? 4 * 512 * 128 : 512 * XP;
constexpr unsigned SRC_FLAG = 2 * XBUF;
constexpr unsigned SRC_STRIDE = SRC_FLAG + 16384;
constexpr unsigned CNT_OFF = NR * SRC_STRIDE;
constexpr unsigned WS_USED = CNT_OFF + 256 * 128;

#define PACE 32
#define NTHREADS 512
#define PW2 8
#define FIRSTW 4
#define LEADP 2
#define CAP 100u
#define LOADER_PRIO 3
#define ROTMASK 0xf0
constexpr int PLAN[7][7] = {{0,6,3,4,1,2,5},{1,6,0,3,2,4,5},{2,5,1,6,3,0,4},{3,4,2,1,0,6,5},{0,3,4,2,5,6,1},{1,4,5,6,0,3,2},{2,5,6,4,3,0,1}};
constexpr unsigned plan_word(int mt) {
  unsigned w = 0;
  for (int j = 0; j < 7; ++j) w |= (unsigned)(PLAN[mt][j] + 1) << (4 * j + 4);
  return w;
}
constexpr int due(int mt, int c) {
  int j = 0;
  while (PLAN[mt][j] != c) ++j;
  return (j ? 60 + (j - 1) * (mt < 4 ? 85 : 75) : -48) + (mt < 4 ? 0 : 70);
}
constexpr unsigned send_word(int c) {
  unsigned w = 0;
  for (int mt = 0; mt < 7; ++mt) {
    int k = 0;
    for (int o = 0; o < 7; ++o) k += due(o, c) < due(mt, c) || (due(o, c) == due(mt, c) && o < mt);
    w |= (unsigned)mt << (4 * k);
  }
  return w;
}
template <unsigned (*F)(int)> __device__ __forceinline__ unsigned pick7(int i) {
  constexpr unsigned a = F(0), b = F(1), c = F(2), d = F(3), e = F(4), f = F(5), g = F(6);
  unsigned r = i == 1 ? b : a;
  r = i == 2 ? c : r; r = i == 3 ? d : r; r = i == 4 ? e : r; r = i == 5 ? f : r;
  return i == 6 ? g : r;
}
constexpr unsigned long long DEADLINE_TICKS = 50000000ull;

constexpr int NA = 6, NB = 6;
constexpr unsigned BSLOT = 16384, ASLOT = 10240;
constexpr unsigned LDS_B = 0, LDS_A = NB * BSLOT, LDS_SNAP = LDS_A + NA * ASLOT, LDS_TMO = LDS_SNAP + 256;
constexpr unsigned LDS_JUNK = LDS_TMO + 64, LDS_BYTES = LDS_JUNK + 256;
#define POLLW 8

struct Link { const signed char* rows; unsigned char* region; };
struct Params {
  const signed char* x;
  const signed char* w;
  unsigned char* myws;
  float* out;
  const float* sx;
  const float* sw;
  int rank;
  int pad[3];
  Link link[NR];
};
typedef unsigned u32x8 __attribute__((ext_vector_type(8)));
template <class T> __device__ __forceinline__ T* ptr_of(unsigned lo, unsigned hi) {
  return (T*)(__attribute__((address_space(1))) T*)(((unsigned long long)hi << 32) | lo);
}

template <int N> __device__ __forceinline__ void wait_vm() { asm volatile("s_waitcnt vmcnt(%0)" :: "i"(N) : "memory"); }
template <int N, int I = 0, class F> __device__ __forceinline__ void static_for(F&& f) {
  if constexpr (I < N) { f(std::integral_constant<int, I>{}); static_for<N, I + 1>(f); }
}
#define IC(x) std::integral_constant<int, (x)>{}
#define BARRIER() __builtin_amdgcn_s_barrier()

#define XBITS " sc0 sc1"
#define DMA_ONE(n, bits) ".if %[cnt] > " #n "\n\tbuffer_load_dwordx4 %[v" #n "], %[srd], %[so] offen offset:1024*" #n bits " lds\n\t.endif\n\t"
#define DMA_BURST(bits) \
  asm volatile(DMA_ONE(0, bits) DMA_ONE(1, bits) DMA_ONE(2, bits) DMA_ONE(3, bits) \
               :: [v0] "v"(v0), [v1] "v"(v1), [v2] "v"(v2), [v3] "v"(v3), [srd] "s"(srd), [so] "s"(soff), [cnt] "i"(N) : "memory")
template <int N>
__device__ __forceinline__ void dma(u32x4 srd, unsigned v0, unsigned v1, unsigned v2, unsigned v3, unsigned soff) {
  DMA_BURST("");
}
template <int N>
__device__ __forceinline__ void dma_sys(u32x4 srd, unsigned v0, unsigned v1, unsigned v2, unsigned v3, unsigned soff) {
  DMA_BURST(XBITS);
}
#define SET_M0(base, IMM, TAIL) asm volatile("s_add_u32 m0, %0, %1\n\t" TAIL :: "s"(base), "i"(IMM) : "memory", "scc")
template <int IMM> __device__ __forceinline__ void set_m0(unsigned base) { SET_M0(base, IMM, "s_nop 0"); }
template <int IMM> __device__ __forceinline__ void set_m0_barrier(unsigned base) { SET_M0(base, IMM, "s_barrier"); }
__device__ __forceinline__ void touch(u32x4 srd, unsigned voff, unsigned soff, unsigned lds) {
  asm volatile("s_mov_b32 m0, %3\n\ts_nop 0\n\tbuffer_load_dword %0, %1, %2 offen lds" :: "v"(voff), "s"(srd), "s"(soff), "s"(lds) : "memory");
}
__device__ __forceinline__ void load16(u32x4& v, u32x4 srd, unsigned voff, unsigned soff) {
  asm volatile("buffer_load_dwordx4 %0, %1, %2, %3 offen" : "=v"(v) : "v"(voff), "s"(srd), "s"(soff) : "memory");
}
__device__ __forceinline__ void load80(u32x4 (&r)[5], u32x4 srd, const unsigned (&v)[5], unsigned soff) {
  asm volatile("buffer_load_dwordx4 %0, %5, %10, %11 offen\n\tbuffer_load_dwordx4 %1, %6, %10, %11 offen\n\t"
               "buffer_load_dwordx4 %2, %7, %10, %11 offen\n\tbuffer_load_dwordx4 %3, %8, %10, %11 offen\n\t"
               "buffer_load_dwordx4 %4, %9, %10, %11 offen\n\ts_waitcnt vmcnt(0)"
               : "=&v"(r[0]), "=&v"(r[1]), "=&v"(r[2]), "=&v"(r[3]), "=&v"(r[4])
               : "v"(v[0]), "v"(v[1]), "v"(v[2]), "v"(v[3]), "v"(v[4]), "s"(srd), "s"(soff) : "memory");
}
__device__ __forceinline__ void store16_sys(void* ptr, u32x4 v) {
  asm volatile("global_store_dwordx4 %0, %1, off sc0 sc1\n\ts_nop 1" :: "v"(ptr), "v"(v) : "memory");
}
__device__ __forceinline__ void store4_sys(void* ptr, unsigned v) {
  asm volatile("global_store_dword %0, %1, off sc0 sc1" :: "v"(ptr), "v"(v) : "memory");
}
__device__ __forceinline__ unsigned sload(const void* p) {
  unsigned v;
  asm volatile("s_load_dword %0, %1, 0x0\n\ts_waitcnt lgkmcnt(0)" : "=s"(v) : "s"(p) : "memory");
  return v;
}
__device__ __forceinline__ unsigned now32() { return (unsigned)__builtin_amdgcn_s_memrealtime(); }
__device__ __forceinline__ unsigned xrow_off(unsigned row) { return row * XP; }
__device__ __forceinline__ unsigned xq_off(unsigned q) { return q * XQ; }
__device__ __forceinline__ int unit_begin(int mt) { return 4 * mt + min(mt, 4); }

constexpr int stage_of(int r) { return r < 8 ? r + 4 : r < 12 ? r + 8 : r < 14 ? r + 18 : r + 10; }
constexpr int read_in(int r) { return r == 12 || r == 13 ? 12 : stage_of(r) / 2 + 2; }
constexpr int w_ring(int T) { return T < 6 ? T + 1 : T < 7 ? 8 : T <= 12 ? T + 3 : T + 4; }
constexpr int w_cnt(int T) { return T > 15 ? 0 : T == 5 || T == 6 || T == 12 ? 2 : 1; }
constexpr int in_flight(int T, int nx) {
  int n = 0;
  for (int k = T; k >= T - 1; --k) {
    for (int i = w_cnt(k) - 1; i >= 0; --i) { if (read_in(w_ring(k) + i) <= T + 1) return n; n += 4; }
    if (k == T) n += 2 * nx;
  }
  return n;
}
constexpr int in_flight_loop(int nx) {
  int m = 63;
  for (int T = 7; T < 11; ++T) m = in_flight(T, nx) < m ? in_flight(T, nx) : m;
  return m;
}
constexpr bool w_schedule_ok() {
  for (int T = 5; T < 16; ++T) {
    if (read_in(w_ring(T) + w_cnt(T) - 1 - NB) >= T) return false;
    if (read_in(w_ring(T)) <= T) return false;
    if (w_ring(T) != w_ring(T - 1) + w_cnt(T - 1)) return false;
  }
  return w_ring(15) == 19;
}
static_assert(w_schedule_ok() && NB == 6 && NA == 6, "W schedule");

#define WB0(s) "v[" #s "+0:" #s "+3]"
#define WB1(s) "v[" #s "+4:" #s "+7]"
#define XU(s, u) "v[" #s "+8+4*" #u ":" #s "+11+4*" #u "]"
#define c00 "v[24:27]"
#define c01 "v[28:31]"
#define c10 "v[32:35]"
#define c11 "v[36:39]"
#define c20 "v[40:43]"
#define c21 "v[44:47]"
#define c30 "v[48:51]"
#define c31 "v[52:55]"
#define c40 "v[56:59]"
#define c41 "v[60:63]"
#define MF(acc, w, x) "v_mfma_i32_16x16x64_i8 " acc ", " x ", " w ", " acc "\n\t"
#define TRL(s, o, p, off, k) "ds_read_b64_tr_b8 v[" #s "+" #o ":" #s "+" #o "+1], %[" #p "] offset:" #off "+%[stepB]*(" #k "&1)\n\t"
#define RX(s, u, p, k) "ds_read_b128 " XU(s, u) ", %[" #p "] offset:2048*" #u "+%[aslot]*(" #k "&1)\n\t"
#define IF5(x) ".if %[nu] == 5\n\t" x ".endif\n\t"
#define IF4(x) ".if %[nu] == 4\n\t" x ".endif\n\t"

#define PICK(d, m, i, wrap, step) "s_bitcmp1_b32 %[" #m "], " #i "\n\t", "s_cselect_b32 %[" #d "], %[" #wrap "], 2*%[" #step "]\n\t"
#define ADV(p, d) "v_add_u32 %[" #p "], %[" #d "], %[" #p "]\n\t"
#define BAR(k) ".if (" #k "&1) == 0\n\ts_barrier\n\t.endif\n\t"
#define HALF_AE(k) \
    MF(c00, WB0(92), XU(92, 0)) \
    MF(c01, WB1(92), XU(92, 0)) \
    "s_waitcnt lgkmcnt(0)\n\ts_barrier\n\t" \
    TRL(64, 0, pw0, 0, k) TRL(64, 2, pw0, 1024, k) \
    MF(c11, WB1(92), XU(92, 1)) RX(64, 0, pa0, k) \
    MF(c10, WB0(92), XU(92, 1)) TRL(64, 4, pw1, 0, k) TRL(64, 6, pw1, 1024, k) \
    MF(c20, WB0(92), XU(92, 2)) RX(64, 1, pa0, k) \
    MF(c21, WB1(92), XU(92, 2)) RX(64, 2, pa0, k) \
    MF(c31, WB1(92), XU(92, 3)) RX(64, 3, pa0, k) \
    MF(c30, WB0(92), XU(92, 3)) IF5(RX(64, 4, pa0, k)) \
    IF5(MF(c40, WB0(92), XU(92, 4)) MF(c41, WB1(92), XU(92, 4)))
#define HALF_AO(k) \
    MF(c00, WB0(92), XU(92, 0)) TRL(64, 0, pw0, 0, k) TRL(64, 2, pw0, 1024, k) \
    MF(c01, WB1(92), XU(92, 0)) RX(64, 0, pa0, k) \
    "s_waitcnt lgkmcnt(%[nu])\n\t" \
    MF(c11, WB1(92), XU(92, 1)) TRL(64, 4, pw1, 0, k) TRL(64, 6, pw1, 1024, k) \
    MF(c10, WB0(92), XU(92, 1)) RX(64, 1, pa0, k) \
    MF(c20, WB0(92), XU(92, 2)) RX(64, 2, pa0, k) \
    MF(c21, WB1(92), XU(92, 2)) RX(64, 3, pa0, k) \
    "s_waitcnt lgkmcnt(8)\n\t" \
    MF(c31, WB1(92), XU(92, 3)) IF5(RX(64, 4, pa0, k)) \
    MF(c30, WB0(92), XU(92, 3)) \
    IF5(MF(c40, WB0(92), XU(92, 4)) MF(c41, WB1(92), XU(92, 4)))
#define HALF_B_(k, E1, E2, E3, E4, E5) \
    "s_waitcnt lgkmcnt(%[nu]-1)\n\t" \
    MF(c00, WB0(64), XU(64, 0)) TRL(92, 0, pw0, 8192, k) TRL(92, 2, pw0, 9216, k) \
    MF(c01, WB1(64), XU(64, 0)) RX(92, 0, pa1, k) \
    "s_waitcnt lgkmcnt(%[nu])\n\t" \
    MF(c11, WB1(64), XU(64, 1)) TRL(92, 4, pw1, 8192, k) TRL(92, 6, pw1, 9216, k) \
    MF(c10, WB0(64), XU(64, 1)) RX(92, 1, pa1, k) E1 \
    MF(c20, WB0(64), XU(64, 2)) RX(92, 2, pa1, k) E2 \
    MF(c21, WB1(64), XU(64, 2)) RX(92, 3, pa1, k) \
    "s_waitcnt lgkmcnt(8)\n\t" \
    MF(c31, WB1(64), XU(64, 3)) IF5(RX(92, 4, pa1, k)) E3 \
    IF5(MF(c30, WB0(64), XU(64, 3))) E4 \
    IF5(MF(c40, WB0(64), XU(64, 4))) E5 \
    IF5(MF(c41, WB1(64), XU(64, 4))) IF4(MF(c30, WB0(64), XU(64, 3))) \
    "s_waitcnt lgkmcnt(%[nu]-1)\n\t"
#define HALF_B(...) HALF_B_(__VA_ARGS__)

#define RW(b, k, o) "v[" #b "+16*" #k "+" #o ":" #b "+16*" #k "+" #o "+3]"
#define REG_A(k, P0, P1, X) \
    MF(c00, P0, XU(92, 0)) \
    MF(c01, P1, XU(92, 0)) \
    "s_waitcnt lgkmcnt(0)\n\t" BAR(k) \
    RX(64, 0, pa0, k) \
    MF(c11, P1, XU(92, 1)) RX(64, 1, pa0, k) \
    MF(c10, P0, XU(92, 1)) RX(64, 2, pa0, k) \
    MF(c20, P0, XU(92, 2)) RX(64, 3, pa0, k) \
    MF(c21, P1, XU(92, 2)) IF5(RX(64, 4, pa0, k)) IF4(X(k, 0, pw0, 0) X(k, 4, pw1, 0)) \
    MF(c31, P1, XU(92, 3)) IF5(X(k, 0, pw0, 0)) IF4(X(k, 8, pw0, 8192)) \
    MF(c30, P0, XU(92, 3)) IF5(X(k, 4, pw1, 0)) IF4(X(k, 12, pw1, 8192)) \
    IF5(MF(c40, P0, XU(92, 4)) X(k, 8, pw0, 8192) MF(c41, P1, XU(92, 4)) X(k, 12, pw1, 8192))
#define NOX(k, o, p, off)
#define LASTW(k, o, p, off) TRL(152+16*k, o, p, off, k) TRL(152+16*k, o+2, p, off+1024, k)
#define REG_B_(k, N, C0, C1, E1, E2, E3) \
    "s_waitcnt lgkmcnt(" #N ")\n\t" \
    MF(c00, C0, XU(64, 0)) RX(92, 0, pa1, k) \
    MF(c01, C1, XU(64, 0)) RX(92, 1, pa1, k) \
    MF(c11, C1, XU(64, 1)) RX(92, 2, pa1, k) \
    MF(c10, C0, XU(64, 1)) RX(92, 3, pa1, k) \
    MF(c20, C0, XU(64, 2)) IF5(RX(92, 4, pa1, k)) E1 \
    MF(c21, C1, XU(64, 2)) E2 \
    MF(c31, C1, XU(64, 3)) E3 \
    MF(c30, C0, XU(64, 3)) \
    IF5(MF(c40, C0, XU(64, 4)) MF(c41, C1, XU(64, 4))) \
    "s_waitcnt lgkmcnt(%[nu]-1)\n\t"
#define REG_B(...) REG_B_(__VA_ARGS__)
#define MOV2(d, s) "v_mov_b64 v[" #d ":" #d "+1], v[" #s ":" #s "+1]\n\tv_mov_b64 v[" #d "+2:" #d "+3], v[" #s "+2:" #s "+3]\n\t"
#define REG_CHUNK(b, X, N, EB) \
    REG_A(0, WB0(92), WB1(92), X) REG_B(0, N, RW(b, 0, 0), RW(b, 0, 4), PICK(dA, wrapsA, 0, wrapA, aslot), "s_add_u32 %[C], %[C], 1\n\t") \
    REG_A(1, RW(b, 0, 8), RW(b, 0, 12), X) REG_B(1, N, RW(b, 1, 0), RW(b, 1, 4), ADV(pa0, dA), ADV(pa1, dA), EB) \
    REG_A(2, RW(b, 1, 8), RW(b, 1, 12), NOX) REG_B(2, 0, RW(b, 2, 0), RW(b, 2, 4), PICK(dA, wrapsA, 1, wrapA, aslot), "s_lshr_b32 %[wrapsA], %[wrapsA], 2\n\t") \
    REG_A(3, RW(b, 2, 8), RW(b, 2, 12), NOX) REG_B(3, 0, RW(b, 3, 0), RW(b, 3, 4), MOV2(92, b+56), MOV2(96, b+60), ADV(pa0, dA) ADV(pa1, dA))
#define TRR(d, p, off) "ds_read_b64_tr_b8 v[" d ":" d "+1], %[" #p "] offset:" off "\n\t"
#define PASS(n) \
    "s_barrier\n\t" \
    TRR("56+16*" #n, pw0, "(" #n "&3)*%[stepB]") TRR("58+16*" #n, pw0, "(" #n "&3)*%[stepB]+1024") \
    TRR("60+16*" #n, pw1, "(" #n "&3)*%[stepB]") TRR("62+16*" #n, pw1, "(" #n "&3)*%[stepB]+1024") \
    TRR("64+16*" #n, pw0, "(" #n "&3)*%[stepB]+8192") TRR("66+16*" #n, pw0, "(" #n "&3)*%[stepB]+9216") \
    TRR("68+16*" #n, pw1, "(" #n "&3)*%[stepB]+8192") TRR("70+16*" #n, pw1, "(" #n "&3)*%[stepB]+9216") \
    "s_waitcnt lgkmcnt(0)\n\t"
#define FINAL5 \
    "ds_read_b64_tr_b8 v[120:121], %[pw0] offset:0\n\tds_read_b64_tr_b8 v[122:123], %[pw0] offset:1024\n\tds_read_b64_tr_b8 v[124:125], %[pw1] offset:0\n\t" \
    "v_mfma_i32_16x16x64_i8 v[36:39], v[104:107], v[96:99], v[36:39]\n\tds_read_b64_tr_b8 v[126:127], %[pw1] offset:1024\n\t" \
    "ds_read_b128 v[184:187], %[pa0] offset:0\n\tv_mfma_i32_16x16x64_i8 v[32:35], v[104:107], v[92:95], v[32:35]\n\t" \
    "ds_read_b64_tr_b8 v[128:129], %[pw0] offset:8192\n\tds_read_b64_tr_b8 v[130:131], %[pw0] offset:9216\n\t" \
    "v_mfma_i32_16x16x64_i8 v[40:43], v[108:111], v[92:95], v[40:43]\n\tds_read_b64_tr_b8 v[132:133], %[pw1] offset:8192\n\t" \
    "ds_read_b64_tr_b8 v[134:135], %[pw1] offset:9216\n\tv_mfma_i32_16x16x64_i8 v[44:47], v[108:111], v[96:99], v[44:47]\n\t" \
    "ds_read_b128 v[188:191], %[pa1] offset:0\n\tds_read_b64_tr_b8 v[136:137], %[pw0] offset:16384\n\tds_read_b64_tr_b8 v[138:139], %[pw0] offset:17408\n\t" \
    "v_mfma_i32_16x16x64_i8 v[52:55], v[112:115], v[96:99], v[52:55]\n\tds_read_b64_tr_b8 v[140:141], %[pw1] offset:16384\n\t" \
    "ds_read_b64_tr_b8 v[142:143], %[pw1] offset:17408\n\tv_mfma_i32_16x16x64_i8 v[48:51], v[112:115], v[92:95], v[48:51]\n\t" \
    "ds_read_b128 v[192:195], %[pa0] offset:10240\n\ts_waitcnt lgkmcnt(9)\n\tds_read_b64_tr_b8 v[144:145], %[pw0] offset:24576\n\t" \
    "ds_read_b64_tr_b8 v[146:147], %[pw0] offset:25600\n\tv_mfma_i32_16x16x64_i8 v[56:59], v[116:119], v[92:95], v[56:59]\n\t" \
    "ds_read_b64_tr_b8 v[148:149], %[pw1] offset:24576\n\tds_read_b64_tr_b8 v[150:151], %[pw1] offset:25600\n\t" \
    "v_mfma_i32_16x16x64_i8 v[60:63], v[116:119], v[96:99], v[60:63]\n\tds_read_b128 v[196:199], %[pa1] offset:10240\n\t" \
    "ds_read_b128 v[200:203], %[pa0] offset:2048\n\ts_waitcnt lgkmcnt(9)\n\tds_read_b128 v[204:207], %[pa1] offset:2048\n\t" \
    "v_mfma_i32_16x16x64_i8 v[24:27], v[184:187], v[120:123], v[24:27]\n\tds_read_b128 v[208:211], %[pa0] offset:12288\n\t" \
    "ds_read_b128 v[212:215], %[pa1] offset:12288\n\tv_mfma_i32_16x16x64_i8 v[28:31], v[184:187], v[124:127], v[28:31]\n\t" \
    "v_subrev_u32 v64, 4*%[aslot], %[pa0]\n\tv_subrev_u32 v65, 4*%[aslot], %[pa1]\n\tv_mfma_i32_16x16x64_i8 v[24:27], v[188:191], v[128:131], v[24:27]\n\t" \
    "v_add_u32 v69, 16384, %[vo]\n\tv_add_u32 v70, 32768, %[vo]\n\tv_mfma_i32_16x16x64_i8 v[28:31], v[188:191], v[132:135], v[28:31]\n\t" \
    "v_add_u32 v71, 49152, %[vo]\n\ts_waitcnt lgkmcnt(7)\n\tv_mfma_i32_16x16x64_i8 v[24:27], v[192:195], v[136:139], v[24:27]\n\t" \
    "v_mfma_i32_16x16x64_i8 v[28:31], v[192:195], v[140:143], v[28:31]\n\ts_waitcnt lgkmcnt(2)\n\t" \
    "v_mfma_i32_16x16x64_i8 v[24:27], v[196:199], v[144:147], v[24:27]\n\tv_mfma_i32_16x16x64_i8 v[28:31], v[196:199], v[148:151], v[28:31]\n\t" \
    "s_barrier\n\tds_read_b32 v78, %[vtmo]\n\tds_read_b128 v[216:219], v64 offset:0\n\tds_read_b128 v[220:223], v65 offset:0\n\t" \
    "v_mfma_i32_16x16x64_i8 v[32:35], v[200:203], v[120:123], v[32:35]\n\tds_read_b128 v[224:227], v64 offset:10240\n\t" \
    "ds_read_b128 v[228:231], v65 offset:10240\n\tv_mfma_i32_16x16x64_i8 v[36:39], v[200:203], v[124:127], v[36:39]\n\t" \
    "ds_read_b128 v[232:235], v64 offset:2048\n\tds_read_b128 v[236:239], v65 offset:2048\n\t" \
    "v_mfma_i32_16x16x64_i8 v[32:35], v[204:207], v[128:131], v[32:35]\n\tv_mov_b32 v79, 0x7fc00000\n\t" \
    "v_mfma_i32_16x16x64_i8 v[36:39], v[204:207], v[132:135], v[36:39]\n\tds_read_b128 v[240:243], v64 offset:12288\n\ts_waitcnt lgkmcnt(8)\n\t" \
    "v_mfma_i32_16x16x64_i8 v[32:35], v[208:211], v[136:139], v[32:35]\n\ts_waitcnt lgkmcnt(7)\n\tv_cmp_ne_u32 vcc, 0, v78\n\ts_nop 1\n\t" \
    "v_mfma_i32_16x16x64_i8 v[36:39], v[208:211], v[140:143], v[36:39]\n\tds_read_b128 v[244:247], v65 offset:12288\n\t" \
    "v_cndmask_b32 v76, %[sc], v79, vcc\n\tv_mfma_i32_16x16x64_i8 v[32:35], v[212:215], v[144:147], v[32:35]\n\tv_cndmask_b32 v77, %[sc], v79, vcc\n\t" \
    "v_mfma_i32_16x16x64_i8 v[36:39], v[212:215], v[148:151], v[36:39]\n\tds_read_b128 v[184:187], %[pa0] offset:4096\n\ts_waitcnt lgkmcnt(5)\n\t" \
    "v_mfma_i32_16x16x64_i8 v[24:27], v[216:219], v[152:155], v[24:27]\n\tv_mfma_i32_16x16x64_i8 v[28:31], v[216:219], v[156:159], v[28:31]\n\t" \
    "ds_read_b128 v[188:191], %[pa1] offset:4096\n\tv_mfma_i32_16x16x64_i8 v[24:27], v[220:223], v[160:163], v[24:27]\n\t" \
    "v_mfma_i32_16x16x64_i8 v[28:31], v[220:223], v[164:167], v[28:31]\n\tds_read_b128 v[192:195], %[pa0] offset:14336\n\t" \
    "v_mfma_i32_16x16x64_i8 v[24:27], v[224:227], v[168:171], v[24:27]\n\tv_mfma_i32_16x16x64_i8 v[28:31], v[224:227], v[172:175], v[28:31]\n\t" \
    "ds_read_b128 v[196:199], %[pa1] offset:14336\n\tv_mfma_i32_16x16x64_i8 v[24:27], v[228:231], v[176:179], v[24:27]\n\t" \
    "v_mfma_i32_16x16x64_i8 v[28:31], v[228:231], v[180:183], v[28:31]\n\tds_read_b128 v[200:203], v64 offset:4096\n\ts_waitcnt lgkmcnt(5)\n\t" \
    "v_mfma_i32_16x16x64_i8 v[32:35], v[232:235], v[152:155], v[32:35]\n\ts_mov_b32 %[dA], 0\n\t" \
    "v_mfma_i32_16x16x64_i8 v[36:39], v[232:235], v[156:159], v[36:39]\n\tds_read_b128 v[204:207], v65 offset:4096\n\t" \
    "v_mfma_i32_16x16x64_i8 v[32:35], v[236:239], v[160:163], v[32:35]\n\tv_mfma_i32_16x16x64_i8 v[36:39], v[236:239], v[164:167], v[36:39]\n\t" \
    "ds_read_b128 v[208:211], v64 offset:14336\n\tv_mfma_i32_16x16x64_i8 v[32:35], v[240:243], v[168:171], v[32:35]\n\tv_cvt_f32_i32 v24, v24\n\t" \
    "v_cvt_f32_i32 v25, v25\n\tv_cvt_f32_i32 v28, v28\n\tv_mfma_i32_16x16x64_i8 v[36:39], v[240:243], v[172:175], v[36:39]\n\t" \
    "ds_read_b128 v[212:215], v65 offset:14336\n\tv_cvt_f32_i32 v29, v29\n\tv_mul_f32 v24, v76, v24\n\t" \
    "v_mfma_i32_16x16x64_i8 v[32:35], v[244:247], v[176:179], v[32:35]\n\tv_mul_f32 v28, v76, v28\n\tv_mul_f32 v25, v76, v25\n\tv_mul_f32 v29, v76, v29\n\t" \
    "v_mfma_i32_16x16x64_i8 v[36:39], v[244:247], v[180:183], v[36:39]\n\tds_read_b128 v[216:219], %[pa0] offset:6144\n\tv_maximum3_f32 v24, v24, 0, 0\n\t" \
    "v_maximum3_f32 v28, v28, 0, 0\n\ts_waitcnt lgkmcnt(5)\n\tv_mfma_i32_16x16x64_i8 v[40:43], v[184:187], v[120:123], v[40:43]\n\t" \
    "v_maximum3_f32 v25, v25, 0, 0\n\tv_maximum3_f32 v29, v29, 0, 0\n\tv_cvt_f32_i32 v26, v26\n\t" \
    "v_mfma_i32_16x16x64_i8 v[44:47], v[184:187], v[124:127], v[44:47]\n\tds_read_b128 v[220:223], %[pa1] offset:6144\n\t" \
    "buffer_store_dword v24, %[vo], %[srd], %[dA] offen sc1\n\tv_cvt_f32_i32 v27, v27\n\t" \
    "v_mfma_i32_16x16x64_i8 v[40:43], v[188:191], v[128:131], v[40:43]\n\tv_cvt_f32_i32 v30, v30\n\tv_cvt_f32_i32 v31, v31\n\t" \
    "buffer_store_dword v28, %[vo], %[srd], %[dA] offen offset:64 sc1\n\tv_mfma_i32_16x16x64_i8 v[44:47], v[188:191], v[132:135], v[44:47]\n\t" \
    "ds_read_b128 v[224:227], %[pa0] offset:16384\n\tv_mul_f32 v26, v76, v26\n\tv_mul_f32 v30, v76, v30\n\t" \
    "v_mfma_i32_16x16x64_i8 v[40:43], v[192:195], v[136:139], v[40:43]\n\tbuffer_store_dword v25, v69, %[srd], %[dA] offen sc1\n\t" \
    "v_mul_f32 v27, v76, v27\n\tv_mul_f32 v31, v76, v31\n\tv_mfma_i32_16x16x64_i8 v[44:47], v[192:195], v[140:143], v[44:47]\n\t" \
    "ds_read_b128 v[228:231], %[pa1] offset:16384\n\tbuffer_store_dword v29, v69, %[srd], %[dA] offen offset:64 sc1\n\tv_maximum3_f32 v26, v26, 0, 0\n\t" \
    "v_mfma_i32_16x16x64_i8 v[40:43], v[196:199], v[144:147], v[40:43]\n\tv_maximum3_f32 v30, v30, 0, 0\n\tv_maximum3_f32 v27, v27, 0, 0\n\t" \
    "v_maximum3_f32 v31, v31, 0, 0\n\tv_mfma_i32_16x16x64_i8 v[44:47], v[196:199], v[148:151], v[44:47]\n\tds_read_b128 v[232:235], v64 offset:6144\n\t" \
    "v_cvt_f32_i32 v32, v32\n\tbuffer_store_dword v26, v70, %[srd], %[dA] offen sc1\n\ts_waitcnt lgkmcnt(5)\n\t" \
    "v_mfma_i32_16x16x64_i8 v[40:43], v[200:203], v[152:155], v[40:43]\n\tv_cvt_f32_i32 v33, v33\n\tv_cvt_f32_i32 v36, v36\n\t" \
    "buffer_store_dword v30, v70, %[srd], %[dA] offen offset:64 sc1\n\tv_mfma_i32_16x16x64_i8 v[44:47], v[200:203], v[156:159], v[44:47]\n\t" \
    "ds_read_b128 v[236:239], v65 offset:6144\n\tv_cvt_f32_i32 v37, v37\n\tv_mul_f32 v32, v76, v32\n\t" \
    "v_mfma_i32_16x16x64_i8 v[40:43], v[204:207], v[160:163], v[40:43]\n\tbuffer_store_dword v27, v71, %[srd], %[dA] offen sc1\n\t" \
    "v_mul_f32 v36, v76, v36\n\tv_mul_f32 v33, v76, v33\n\tv_mfma_i32_16x16x64_i8 v[44:47], v[204:207], v[164:167], v[44:47]\n\t" \
    "ds_read_b128 v[240:243], v64 offset:16384\n\tbuffer_store_dword v31, v71, %[srd], %[dA] offen offset:64 sc1\n\tv_mul_f32 v37, v76, v37\n\t" \
    "v_mfma_i32_16x16x64_i8 v[40:43], v[208:211], v[168:171], v[40:43]\n\tv_maximum3_f32 v32, v32, 0, 0\n\tv_maximum3_f32 v36, v36, 0, 0\n\t" \
    "s_mov_b32 %[dA], 262144\n\tv_mfma_i32_16x16x64_i8 v[44:47], v[208:211], v[172:175], v[44:47]\n\tds_read_b128 v[244:247], v65 offset:16384\n\t" \
    "v_maximum3_f32 v33, v33, 0, 0\n\tv_maximum3_f32 v37, v37, 0, 0\n\tv_mfma_i32_16x16x64_i8 v[40:43], v[212:215], v[176:179], v[40:43]\n\t" \
    "v_cvt_f32_i32 v34, v34\n\tbuffer_store_dword v32, %[vo], %[srd], %[dA] offen sc1\n\tv_cvt_f32_i32 v35, v35\n\t" \
    "v_mfma_i32_16x16x64_i8 v[44:47], v[212:215], v[180:183], v[44:47]\n\tds_read_b128 v[184:187], %[pa0] offset:8192\n\tv_cvt_f32_i32 v38, v38\n\t" \
    "buffer_store_dword v36, %[vo], %[srd], %[dA] offen offset:64 sc1\n\ts_waitcnt lgkmcnt(5)\n\t" \
    "v_mfma_i32_16x16x64_i8 v[48:51], v[216:219], v[120:123], v[48:51]\n\tv_cvt_f32_i32 v39, v39\n\tv_mul_f32 v34, v76, v34\n\t" \
    "buffer_store_dword v33, v69, %[srd], %[dA] offen sc1\n\tv_mfma_i32_16x16x64_i8 v[52:55], v[216:219], v[124:127], v[52:55]\n\t" \
    "ds_read_b128 v[188:191], %[pa1] offset:8192\n\tv_mul_f32 v38, v76, v38\n\tv_mul_f32 v35, v76, v35\n\t" \
    "v_mfma_i32_16x16x64_i8 v[48:51], v[220:223], v[128:131], v[48:51]\n\tbuffer_store_dword v37, v69, %[srd], %[dA] offen offset:64 sc1\n\t" \
    "v_mul_f32 v39, v76, v39\n\tv_maximum3_f32 v34, v34, 0, 0\n\tv_mfma_i32_16x16x64_i8 v[52:55], v[220:223], v[132:135], v[52:55]\n\t" \
    "ds_read_b128 v[192:195], %[pa0] offset:18432\n\tv_maximum3_f32 v38, v38, 0, 0\n\tv_maximum3_f32 v35, v35, 0, 0\n\t" \
    "v_mfma_i32_16x16x64_i8 v[48:51], v[224:227], v[136:139], v[48:51]\n\tv_maximum3_f32 v39, v39, 0, 0\n\tv_cvt_f32_i32 v40, v40\n\t" \
    "buffer_store_dword v34, v70, %[srd], %[dA] offen sc1\n\tv_mfma_i32_16x16x64_i8 v[52:55], v[224:227], v[140:143], v[52:55]\n\t" \
    "ds_read_b128 v[196:199], %[pa1] offset:18432\n\tv_cvt_f32_i32 v41, v41\n\tv_cvt_f32_i32 v44, v44\n\t" \
    "v_mfma_i32_16x16x64_i8 v[48:51], v[228:231], v[144:147], v[48:51]\n\tbuffer_store_dword v38, v70, %[srd], %[dA] offen offset:64 sc1\n\t" \
    "v_cvt_f32_i32 v45, v45\n\tv_mul_f32 v40, v76, v40\n\tv_mfma_i32_16x16x64_i8 v[52:55], v[228:231], v[148:151], v[52:55]\n\t" \
    "ds_read_b128 v[200:203], v64 offset:8192\n\tbuffer_store_dword v35, v71, %[srd], %[dA] offen sc1\n\tv_mul_f32 v44, v76, v44\n\t" \
    "s_waitcnt lgkmcnt(5)\n\tv_mfma_i32_16x16x64_i8 v[48:51], v[232:235], v[152:155], v[48:51]\n\tv_mul_f32 v41, v76, v41\n\t" \
    "buffer_store_dword v39, v71, %[srd], %[dA] offen offset:64 sc1\n\tv_mul_f32 v45, v76, v45\n\t" \
    "v_mfma_i32_16x16x64_i8 v[52:55], v[232:235], v[156:159], v[52:55]\n\tds_read_b128 v[204:207], v65 offset:8192\n\tv_maximum3_f32 v40, v40, 0, 0\n\t" \
    "s_mov_b32 %[dA], 524288\n\tv_mfma_i32_16x16x64_i8 v[48:51], v[236:239], v[160:163], v[48:51]\n\tv_maximum3_f32 v44, v44, 0, 0\n\t" \
    "v_maximum3_f32 v41, v41, 0, 0\n\tv_maximum3_f32 v45, v45, 0, 0\n\tv_mfma_i32_16x16x64_i8 v[52:55], v[236:239], v[164:167], v[52:55]\n\t" \
    "ds_read_b128 v[208:211], v64 offset:18432\n\tv_cvt_f32_i32 v42, v42\n\tbuffer_store_dword v40, %[vo], %[srd], %[dA] offen sc1\n\t" \
    "v_mfma_i32_16x16x64_i8 v[48:51], v[240:243], v[168:171], v[48:51]\n\tv_cvt_f32_i32 v43, v43\n\tv_cvt_f32_i32 v46, v46\n\tv_cvt_f32_i32 v47, v47\n\t" \
    "v_mfma_i32_16x16x64_i8 v[52:55], v[240:243], v[172:175], v[52:55]\n\tds_read_b128 v[212:215], v65 offset:18432\n\t" \
    "buffer_store_dword v44, %[vo], %[srd], %[dA] offen offset:64 sc1\n\tv_mul_f32 v42, v76, v42\n\t" \
    "v_mfma_i32_16x16x64_i8 v[48:51], v[244:247], v[176:179], v[48:51]\n\tv_mul_f32 v46, v76, v46\n\tv_mul_f32 v43, v76, v43\n\t" \
    "buffer_store_dword v41, v69, %[srd], %[dA] offen sc1\n\tv_mfma_i32_16x16x64_i8 v[52:55], v[244:247], v[180:183], v[52:55]\n\t" \
    "v_mul_f32 v47, v76, v47\n\tv_maximum3_f32 v42, v42, 0, 0\n\tv_maximum3_f32 v46, v46, 0, 0\n\ts_waitcnt lgkmcnt(4)\n\t" \
    "v_mfma_i32_16x16x64_i8 v[56:59], v[184:187], v[120:123], v[56:59]\n\tbuffer_store_dword v45, v69, %[srd], %[dA] offen offset:64 sc1\n\t" \
    "v_maximum3_f32 v43, v43, 0, 0\n\tv_maximum3_f32 v47, v47, 0, 0\n\tv_mfma_i32_16x16x64_i8 v[60:63], v[184:187], v[124:127], v[60:63]\n\t" \
    "v_mfma_i32_16x16x64_i8 v[56:59], v[188:191], v[128:131], v[56:59]\n\tbuffer_store_dword v42, v70, %[srd], %[dA] offen sc1\n\t" \
    "v_cvt_f32_i32 v48, v48\n\tv_cvt_f32_i32 v49, v49\n\tv_mfma_i32_16x16x64_i8 v[60:63], v[188:191], v[132:135], v[60:63]\n\tv_cvt_f32_i32 v52, v52\n\t" \
    "buffer_store_dword v46, v70, %[srd], %[dA] offen offset:64 sc1\n\tv_cvt_f32_i32 v53, v53\n\t" \
    "v_mfma_i32_16x16x64_i8 v[56:59], v[192:195], v[136:139], v[56:59]\n\tv_mul_f32 v48, v76, v48\n\tv_mul_f32 v52, v76, v52\n\t" \
    "buffer_store_dword v43, v71, %[srd], %[dA] offen sc1\n\tv_mfma_i32_16x16x64_i8 v[60:63], v[192:195], v[140:143], v[60:63]\n\t" \
    "v_mul_f32 v49, v76, v49\n\tv_mul_f32 v53, v76, v53\n\tv_maximum3_f32 v48, v48, 0, 0\n\t" \
    "v_mfma_i32_16x16x64_i8 v[56:59], v[196:199], v[144:147], v[56:59]\n\tbuffer_store_dword v47, v71, %[srd], %[dA] offen offset:64 sc1\n\t" \
    "v_maximum3_f32 v52, v52, 0, 0\n\tv_maximum3_f32 v49, v49, 0, 0\n\tv_mfma_i32_16x16x64_i8 v[60:63], v[196:199], v[148:151], v[60:63]\n\t" \
    "v_maximum3_f32 v53, v53, 0, 0\n\ts_mov_b32 %[dA], 786432\n\tv_cvt_f32_i32 v50, v50\n\ts_waitcnt lgkmcnt(0)\n\t" \
    "v_mfma_i32_16x16x64_i8 v[56:59], v[200:203], v[152:155], v[56:59]\n\tbuffer_store_dword v48, %[vo], %[srd], %[dA] offen sc1\n\t" \
    "v_cvt_f32_i32 v51, v51\n\tv_cvt_f32_i32 v54, v54\n\tv_mfma_i32_16x16x64_i8 v[60:63], v[200:203], v[156:159], v[60:63]\n\tv_cvt_f32_i32 v55, v55\n\t" \
    "buffer_store_dword v52, %[vo], %[srd], %[dA] offen offset:64 sc1\n\tv_mul_f32 v50, v76, v50\n\t" \
    "v_mfma_i32_16x16x64_i8 v[56:59], v[204:207], v[160:163], v[56:59]\n\tv_mul_f32 v54, v76, v54\n\tv_mul_f32 v51, v76, v51\n\t" \
    "buffer_store_dword v49, v69, %[srd], %[dA] offen sc1\n\tv_mfma_i32_16x16x64_i8 v[60:63], v[204:207], v[164:167], v[60:63]\n\t" \
    "v_mul_f32 v55, v76, v55\n\tv_maximum3_f32 v50, v50, 0, 0\n\tv_maximum3_f32 v54, v54, 0, 0\n\t" \
    "v_mfma_i32_16x16x64_i8 v[56:59], v[208:211], v[168:171], v[56:59]\n\tbuffer_store_dword v53, v69, %[srd], %[dA] offen offset:64 sc1\n\t" \
    "v_maximum3_f32 v51, v51, 0, 0\n\tv_maximum3_f32 v55, v55, 0, 0\n\tv_mfma_i32_16x16x64_i8 v[60:63], v[208:211], v[172:175], v[60:63]\n\t" \
    "v_mfma_i32_16x16x64_i8 v[56:59], v[212:215], v[176:179], v[56:59]\n\tbuffer_store_dword v50, v70, %[srd], %[dA] offen sc1\n\t" \
    "v_mfma_i32_16x16x64_i8 v[60:63], v[212:215], v[180:183], v[60:63]\n\tbuffer_store_dword v54, v70, %[srd], %[dA] offen offset:64 sc1\n\t" \
    "buffer_store_dword v51, v71, %[srd], %[dA] offen sc1\n\tbuffer_store_dword v55, v71, %[srd], %[dA] offen offset:64 sc1\n\ts_mov_b32 %[dA], 1048576\n\t" \
    "s_nop 5\n\tv_cvt_f32_i32 v56, v56\n\tv_cvt_f32_i32 v57, v57\n\tv_cvt_f32_i32 v60, v60\n\tv_cvt_f32_i32 v61, v61\n\tv_mul_f32 v56, v76, v56\n\t" \
    "v_mul_f32 v60, v76, v60\n\tv_mul_f32 v57, v76, v57\n\tv_mul_f32 v61, v76, v61\n\tv_maximum3_f32 v56, v56, 0, 0\n\tv_maximum3_f32 v60, v60, 0, 0\n\t" \
    "v_maximum3_f32 v57, v57, 0, 0\n\tv_maximum3_f32 v61, v61, 0, 0\n\tv_cvt_f32_i32 v58, v58\n\tbuffer_store_dword v56, %[vo], %[srd], %[dA] offen sc1\n\t" \
    "v_cvt_f32_i32 v59, v59\n\tv_cvt_f32_i32 v62, v62\n\tv_cvt_f32_i32 v63, v63\n\tv_mul_f32 v58, v76, v58\n\t" \
    "buffer_store_dword v60, %[vo], %[srd], %[dA] offen offset:64 sc1\n\tv_mul_f32 v62, v76, v62\n\tv_mul_f32 v59, v76, v59\n\tv_mul_f32 v63, v76, v63\n\t" \
    "v_maximum3_f32 v58, v58, 0, 0\n\tbuffer_store_dword v57, v69, %[srd], %[dA] offen sc1\n\tv_maximum3_f32 v62, v62, 0, 0\n\t" \
    "v_maximum3_f32 v59, v59, 0, 0\n\tv_maximum3_f32 v63, v63, 0, 0\n\tbuffer_store_dword v61, v69, %[srd], %[dA] offen offset:64 sc1\n\t" \
    "buffer_store_dword v58, v70, %[srd], %[dA] offen sc1\n\tbuffer_store_dword v62, v70, %[srd], %[dA] offen offset:64 sc1\n\t" \
    "buffer_store_dword v59, v71, %[srd], %[dA] offen sc1\n\tbuffer_store_dword v63, v71, %[srd], %[dA] offen offset:64 sc1\n\t"
#define FINAL4 \
    "ds_read_b64_tr_b8 v[120:121], %[pw0] offset:0\n\tds_read_b64_tr_b8 v[122:123], %[pw0] offset:1024\n\tds_read_b64_tr_b8 v[124:125], %[pw1] offset:0\n\t" \
    "v_mfma_i32_16x16x64_i8 v[36:39], v[104:107], v[96:99], v[36:39]\n\tds_read_b64_tr_b8 v[126:127], %[pw1] offset:1024\n\t" \
    "ds_read_b128 v[184:187], %[pa0] offset:0\n\tds_read_b64_tr_b8 v[128:129], %[pw0] offset:8192\n\t" \
    "v_mfma_i32_16x16x64_i8 v[32:35], v[104:107], v[92:95], v[32:35]\n\tds_read_b64_tr_b8 v[130:131], %[pw0] offset:9216\n\t" \
    "ds_read_b64_tr_b8 v[132:133], %[pw1] offset:8192\n\tds_read_b64_tr_b8 v[134:135], %[pw1] offset:9216\n\t" \
    "v_mfma_i32_16x16x64_i8 v[40:43], v[108:111], v[92:95], v[40:43]\n\tds_read_b128 v[188:191], %[pa1] offset:0\n\t" \
    "ds_read_b64_tr_b8 v[136:137], %[pw0] offset:16384\n\tds_read_b64_tr_b8 v[138:139], %[pw0] offset:17408\n\t" \
    "v_mfma_i32_16x16x64_i8 v[44:47], v[108:111], v[96:99], v[44:47]\n\tds_read_b64_tr_b8 v[140:141], %[pw1] offset:16384\n\t" \
    "ds_read_b64_tr_b8 v[142:143], %[pw1] offset:17408\n\tds_read_b128 v[192:195], %[pa0] offset:10240\n\t" \
    "v_mfma_i32_16x16x64_i8 v[52:55], v[112:115], v[96:99], v[52:55]\n\ts_waitcnt lgkmcnt(9)\n\tds_read_b64_tr_b8 v[144:145], %[pw0] offset:24576\n\t" \
    "ds_read_b64_tr_b8 v[146:147], %[pw0] offset:25600\n\tds_read_b64_tr_b8 v[148:149], %[pw1] offset:24576\n\t" \
    "v_mfma_i32_16x16x64_i8 v[48:51], v[112:115], v[92:95], v[48:51]\n\tds_read_b64_tr_b8 v[150:151], %[pw1] offset:25600\n\t" \
    "ds_read_b128 v[196:199], %[pa1] offset:10240\n\tds_read_b128 v[200:203], %[pa0] offset:2048\n\t" \
    "v_mfma_i32_16x16x64_i8 v[24:27], v[184:187], v[120:123], v[24:27]\n\ts_waitcnt lgkmcnt(9)\n\tds_read_b128 v[204:207], %[pa1] offset:2048\n\t" \
    "ds_read_b128 v[208:211], %[pa0] offset:12288\n\tv_mfma_i32_16x16x64_i8 v[28:31], v[184:187], v[124:127], v[28:31]\n\t" \
    "ds_read_b128 v[212:215], %[pa1] offset:12288\n\tv_subrev_u32 v64, 4*%[aslot], %[pa0]\n\t" \
    "v_mfma_i32_16x16x64_i8 v[24:27], v[188:191], v[128:131], v[24:27]\n\tv_subrev_u32 v65, 4*%[aslot], %[pa1]\n\tv_add_u32 v69, 16384, %[vo]\n\t" \
    "v_mfma_i32_16x16x64_i8 v[28:31], v[188:191], v[132:135], v[28:31]\n\tv_add_u32 v70, 32768, %[vo]\n\tv_add_u32 v71, 49152, %[vo]\n\t" \
    "s_waitcnt lgkmcnt(9)\n\tv_mfma_i32_16x16x64_i8 v[24:27], v[192:195], v[136:139], v[24:27]\n\t" \
    "v_mfma_i32_16x16x64_i8 v[28:31], v[192:195], v[140:143], v[28:31]\n\ts_waitcnt lgkmcnt(3)\n\t" \
    "v_mfma_i32_16x16x64_i8 v[24:27], v[196:199], v[144:147], v[24:27]\n\tv_mfma_i32_16x16x64_i8 v[28:31], v[196:199], v[148:151], v[28:31]\n\t" \
    "s_barrier\n\tds_read_b32 v78, %[vtmo]\n\tds_read_b128 v[216:219], v64 offset:0\n\tds_read_b128 v[220:223], v65 offset:0\n\t" \
    "v_mfma_i32_16x16x64_i8 v[32:35], v[200:203], v[120:123], v[32:35]\n\tds_read_b128 v[224:227], v64 offset:10240\n\t" \
    "ds_read_b128 v[228:231], v65 offset:10240\n\tv_mfma_i32_16x16x64_i8 v[36:39], v[200:203], v[124:127], v[36:39]\n\t" \
    "ds_read_b128 v[232:235], v64 offset:2048\n\tds_read_b128 v[236:239], v65 offset:2048\n\ts_waitcnt lgkmcnt(7)\n\t" \
    "v_mfma_i32_16x16x64_i8 v[32:35], v[204:207], v[128:131], v[32:35]\n\tv_mov_b32 v79, 0x7fc00000\n\t" \
    "v_mfma_i32_16x16x64_i8 v[36:39], v[204:207], v[132:135], v[36:39]\n\tds_read_b128 v[240:243], v64 offset:12288\n\t" \
    "v_mfma_i32_16x16x64_i8 v[32:35], v[208:211], v[136:139], v[32:35]\n\ts_waitcnt lgkmcnt(7)\n\tv_cmp_ne_u32 vcc, 0, v78\n\ts_nop 1\n\t" \
    "v_mfma_i32_16x16x64_i8 v[36:39], v[208:211], v[140:143], v[36:39]\n\tds_read_b128 v[244:247], v65 offset:12288\n\t" \
    "v_cndmask_b32 v76, %[sc], v79, vcc\n\tv_mfma_i32_16x16x64_i8 v[32:35], v[212:215], v[144:147], v[32:35]\n\tv_cndmask_b32 v77, %[sc], v79, vcc\n\t" \
    "v_mfma_i32_16x16x64_i8 v[36:39], v[212:215], v[148:151], v[36:39]\n\tds_read_b128 v[184:187], %[pa0] offset:4096\n\ts_waitcnt lgkmcnt(5)\n\t" \
    "v_mfma_i32_16x16x64_i8 v[24:27], v[216:219], v[152:155], v[24:27]\n\tv_mfma_i32_16x16x64_i8 v[28:31], v[216:219], v[156:159], v[28:31]\n\t" \
    "ds_read_b128 v[188:191], %[pa1] offset:4096\n\tv_mfma_i32_16x16x64_i8 v[24:27], v[220:223], v[160:163], v[24:27]\n\t" \
    "v_mfma_i32_16x16x64_i8 v[28:31], v[220:223], v[164:167], v[28:31]\n\tds_read_b128 v[192:195], %[pa0] offset:14336\n\t" \
    "v_mfma_i32_16x16x64_i8 v[24:27], v[224:227], v[168:171], v[24:27]\n\tv_mfma_i32_16x16x64_i8 v[28:31], v[224:227], v[172:175], v[28:31]\n\t" \
    "ds_read_b128 v[196:199], %[pa1] offset:14336\n\tv_mfma_i32_16x16x64_i8 v[24:27], v[228:231], v[176:179], v[24:27]\n\t" \
    "v_mfma_i32_16x16x64_i8 v[28:31], v[228:231], v[180:183], v[28:31]\n\tds_read_b128 v[200:203], v64 offset:4096\n\ts_waitcnt lgkmcnt(5)\n\t" \
    "v_mfma_i32_16x16x64_i8 v[32:35], v[232:235], v[152:155], v[32:35]\n\ts_mov_b32 %[dA], 0\n\t" \
    "v_mfma_i32_16x16x64_i8 v[36:39], v[232:235], v[156:159], v[36:39]\n\tds_read_b128 v[204:207], v65 offset:4096\n\t" \
    "v_mfma_i32_16x16x64_i8 v[32:35], v[236:239], v[160:163], v[32:35]\n\tv_mfma_i32_16x16x64_i8 v[36:39], v[236:239], v[164:167], v[36:39]\n\t" \
    "ds_read_b128 v[208:211], v64 offset:14336\n\tv_mfma_i32_16x16x64_i8 v[32:35], v[240:243], v[168:171], v[32:35]\n\tv_cvt_f32_i32 v24, v24\n\t" \
    "v_cvt_f32_i32 v25, v25\n\tv_cvt_f32_i32 v28, v28\n\tv_mfma_i32_16x16x64_i8 v[36:39], v[240:243], v[172:175], v[36:39]\n\t" \
    "ds_read_b128 v[212:215], v65 offset:14336\n\tv_cvt_f32_i32 v29, v29\n\tv_mul_f32 v24, v76, v24\n\t" \
    "v_mfma_i32_16x16x64_i8 v[32:35], v[244:247], v[176:179], v[32:35]\n\tv_mul_f32 v28, v76, v28\n\tv_mul_f32 v25, v76, v25\n\tv_mul_f32 v29, v76, v29\n\t" \
    "v_mfma_i32_16x16x64_i8 v[36:39], v[244:247], v[180:183], v[36:39]\n\tds_read_b128 v[216:219], %[pa0] offset:6144\n\tv_maximum3_f32 v24, v24, 0, 0\n\t" \
    "v_maximum3_f32 v28, v28, 0, 0\n\ts_waitcnt lgkmcnt(5)\n\tv_mfma_i32_16x16x64_i8 v[40:43], v[184:187], v[120:123], v[40:43]\n\t" \
    "v_maximum3_f32 v25, v25, 0, 0\n\tv_maximum3_f32 v29, v29, 0, 0\n\tv_cvt_f32_i32 v26, v26\n\t" \
    "v_mfma_i32_16x16x64_i8 v[44:47], v[184:187], v[124:127], v[44:47]\n\tds_read_b128 v[220:223], %[pa1] offset:6144\n\t" \
    "buffer_store_dword v24, %[vo], %[srd], %[dA] offen sc1\n\tv_cvt_f32_i32 v27, v27\n\t" \
    "v_mfma_i32_16x16x64_i8 v[40:43], v[188:191], v[128:131], v[40:43]\n\tv_cvt_f32_i32 v30, v30\n\tv_cvt_f32_i32 v31, v31\n\t" \
    "buffer_store_dword v28, %[vo], %[srd], %[dA] offen offset:64 sc1\n\tv_mfma_i32_16x16x64_i8 v[44:47], v[188:191], v[132:135], v[44:47]\n\t" \
    "ds_read_b128 v[224:227], %[pa0] offset:16384\n\tv_mul_f32 v26, v76, v26\n\tv_mul_f32 v30, v76, v30\n\t" \
    "v_mfma_i32_16x16x64_i8 v[40:43], v[192:195], v[136:139], v[40:43]\n\tbuffer_store_dword v25, v69, %[srd], %[dA] offen sc1\n\t" \
    "v_mul_f32 v27, v76, v27\n\tv_mul_f32 v31, v76, v31\n\tv_mfma_i32_16x16x64_i8 v[44:47], v[192:195], v[140:143], v[44:47]\n\t" \
    "ds_read_b128 v[228:231], %[pa1] offset:16384\n\tbuffer_store_dword v29, v69, %[srd], %[dA] offen offset:64 sc1\n\tv_maximum3_f32 v26, v26, 0, 0\n\t" \
    "v_mfma_i32_16x16x64_i8 v[40:43], v[196:199], v[144:147], v[40:43]\n\tv_maximum3_f32 v30, v30, 0, 0\n\tv_maximum3_f32 v27, v27, 0, 0\n\t" \
    "v_maximum3_f32 v31, v31, 0, 0\n\tv_mfma_i32_16x16x64_i8 v[44:47], v[196:199], v[148:151], v[44:47]\n\tds_read_b128 v[232:235], v64 offset:6144\n\t" \
    "v_cvt_f32_i32 v32, v32\n\tbuffer_store_dword v26, v70, %[srd], %[dA] offen sc1\n\ts_waitcnt lgkmcnt(5)\n\t" \
    "v_mfma_i32_16x16x64_i8 v[40:43], v[200:203], v[152:155], v[40:43]\n\tv_cvt_f32_i32 v33, v33\n\tv_cvt_f32_i32 v36, v36\n\t" \
    "buffer_store_dword v30, v70, %[srd], %[dA] offen offset:64 sc1\n\tv_mfma_i32_16x16x64_i8 v[44:47], v[200:203], v[156:159], v[44:47]\n\t" \
    "ds_read_b128 v[236:239], v65 offset:6144\n\tv_cvt_f32_i32 v37, v37\n\tv_mul_f32 v32, v76, v32\n\t" \
    "v_mfma_i32_16x16x64_i8 v[40:43], v[204:207], v[160:163], v[40:43]\n\tbuffer_store_dword v27, v71, %[srd], %[dA] offen sc1\n\t" \
    "v_mul_f32 v36, v76, v36\n\tv_mul_f32 v33, v76, v33\n\tv_mfma_i32_16x16x64_i8 v[44:47], v[204:207], v[164:167], v[44:47]\n\t" \
    "ds_read_b128 v[240:243], v64 offset:16384\n\tbuffer_store_dword v31, v71, %[srd], %[dA] offen offset:64 sc1\n\tv_mul_f32 v37, v76, v37\n\t" \
    "v_mfma_i32_16x16x64_i8 v[40:43], v[208:211], v[168:171], v[40:43]\n\tv_maximum3_f32 v32, v32, 0, 0\n\tv_maximum3_f32 v36, v36, 0, 0\n\t" \
    "s_mov_b32 %[dA], 262144\n\tv_mfma_i32_16x16x64_i8 v[44:47], v[208:211], v[172:175], v[44:47]\n\tds_read_b128 v[244:247], v65 offset:16384\n\t" \
    "v_maximum3_f32 v33, v33, 0, 0\n\tv_maximum3_f32 v37, v37, 0, 0\n\tv_mfma_i32_16x16x64_i8 v[40:43], v[212:215], v[176:179], v[40:43]\n\t" \
    "v_cvt_f32_i32 v34, v34\n\tbuffer_store_dword v32, %[vo], %[srd], %[dA] offen sc1\n\tv_cvt_f32_i32 v35, v35\n\t" \
    "v_mfma_i32_16x16x64_i8 v[44:47], v[212:215], v[180:183], v[44:47]\n\tv_cvt_f32_i32 v38, v38\n\tv_cvt_f32_i32 v39, v39\n\t" \
    "buffer_store_dword v36, %[vo], %[srd], %[dA] offen offset:64 sc1\n\ts_waitcnt lgkmcnt(4)\n\t" \
    "v_mfma_i32_16x16x64_i8 v[48:51], v[216:219], v[120:123], v[48:51]\n\tv_mul_f32 v34, v76, v34\n\tv_mul_f32 v38, v76, v38\n\t" \
    "buffer_store_dword v33, v69, %[srd], %[dA] offen sc1\n\tv_mfma_i32_16x16x64_i8 v[52:55], v[216:219], v[124:127], v[52:55]\n\t" \
    "v_mul_f32 v35, v76, v35\n\tv_mul_f32 v39, v76, v39\n\tv_maximum3_f32 v34, v34, 0, 0\n\t" \
    "v_mfma_i32_16x16x64_i8 v[48:51], v[220:223], v[128:131], v[48:51]\n\tbuffer_store_dword v37, v69, %[srd], %[dA] offen offset:64 sc1\n\t" \
    "v_maximum3_f32 v38, v38, 0, 0\n\tv_maximum3_f32 v35, v35, 0, 0\n\tv_mfma_i32_16x16x64_i8 v[52:55], v[220:223], v[132:135], v[52:55]\n\t" \
    "v_maximum3_f32 v39, v39, 0, 0\n\tv_cvt_f32_i32 v40, v40\n\tbuffer_store_dword v34, v70, %[srd], %[dA] offen sc1\n\t" \
    "v_mfma_i32_16x16x64_i8 v[48:51], v[224:227], v[136:139], v[48:51]\n\tv_cvt_f32_i32 v41, v41\n\tv_cvt_f32_i32 v44, v44\n\tv_cvt_f32_i32 v45, v45\n\t" \
    "v_mfma_i32_16x16x64_i8 v[52:55], v[224:227], v[140:143], v[52:55]\n\tbuffer_store_dword v38, v70, %[srd], %[dA] offen offset:64 sc1\n\t" \
    "v_mul_f32 v40, v76, v40\n\tv_mul_f32 v44, v76, v44\n\tv_mfma_i32_16x16x64_i8 v[48:51], v[228:231], v[144:147], v[48:51]\n\tv_mul_f32 v41, v76, v41\n\t" \
    "buffer_store_dword v35, v71, %[srd], %[dA] offen sc1\n\tv_mul_f32 v45, v76, v45\n\t" \
    "v_mfma_i32_16x16x64_i8 v[52:55], v[228:231], v[148:151], v[52:55]\n\tv_maximum3_f32 v40, v40, 0, 0\n\tv_maximum3_f32 v44, v44, 0, 0\n\t" \
    "buffer_store_dword v39, v71, %[srd], %[dA] offen offset:64 sc1\n\ts_waitcnt lgkmcnt(0)\n\t" \
    "v_mfma_i32_16x16x64_i8 v[48:51], v[232:235], v[152:155], v[48:51]\n\tv_maximum3_f32 v41, v41, 0, 0\n\tv_maximum3_f32 v45, v45, 0, 0\n\t" \
    "s_mov_b32 %[dA], 524288\n\tv_mfma_i32_16x16x64_i8 v[52:55], v[232:235], v[156:159], v[52:55]\n\tv_cvt_f32_i32 v42, v42\n\t" \
    "buffer_store_dword v40, %[vo], %[srd], %[dA] offen sc1\n\tv_cvt_f32_i32 v43, v43\n\t" \
    "v_mfma_i32_16x16x64_i8 v[48:51], v[236:239], v[160:163], v[48:51]\n\tv_cvt_f32_i32 v46, v46\n\tv_cvt_f32_i32 v47, v47\n\t" \
    "buffer_store_dword v44, %[vo], %[srd], %[dA] offen offset:64 sc1\n\tv_mfma_i32_16x16x64_i8 v[52:55], v[236:239], v[164:167], v[52:55]\n\t" \
    "v_mul_f32 v42, v76, v42\n\tv_mul_f32 v46, v76, v46\n\tv_mul_f32 v43, v76, v43\n\tv_mfma_i32_16x16x64_i8 v[48:51], v[240:243], v[168:171], v[48:51]\n\t" \
    "buffer_store_dword v41, v69, %[srd], %[dA] offen sc1\n\tv_mul_f32 v47, v76, v47\n\tv_maximum3_f32 v42, v42, 0, 0\n\t" \
    "v_mfma_i32_16x16x64_i8 v[52:55], v[240:243], v[172:175], v[52:55]\n\tv_maximum3_f32 v46, v46, 0, 0\n\t" \
    "buffer_store_dword v45, v69, %[srd], %[dA] offen offset:64 sc1\n\tv_maximum3_f32 v43, v43, 0, 0\n\t" \
    "v_mfma_i32_16x16x64_i8 v[48:51], v[244:247], v[176:179], v[48:51]\n\tv_maximum3_f32 v47, v47, 0, 0\n\t" \
    "v_mfma_i32_16x16x64_i8 v[52:55], v[244:247], v[180:183], v[52:55]\n\tbuffer_store_dword v42, v70, %[srd], %[dA] offen sc1\n\t" \
    "buffer_store_dword v46, v70, %[srd], %[dA] offen offset:64 sc1\n\tbuffer_store_dword v43, v71, %[srd], %[dA] offen sc1\n\t" \
    "buffer_store_dword v47, v71, %[srd], %[dA] offen offset:64 sc1\n\ts_mov_b32 %[dA], 786432\n\ts_nop 4\n\tv_cvt_f32_i32 v48, v48\n\t" \
    "v_cvt_f32_i32 v49, v49\n\tv_cvt_f32_i32 v52, v52\n\tv_cvt_f32_i32 v53, v53\n\tv_mul_f32 v48, v76, v48\n\tv_mul_f32 v52, v76, v52\n\t" \
    "v_mul_f32 v49, v76, v49\n\tv_mul_f32 v53, v76, v53\n\tv_maximum3_f32 v48, v48, 0, 0\n\tv_maximum3_f32 v52, v52, 0, 0\n\t" \
    "v_maximum3_f32 v49, v49, 0, 0\n\tv_maximum3_f32 v53, v53, 0, 0\n\tv_cvt_f32_i32 v50, v50\n\tbuffer_store_dword v48, %[vo], %[srd], %[dA] offen sc1\n\t" \
    "v_cvt_f32_i32 v51, v51\n\tv_cvt_f32_i32 v54, v54\n\tv_cvt_f32_i32 v55, v55\n\tv_mul_f32 v50, v76, v50\n\t" \
    "buffer_store_dword v52, %[vo], %[srd], %[dA] offen offset:64 sc1\n\tv_mul_f32 v54, v76, v54\n\tv_mul_f32 v51, v76, v51\n\tv_mul_f32 v55, v76, v55\n\t" \
    "v_maximum3_f32 v50, v50, 0, 0\n\tbuffer_store_dword v49, v69, %[srd], %[dA] offen sc1\n\tv_maximum3_f32 v54, v54, 0, 0\n\t" \
    "v_maximum3_f32 v51, v51, 0, 0\n\tv_maximum3_f32 v55, v55, 0, 0\n\tbuffer_store_dword v53, v69, %[srd], %[dA] offen offset:64 sc1\n\t" \
    "buffer_store_dword v50, v70, %[srd], %[dA] offen sc1\n\tbuffer_store_dword v54, v70, %[srd], %[dA] offen offset:64 sc1\n\t" \
    "buffer_store_dword v51, v71, %[srd], %[dA] offen sc1\n\tbuffer_store_dword v55, v71, %[srd], %[dA] offen offset:64 sc1\n\t"
#define V8(a) "v" #a "0", "v" #a "1", "v" #a "2", "v" #a "3", "v" #a "4", "v" #a "5", "v" #a "6", "v" #a "7"
#define V10(a) V8(a), "v" #a "8", "v" #a "9"

template <int NU>
__device__ __forceinline__ void k_loop(unsigned pW0, unsigned pW1, unsigned pA0, unsigned pA1, u32x4 srdO, unsigned vO, float scale,
                                       unsigned vTmo) {
  unsigned C = 0, wrapsA = 0x4924u, wrapsB = 0x492u, dA, dB;
  asm volatile(
    ".irp r,24,26,28,30,32,34,36,38,40,42,44,46,48,50,52,54,56,58,60,62\n\tv_mov_b64 v[\\r:\\r+1], 0\n\t.endr\n\t"
    "s_branch 2f\n\t"
    "1:\n\t"
    HALF_AE(0)
    "3:\n\t"
    HALF_B(0, PICK(dB, wrapsB, 0, wrapB, stepB), PICK(dA, wrapsA, 0, wrapA, aslot), "")
    HALF_AO(1) HALF_B(1, ADV(pw0, dB), ADV(pw1, dB), ADV(pa0, dA), ADV(pa1, dA), "s_add_u32 %[C], %[C], 1\n\t")
    HALF_AE(2) HALF_B(2, PICK(dB, wrapsB, 1, wrapB, stepB), PICK(dA, wrapsA, 1, wrapA, aslot),
                         "s_lshr_b32 %[wrapsA], %[wrapsA], 2\n\ts_lshr_b32 %[wrapsB], %[wrapsB], 2\n\t")
    HALF_AO(3) HALF_B(3, ADV(pw0, dB), ADV(pw1, dB), ADV(pa0, dA), ADV(pa1, dA), "s_bitcmp0_b32 %[special], %[C]\n\t")
    "s_cbranch_scc1 1b\n\t"
    "s_cmp_eq_u32 %[C], 5\n\t"
    "s_cbranch_scc1 6f\n\t"
    "s_cmp_lg_u32 %[C], 3\n\t"
    "s_cbranch_scc1 4f\n\t"
    REG_CHUNK(120, NOX, 0, "")
    "s_branch 1b\n\t"
    "6:\n\t"
    REG_CHUNK(184, LASTW, 8, "v_add_u32 %[pw0], 2*%[stepB], %[pw0]\n\tv_add_u32 %[pw1], 2*%[stepB], %[pw1]\n\ts_lshr_b32 %[wrapsB], %[wrapsB], 1\n\t")
    "s_branch 1b\n\t"
    "4:\n\t"
    MF(c00, WB0(92), XU(92, 0))
    "s_cmp_eq_u32 %[C], 7\n\t"
    MF(c01, WB1(92), XU(92, 0))
    "s_waitcnt lgkmcnt(0)\n\t"
    "s_cbranch_scc1 5f\n\t"
    MF(c11, WB1(92), XU(92, 1)) MF(c10, WB0(92), XU(92, 1))
    MF(c20, WB0(92), XU(92, 2)) MF(c21, WB1(92), XU(92, 2)) MF(c31, WB1(92), XU(92, 3)) MF(c30, WB0(92), XU(92, 3))
    IF5(MF(c40, WB0(92), XU(92, 4)) MF(c41, WB1(92), XU(92, 4)))
    "v_add_u32 %[pw0], 2*%[stepB], %[pw0]\n\tv_add_u32 %[pw1], 2*%[stepB], %[pw1]\n\t"
    PASS(4) PASS(5) PASS(6) PASS(7) PASS(8) PASS(9) PASS(10) PASS(11)
    "v_subrev_u32 %[pw0], 2*%[stepB], %[pw0]\n\tv_subrev_u32 %[pw1], 2*%[stepB], %[pw1]\n\t"
    "2:\n\t"
    "s_barrier\n\t"
    TRL(64, 0, pw0, 0, 0) TRL(64, 2, pw0, 1024, 0) RX(64, 0, pa0, 0) TRL(64, 4, pw1, 0, 0) TRL(64, 6, pw1, 1024, 0)
    RX(64, 1, pa0, 0) RX(64, 2, pa0, 0) RX(64, 3, pa0, 0) IF5(RX(64, 4, pa0, 0))
    "s_branch 3b\n\t"
    "5:\n\t"
    "s_barrier\n\t"
    IF5(FINAL5) IF4(FINAL4)
    : [pw0] "+v"(pW0), [pw1] "+v"(pW1), [pa0] "+v"(pA0), [pa1] "+v"(pA1), [sc] "+v"(scale),
      [C] "+&s"(C), [wrapsA] "+&s"(wrapsA), [wrapsB] "+&s"(wrapsB), [dA] "=&s"(dA), [dB] "=&s"(dB)
    : [nu] "i"(NU), [stepB] "i"(BSLOT), [aslot] "i"(ASLOT), [wrapA] "s"(-4 * (int)ASLOT), [wrapB] "s"(-4 * (int)BSLOT),
      [special] "s"((1u << 1) | (1u << 3) | (1u << 5) | (1u << 7)), [srd] "s"(srdO), [vo] "v"(vO), [vtmo] "v"(vTmo)
    : "memory", "scc", "vcc", "v24", "v25", "v26", "v27", "v28", "v29", V10(3), V10(4), V10(5), V10(6),
      V10(7), V10(8), V10(9), V10(10), V10(11), V10(12), V10(13), V10(14), V10(15), V10(16), V10(17), V10(18), V10(19),
      V10(20), V10(21), V10(22), V10(23), V8(24));
}
static_assert(BSLOT == 16384 && ASLOT == 10240 && NN == 4096, "offsets in the generated schedule");

struct Flags {
  u32x4 srd;
  unsigned vF;
  unsigned vSnap, snap;
  unsigned vTmo;
  unsigned call, t0;
  unsigned long long seen;
  unsigned long long near;
};
__device__ __forceinline__ unsigned lds_word(unsigned addr) { return *(volatile LDSP unsigned*)(size_t)addr; }
__device__ __forceinline__ void poll(const Flags& f, unsigned long long want) {
  asm volatile("s_andn2_b64 exec, %3, %4\n\ts_cbranch_scc0 9f\n\ts_mov_b32 m0, %2\n\ts_nop 0\n\t"
               "buffer_load_dword %0, %1, 0 offen sc0 sc1 lds\n\t9:\n\ts_mov_b64 exec, -1"
               :: "v"(f.vF), "s"(f.srd), "s"(f.snap), "s"(want), "s"(f.seen) : "memory", "scc");
}
template <int IMM, int BAR>
__device__ __forceinline__ void gate(Flags& f, unsigned base, unsigned bit) {
  unsigned long long ts;
  asm volatile(
    "s_add_u32 m0, %[base], %[imm]\n\t"
    "s_bitcmp1_b64 %[seen], %[bit]\n\t"
    ".if %[bar]\n\ts_barrier\n\t.endif\n\t"
    "s_cbranch_scc0 7f\n\t"
    "6:\n\t"
    ".subsection 1\n\t"
    "7:\n\t"
    "s_mov_b32 m0, %[snap]\n\t"
    "8:\n\t"
    "ds_read_b32 v63, %[vsnap]\n\t"
    "s_lshl_b64 exec, %[near], %[bit]\n\t"
    "s_cbranch_scc0 5f\n\t"
    "buffer_load_dword %[vf], %[srd], 0 offen sc0 sc1 lds\n\t"
    "5:\n\t"
    "s_mov_b64 exec, -1\n\t"
    "s_memrealtime %[ts]\n\t"
    "s_waitcnt lgkmcnt(0)\n\t"
    "v_cmp_le_u32 %[seen], %[call], v63\n\t"
    "s_bitcmp1_b64 %[seen], %[bit]\n\t"
    "s_cbranch_scc1 9f\n\t"
    "s_waitcnt vmcnt(0)\n\t"
    "s_lshl_b64 %[ts], %[ts], 32\n\ts_lshr_b64 vcc, %[ts], 32\n\t"
    "s_sub_u32 vcc_lo, vcc_lo, %[t0]\n\t"
    "s_cmp_lt_u32 vcc_lo, %[dl]\n\t"
    "s_cbranch_scc1 8b\n\t"
    "v_mov_b32 v63, 1\n\t"
    "ds_write_b32 %[vtmo], v63\n\t"
    "s_waitcnt lgkmcnt(0)\n\t"
    "9:\n\t"
    "s_add_u32 m0, %[base], %[imm]\n\t"
    "s_branch 6b\n\t"
    ".subsection 0"
    : [seen] "+&s"(f.seen), [ts] "=&s"(ts)
    : [base] "s"(base), [imm] "i"(IMM), [bar] "i"(BAR), [bit] "s"(bit), [snap] "s"(f.snap), [vsnap] "v"(f.vSnap),
      [vf] "v"(f.vF), [srd] "s"(f.srd), [call] "s"(f.call), [t0] "s"(f.t0), [dl] "i"((unsigned)DEADLINE_TICKS), [vtmo] "v"(f.vTmo), [near] "s"(f.near)
    : "memory", "scc", "vcc", "v63");
}

template <int SLOT>
__device__ __forceinline__ void w_burst(u32x4 srdW, const unsigned (&vW)[4], unsigned stage_off, unsigned lB) {
  unsigned so;
  asm volatile("s_add_u32 m0, %1, %2\n\ts_mov_b32 %0, %3" : "=s"(so) : "s"(lB), "i"(SLOT * BSLOT), "s"(stage_off) : "memory", "scc");
  dma<4>(srdW, vW[0], vW[1], vW[2], vW[3], so);
}

__device__ __forceinline__ unsigned lane_of(unsigned tab, int i) {
  unsigned r = __builtin_amdgcn_readlane(tab, i);
  asm volatile("" : "+s"(r));
  return r;
}
constexpr unsigned long long WINDOW = 0x5555555555555555ull >> (64 - 2 * POLLW);

template <int NX>
__device__ __forceinline__ void load_loop(u32x4 srdW, const unsigned (&vW)[4], const unsigned (&vS)[3], unsigned tabW, unsigned tabX,
                                          unsigned tabL, unsigned lA, unsigned h, unsigned long long window, bool early, Flags& f) {
  unsigned sx0 = lane_of(tabX, 8), sx1 = lane_of(tabX, 9), bit = 16 + h, la = lA + 2 * ASLOT;
  auto look = [&]() {
    f.seen = __ballot(lds_word(f.vSnap) >= f.call);
  };
  auto X_pair = [&](auto Sc) {
    const unsigned a = lane_of(tabX, Sc.value), b = lane_of(tabX, Sc.value + 1);
    asm volatile("s_nop 1");
    gate<(Sc.value % NA) * ASLOT, 0>(f, lA, 2 * Sc.value + h);
    dma_sys<NX>(f.srd, vS[0], vS[1], vS[2], vS[2], a);
    gate<(Sc.value % NA + 1) * ASLOT, 0>(f, lA, 2 * Sc.value + 2 + h);
    dma_sys<NX>(f.srd, vS[0], vS[1], vS[2], vS[2], b);
  };
  if (__builtin_expect(early, 1)) {
    wait_vm<8>();
    look();
    BARRIER();
  } else {
    look();
    X_pair(IC(4));
    look();
    if (((unsigned)f.seen >> h & 0x5000) == 0x5000) {
      X_pair(IC(6));
      wait_vm<2 * NX>();
      BARRIER();
    } else {
      wait_vm<0>();
      BARRIER();
      X_pair(IC(6));
    }
  }
  auto tick = [&](auto Tc, int it) {
    constexpr int T = Tc.value;
    gate<0, T != 4>(f, la, bit);
    dma_sys<NX>(f.srd, vS[0], vS[1], vS[2], vS[2], sx0);
    if constexpr (T != 4) poll(f, window << bit);
    gate<ASLOT, 0>(f, la, bit + 2);
    unsigned wv;
    asm volatile("ds_read_b32 %0, %1" : "=v"(wv) : "v"(f.vSnap) : "memory");
    dma_sys<NX>(f.srd, vS[0], vS[1], vS[2], vS[2], sx1);
    __builtin_amdgcn_sched_barrier(0);
    bit += 4;
    sx0 = lane_of(tabX, 2 * (T + it) + 2); sx1 = lane_of(tabX, 2 * (T + it) + 3);
    const unsigned sw = lane_of(tabW, w_ring(T) + it), sl = lane_of(tabL, w_ring(T) + it);
    la = la == lA + 4 * ASLOT ? lA : la + 2 * ASLOT;
    asm volatile("" : "+s"(la), "+s"(bit));
    asm volatile("s_waitcnt lgkmcnt(0)\n\tv_cmp_le_u32 %0, %1, %2" : "=s"(f.seen) : "s"(f.call), "v"(wv) : "memory");
    __builtin_amdgcn_sched_barrier(0);
    w_burst<0>(srdW, vW, sw, sl);
    if constexpr (w_cnt(T) == 2) w_burst<0>(srdW, vW, lane_of(tabW, w_ring(T) + 1), lane_of(tabL, w_ring(T) + 1));
    if constexpr (T == 4) { wait_vm<2 * NX + 4>(); poll(f, window >> 4 << bit); }
    else if constexpr (T != 7) wait_vm<in_flight(T, NX)>();
    else wait_vm<in_flight_loop(NX)>();
    __builtin_amdgcn_sched_barrier(0);
  };
  tick(IC(4), 0); tick(IC(5), 0); tick(IC(6), 0);
#pragma unroll 1
  for (int it = 0; it < 4; ++it) tick(IC(7), it);
  tick(IC(11), 0); tick(IC(12), 0); tick(IC(13), 0); tick(IC(14), 0);
  const unsigned sw = lane_of(tabW, 19), sl = lane_of(tabL, 19);
  BARRIER();
  w_burst<0>(srdW, vW, sw, sl);
  gate<0, 0>(f, la, bit);
  dma_sys<NX>(f.srd, vS[0], vS[1], vS[2], vS[2], sx0);
  gate<ASLOT, 0>(f, la, bit + 2);
  dma_sys<NX>(f.srd, vS[0], vS[1], vS[2], vS[2], sx1);
  wait_vm<2 * NX>();
  BARRIER();
  wait_vm<0>();
  BARRIER();
}

__global__ __attribute__((flatten)) __launch_bounds__(NTHREADS, 2) void a2a_gemm_kernel(Params p) {
  extern __shared__ __attribute__((aligned(16))) unsigned char smem[];
  const int b = blockIdx.x;
  const int wave = __builtin_amdgcn_readfirstlane(threadIdx.x >> 6);
  const int lane = threadIdx.x & 63;
  const int b7 = (b & 31) * 37 >> 8, dj = (b & 31) - 7 * b7;
  asm volatile("s_load_dword s100, %0, 0x0\n\ts_load_dword s101, %0, %1"
               :: "s"(__builtin_amdgcn_kernarg_segment_ptr()), "s"(64 + 16 * dj) : "memory", "s100", "s101");
#define ARGS() u32x8 ka, kb; u32x4 kl; \
  asm volatile("s_load_dwordx8 %0, %3, 0x0\n\ts_load_dwordx8 %1, %3, 0x20\n\ts_load_dwordx4 %2, %3, %4\n\ts_waitcnt lgkmcnt(0)" \
               : "=&s"(ka), "=&s"(kb), "=&s"(kl) : "s"(__builtin_amdgcn_kernarg_segment_ptr()), "s"(64 + 16 * dj) : "memory"); \
  const int rank = kb[4]; unsigned char* const myws = ptr_of<unsigned char>(ka[4], ka[5]); \
  unsigned* const cnt = (unsigned*)(myws + CNT_OFF + b * 128); (void)rank; (void)cnt
#define SRD(lo, hi, bytes) u32x4{lo, (hi) & 0xffffu, bytes, 0x00020000u}
  if (wave == 3 && b >= NSEND_WG && b < NSEND_WG + 8 * 4) {
    unsigned long long pc; int len;
    asm volatile("s_getpc_b64 %0\n.Lcode_pf:\n\ts_mov_b32 %1, .Lcode_pf_end-.Lcode_pf\n\t.subsection 2\n.Lcode_pf_end:\n\t.subsection 0" : "=s"(pc), "=s"(len));
    touch(SRD((unsigned)pc, (unsigned)(pc >> 32), (unsigned)max(len, 0)), lane * 128, ((b - NSEND_WG) >> 3) * 8192, (unsigned)(size_t)(LDSP unsigned char*)smem + LDS_JUNK);
  }

  if (b >= NSEND_WG) {
  const int g = (b - NSEND_WG) >> 3, xcd = b & 7;
  const int ntl = g * 37 >> 8, mt = g - NMT * ntl, nt = xcd * 4 + ntl;
  const int ub = unit_begin(mt), nu = mt < 4 ? 5 : 4;
  volatile LDSP unsigned* const ltmo = (volatile LDSP unsigned*)(smem + LDS_TMO);
  const unsigned lds0 = (unsigned)(size_t)(LDSP unsigned char*)smem;
  const unsigned plan = pick7<plan_word>(mt);
  auto src_of = [&](int rank, int js) { return (int)(rank + (plan >> (4 * js))) & 7; };
  auto q_of = [&](int S) { return (S + (S < 4 ? mt + ntl : S < 8 ? (nu == 5 ? 0 : 2) : ((ROTMASK >> (S >> 2)) & 1) ? ntl : 0)) & 3; };

  if (wave < 4) {
  const int fg = lane >> 4, fq = (lane & 15) >> 1, fp = lane & 1;
  const unsigned pW0 = lds0 + LDS_B + 2 * BSLOT + (16 * fg + fq) * 128 + (((2 * wave) ^ (((fq >> 1) & 3) | ((fg & 1) << 2))) << 4) + 8 * fp;
  const unsigned pA0 = lds0 + LDS_A + (lane & 15) * 128 + ((fg ^ fq) << 4);
  const unsigned vO = ((ub * 16 + fg * 4) * NN + nt * 128 + wave * 32 + (lane & 15)) * 4;
  ARGS();
  const u32x4 srdO = SRD(ka[6], ka[7], (unsigned)MS * NN * 4);
  {
    const u32x4 srdW = SRD(ka[2], ka[3], 4096u * NN);
    const int slow = (3 - (int)(ka[2] >> 7) - xcd * 4) & 7;
    if (wave == 0) {
      touch(srdO, (ub * 16 + (lane >> 5) * (nu * 16 - 1)) * NN * 4 + nt * 512, 0, lds0 + LDS_JUNK);
      touch(SRD(ka[4], ka[5], CNT_OFF), (lane & 7) * SRC_STRIDE + SRC_FLAG + 8192, 0, lds0 + LDS_JUNK);
    } else if (wave == 1) {
      if (slow < 4 && slow != ntl) {
        const int part = (ntl - slow - 1) & 3;
        const unsigned col = (xcd * 4 + slow) * 128;
        touch(srdW, min(part * 43 + min(lane, 42), 127) * NN, (unsigned)(src_of(rank, 1) * KC + q_of(5) * 128) * NN + col, lds0 + LDS_JUNK);
        if (nu == 4) {
          touch(srdW, lane * NN, (unsigned)(src_of(rank, 3) * KC + 128 + part * 128) * NN + col, lds0 + LDS_JUNK);
          touch(srdW, lane * NN, (unsigned)(src_of(rank, 3) * KC + 192 + part * 128) * NN + col, lds0 + LDS_JUNK);
        }
      }
    } else if (nu == 4) {
      const unsigned row = min(((mt - 4) * 2 + wave - 2) * 43 + min(lane, 42), 255);
      touch(srdW, row * NN, (unsigned)(((rank + 4) & 7) * KC + 256) * NN + nt * 128, lds0 + LDS_JUNK);
    }
  }
  if (wave == 2) ((volatile LDSP unsigned*)(smem + LDS_SNAP))[lane] = 0;
  if (wave == 3) *ltmo = 0;
  float sx, sw;
  asm volatile("s_load_dword %0, %2, 0x0\n\ts_load_dword %1, %3, 0x0\n\ts_waitcnt lgkmcnt(0)"
               : "=&s"(sx), "=&s"(sw) : "s"(ptr_of<float>(kb[0], kb[1])), "s"(ptr_of<float>(kb[2], kb[3])) : "memory");
  if (nu == 5) k_loop<5>(pW0, pW0 ^ 16, pA0, pA0 ^ 64, srdO, vO, sx * sw, lds0 + LDS_TMO);
  else k_loop<4>(pW0, pW0 ^ 16, pA0, pA0 ^ 64, srdO, vO, sx * sw, lds0 + LDS_TMO);
  return;
  }

  {
    const int lw = wave - 4;
    const unsigned swz = (lane & 7) ^ ((lane >> 4) & 3);
    unsigned vW[4];
#pragma unroll
    for (int i = 0; i < 4; ++i) vW[i] = (lw * 32 + (lane >> 3)) * NN + nt * 128 + ((swz ^ ((i & 2) << 1)) << 4) + i * (8 * NN - 1024);
    unsigned lB = lds0 + LDS_B + lw * 4096, q0 = q_of(0) << 19;
    asm volatile("" : "+v"(vW[0]), "+v"(vW[1]), "+v"(vW[2]), "+v"(vW[3]));
    asm volatile("" : "+s"(lB), "+s"(q0));
    ARGS();
    const u32x4 srdW = SRD(ka[2], ka[3], 4096u * NN);
    w_burst<2>(srdW, vW, ((unsigned)rank << 21) + q0, lB);
    __builtin_amdgcn_sched_barrier(0);
    __builtin_amdgcn_s_setprio(LOADER_PRIO);
    const bool odd = lw & 1, a3 = nu == 5 && !odd;
    const int a0 = lw * 2 + (nu == 5 ? (lw + 1) >> 1 : 0);
    const int arow = ub * 16 + a0 * 8;
    unsigned vX[3], vS[3];
#pragma unroll
    for (int i = 0; i < 3; ++i) {
      vX[i] = (arow + (lane >> 3)) * KC + ((swz ^ (((a0 + i) & 1) << 2)) << 4) + i * (8 * KC - 1024);
      vS[i] = xrow_off(arow + 8 * i + (lane >> 3)) + ((swz ^ (((a0 + i) & 1) << 2)) << 4) - i * 1024;
    }
    const unsigned lA = lds0 + LDS_A + a0 * 1024;
    const u32x4 srdX = SRD(ka[0], ka[1], 4096u * KC);
    const u32x4 srdS = SRD(ka[4], ka[5], CNT_OFF);
    auto w_off = [&](int S) { return (unsigned)(src_of(rank, S >> 2) * 4 + q_of(S)) << 19; };
    auto W_load = [&](auto Sc, auto slot) { w_burst<slot.value>(srdW, vW, w_off(Sc.value), lB); };
    auto X_local = [&](auto Sc) {
      constexpr int S = Sc.value;
      const unsigned so = (unsigned)(rank * MS) * KC + q_of(S) * 128;
      set_m0<S * ASLOT>(lA);
      if (a3) dma<3>(srdX, vX[0], vX[1], vX[2], vX[2], so); else dma<2>(srdX, vX[0], vX[1], vX[2], vX[2], so);
    };
    X_local(IC(0));
    W_load(IC(4), IC(0)); W_load(IC(1), IC(3)); X_local(IC(1));
    asm volatile("s_load_dword s100, %0, 0x0" :: "s"(cnt) : "memory", "s100");
    W_load(IC(2), IC(4)); X_local(IC(2));
    wait_vm<6>(); BARRIER();
    W_load(IC(3), IC(5)); X_local(IC(3)); W_load(IC(5), IC(1));
    const unsigned call = sload(cnt) + 1;
    const int Sl = lane & 31;
    Flags f;
    f.srd = srdS; f.call = call; f.t0 = now32(); f.seen = 0; f.near = odd ? 0x55 : 0;
    f.snap = lds0 + LDS_SNAP; f.vSnap = f.snap + lane * 4; f.vTmo = lds0 + LDS_TMO;
    f.vF = src_of(rank, lane >> 3) * SRC_STRIDE + SRC_FLAG + (mt * 4 + q_of(lane >> 1)) * 128 + (lane & 1) * 4;
    const int Sr = min(Sl < 8 ? Sl + 4 : Sl < 12 ? Sl + 8 : Sl < 14 ? Sl + 18 : Sl + 10, 31);
    const unsigned tabW = w_off(Sr), tabL = lB + (Sl % NB) * BSLOT;
    const unsigned tabX = src_of(rank, Sl >> 2) * SRC_STRIDE + (call & 1) * XBUF + xq_off(q_of(Sl));
    const unsigned h = lw >> 1;
    const unsigned long long window = odd ? WINDOW : 0;
    const unsigned sx4 = lane_of(tabX, 4), sx5 = lane_of(tabX, 5), sx6 = lane_of(tabX, 6), sx7 = lane_of(tabX, 7);
    auto X_first = [&](auto nx) {
      set_m0<4 * ASLOT>(lA); dma_sys<nx.value>(srdS, vS[0], vS[1], vS[2], vS[2], sx4);
      set_m0<5 * ASLOT>(lA); dma_sys<nx.value>(srdS, vS[0], vS[1], vS[2], vS[2], sx5);
      set_m0<0 * ASLOT>(lA); dma_sys<nx.value>(srdS, vS[0], vS[1], vS[2], vS[2], sx6);
      set_m0<1 * ASLOT>(lA); dma_sys<nx.value>(srdS, vS[0], vS[1], vS[2], vS[2], sx7);
    };
    wait_vm<4>(); BARRIER();
    W_load(IC(12), IC(2)); W_load(IC(13), IC(3));
    bool early = false;
    static_for<8>([&](auto kc) {
      constexpr int n = 7 + kc.value;
      unsigned so = w_off(n < 8 ? n + 8 : n < 12 ? n + 12 : n - 6);
      asm volatile("" : "+s"(so));
      if constexpr (n > 11) { if (early) wait_vm<16>(); else wait_vm<8>(); }
      else wait_vm<n == 7 ? 4 : 8>();
      if constexpr (n == 7) { BARRIER(); W_load(IC(14), IC(4)); set_m0<(2 + n % 4) * BSLOT>(lB); }
      else set_m0_barrier<(2 + n % 4) * BSLOT>(lB);
      unsigned word = 0;
      if constexpr (n == 11 || n == 13) word = lds_word(f.vSnap);
      dma<4>(srdW, vW[0], vW[1], vW[2], vW[3], so);
      if constexpr (n == 11 || n == 13) {
        f.seen = __ballot(word >= call);
        if (!early && ((unsigned)f.seen >> h & 0x5500) == 0x5500) {
          if (a3) X_first(IC(3)); else X_first(IC(2));
          early = true;
        }
      }
      if constexpr (n == 9) poll(f, window << (8 + h));
      if constexpr (n == 11 || n == 13) poll(f, (window | window << PW2) << (8 + h));
    });
    if (a3) load_loop<3>(srdW, vW, vS, tabW, tabX, tabL, lA, h, window << 4, early, f);
    else load_loop<2>(srdW, vW, vS, tabW, tabX, tabL, lA, h, window << 4, early, f);
    if (threadIdx.x == 256) __hip_atomic_store(cnt, call, __ATOMIC_RELAXED, __HIP_MEMORY_SCOPE_AGENT);
    return;
  }

  }

  {
    const int v = b7 + 4 * wave;
    if (b < 28 && v < 28) {
      const unsigned serve = pick7<send_word>(6 - dj);
      unsigned row0[2], q[2], fl[2], so[2]; bool big[2];
#pragma unroll
      for (int n = 0; n < 2; ++n) {
        const int pi = v + 28 * n, h = pi & 1;
        const int mt = (serve >> (4 * (pi >> 3))) & 7;
        q[n] = ((pi >> 1) + (mt + dj == 10 ? 2 : 0)) & 3;
        big[n] = mt < 4;
        row0[n] = unit_begin(mt) * 16 + h * (big[n] ? 40 : 32);
        fl[n] = (mt * 4 + q[n]) * 128 + h * 4;
        so[n] = row0[n] * KC + q[n] * 128;
      }
      unsigned vL[5];
#pragma unroll
      for (int i = 0; i < 5; ++i) vL[i] = ((lane >> 3) + 8 * i) * KC + (lane & 7) * 16;
      asm volatile("" : "+v"(vL[0]), "+v"(vL[1]), "+v"(vL[2]), "+v"(vL[3]), "+v"(vL[4]));
      asm volatile("" : "+s"(so[0]));
      ARGS();
      const u32x4 srdR = SRD(kl[0], kl[1], (unsigned)MS * KC);
      u32x4 r[5];
      volatile LDSP unsigned* const first = (volatile LDSP unsigned*)(smem + LDS_TMO);
      unsigned call, tref;
      const unsigned tz = now32();
      if (wave < FIRSTW) {
#pragma unroll
        for (int i = 0; i < 5; ++i) load16(r[i], srdR, vL[i], so[0]);
        BARRIER();
        touch(SRD(kl[2], kl[3], wave ? 0u : SRC_STRIDE), (lane & 3) * (SRC_STRIDE / 3 - 64 & ~63u), 0, (unsigned)(size_t)(LDSP unsigned char*)smem);
        call = sload(cnt) + 1;
        asm volatile("s_waitcnt vmcnt(1)" : "+v"(r[0]), "+v"(r[1]), "+v"(r[2]), "+v"(r[3]), "+v"(r[4]) :: "memory");
        tref = (tz + min(now32() - tz, CAP)) | 1;
        if (wave == 0) *first = tref;
        else do tref = __builtin_amdgcn_readfirstlane(*first); while (!tref);
      } else {
        *first = 0;
        asm volatile("s_waitcnt lgkmcnt(0)" ::: "memory");
        BARRIER();
        call = sload(cnt) + 1;
        do { __builtin_amdgcn_s_sleep(1); tref = __builtin_amdgcn_readfirstlane(*first); } while (!tref);
        while ((int)(now32() - tref) < (v - 4 * FIRSTW) * PACE / 4 + 3) __builtin_amdgcn_s_sleep(1);
        load80(r, srdR, vL, so[0]);
      }
      unsigned char* const dreg = ptr_of<unsigned char>(kl[2], kl[3]);
      unsigned char* const dbase = dreg + (call & 1) * XBUF + xrow_off(lane >> 3) + (lane & 7) * 16;
#pragma unroll
      for (int n = 0; n < 2; ++n) {
        while ((int)(now32() - tref) < (v + 28 * n - LEADP) * PACE / 4 - 1) __builtin_amdgcn_s_sleep(1);
        unsigned char* d = dbase + xq_off(q[n]);
#pragma unroll
        for (int i = 0; i < 4; ++i) store16_sys(d + xrow_off(row0[n] + 8 * i), r[i]);
        if (big[n]) store16_sys(d + xrow_off(row0[n] + 32), r[4]);
        wait_vm<0>();
        if (lane == 0) store4_sys(dreg + SRC_FLAG + fl[n], call);
        if (n == 0) load80(r, srdR, vL, so[1]);
      }
      BARRIER();
      if (threadIdx.x == 0) __hip_atomic_store(cnt, call, __ATOMIC_RELAXED, __HIP_MEMORY_SCOPE_AGENT);
    }
  }
}

}

extern "C" void kernel_launch(int rank, int nranks, ncclComm_t comm,
                              hipStream_t stream,
                              void* const* d_in, const int* in_sizes, int n_in,
                              void* d_out, int out_size,
                              const int* in_dtypes, int out_dtype,
                              void* const* d_out_all,
                              void* const* d_ws_all, size_t ws_bytes) {
  if (nranks != NR || (unsigned)rank >= (unsigned)NR || ws_bytes < WS_USED || n_in < 4 || out_size != MS * NN) {
    fprintf(stderr, "unexpected problem geometry\n");
    return;
  }
  static bool init = false;
  if (!init) {
    HIP_CHECK(hipFuncSetAttribute((const void*)a2a_gemm_kernel, hipFuncAttributeMaxDynamicSharedMemorySize, LDS_BYTES));
    init = true;
  }
  Params p;
  p.x = (const signed char*)d_in[0];
  p.w = (const signed char*)d_in[1];
  p.sx = (const float*)d_in[2];
  p.sw = (const float*)d_in[3];
  p.out = (float*)d_out;
  p.myws = (unsigned char*)d_ws_all[rank];
  for (int j = 0; j < NR; ++j) {
    const int dst = (rank + 1 + j) % NR;
    p.link[j].rows = p.x + (size_t)dst * MS * KC;
    p.link[j].region = (unsigned char*)d_ws_all[dst] + (size_t)rank * SRC_STRIDE;
  }
  p.rank = rank;
  p.pad[0] = p.pad[1] = p.pad[2] = 0;
  a2a_gemm_kernel<<<256, NTHREADS, LDS_BYTES, stream>>>(p);
  HIP_CHECK(hipGetLastError());
}
```
